```python
import numpy as np

kernel_name = "gpu_gemm_a2a_m4096_k4096_n32768_bf16_silu"


ABSMAX_RTOL = 0.008

COMPUTE_DTYPE = "bfloat16"
EPILOGUE = "silu"
IN_QUANT = None
OUT_QUANT = None
IN_DTYPES = {"x": "bfloat16", "w_mat": "bfloat16"}
WORLD_SIZE = 8

def _to_bf16(x):
    u = np.ascontiguousarray(x, dtype=np.float32).view(np.uint32)
    return ((u + 0x7FFF + ((u >> 16) & 1)) & 0xFFFF0000).view(np.float32)

def setup_inputs():

    x = _to_bf16(np.random.randn(4096, 4096).astype(np.float32))
    w_mat = _to_bf16((np.random.randn(4096, 32768) / np.sqrt(4096)).astype(np.float32))
    return {"x": x, "w_mat": w_mat}

def _epilogue(y):
    return (y / (1.0 + np.exp(-np.clip(y, -60.0, 60.0)))).astype(np.float32)

def reference(x, w_mat):
    y = (x @ w_mat).astype(np.float32)
    return _epilogue(y)
```

```cpp
#include <hip/hip_runtime.h>
#include <rccl/rccl.h>
#include <cstdio>

#define HIP_CHECK(expr) do { hipError_t _e = (expr); if (_e != hipSuccess) { \
    fprintf(stderr, "HIP error %s at %s:%d\n", hipGetErrorString(_e), __FILE__, __LINE__); \
    return; } } while (0)

typedef unsigned long long u64;
#define RTAB 0x2203ull
#define OWNX 3
#define ROT 0
constexpr int NR = 8;
constexpr unsigned LDSB = 131072;
constexpr size_t WS_CTR = 0;
constexpr size_t WS_TMO = 32768;
constexpr size_t WS_F1 = 65536;
constexpr size_t WS_F2 = 327680;
constexpr size_t WS_CLAIM = 393216;
constexpr size_t WS_STAMP = 524288;
constexpr size_t WS_X = 1u << 20;
constexpr size_t WS_END = WS_X + (8u << 20);

struct Params {
  const char* x; const char* w; char* out; char* myws; char* ws[NR]; int rank; unsigned tab; u64 rtab; unsigned pad, rot;
};

__global__ __launch_bounds__(256, 1) void fused_ag_gemm_silu(const Params p) {
  extern __shared__ __attribute__((aligned(16))) char lds[];
  const unsigned ldsb = (unsigned)(size_t)lds;
  asm volatile(
      "s_getpc_b64 s[92:93]\n\t"
      "CODE_%=:\n\t"
      "s_load_dwordx8 s[84:91], %0, 0\n\t"
      "s_load_dwordx2 s[94:95], %0, 96\n\t"
      "s_load_dwordx4 s[4:7], %0, 104\n\t"
      "s_load_dwordx16 s[16:31], %0, 32\n\t"
      "v_lshrrev_b32 v27, 6, %2\n\t"
      "v_and_b32 v10, 63, %2\n\t"
      "s_nop 0\n\t"
      "v_readfirstlane_b32 s11, v27\n\t"
      "v_mov_b32 v27, 0\n\t"
      "s_lshr_b32 s53, %1, 3\n\t"
      "s_xor_b32 s53, s53, 17\n\t"
      "s_xor_b32 s54, s11, 3\n\t"
      "s_or_b32 s33, s53, s54\n\t"
      "s_cbranch_scc1 NOINV_%=\n\t"
      "buffer_inv sc0 sc1\n\t"
      "NOINV_%=:\n\t"
      "v_lshlrev_b32 v30, 4, v10\n\t"
      "v_lshrrev_b32 v96, 3, v10\n\t"
      "v_and_b32 v97, 0x70, v30\n\t"
      "v_lshl_add_u32 v11, v96, 13, v97\n\t"
      "s_lshl_b32 s53, %1, 7\n\t"
      "v_mov_b32 v12, s53\n\t"
      "s_lshl_b32 s53, s11, 12\n\t"
      "s_add_u32 s51, s53, %3\n\t"
      "s_add_u32 s99, s51, 65536\n\t"
      "s_lshl_b32 s53, s11, 19\n\t"
      "s_and_b32 s54, s11, 1\n\t"
      "s_lshl_b32 s54, s54, 7\n\t"
      "v_lshrrev_b32 v100, 5, v10\n\t"
      "v_and_b32 v101, 0x1f0, v30\n\t"
      "v_lshlrev_b32 v102, 5, v100\n\t"
      "v_xor_b32 v101, v101, v102\n\t"
      "v_xor_b32 v101, s54, v101\n\t"
      "v_lshlrev_b32 v100, 16, v100\n\t"
      "v_add3_u32 v100, v100, v101, s53\n\t"
      "v_xor_b32 v6, 0, v100\n\t"
      "v_xor_b32 v7, 64, v100\n\t"
      "v_add_u32 v7, 130048, v7\n\t"
      "v_xor_b32 v8, 0, v100\n\t"
      "v_add_u32 v8, 260096, v8\n\t"
      "v_xor_b32 v9, 64, v100\n\t"
      "v_add_u32 v9, 390144, v9\n\t"
      "s_getreg_b32 s53, hwreg(HW_REG_XCC_ID, 0, 4)\n\t"
      "s_and_b32 s98, s53, 7\n\t"
      "s_lshl_b32 s98, s98, 2\n\t"
      "s_lshl_b32 s53, s53, 7\n\t"
      "s_add_u32 s10, s53, 32896\n\t"
      "v_mov_b32 v31, s10\n\t"
      "s_lshr_b32 s96, %1, 7\n\t"
      "s_bfe_u32 s97, %1, 0x40003\n\t"
      "v_mov_b32 v41, 1\n\t"
      "s_lshr_b32 s53, %1, 4\n\t"
      "s_mul_i32 s53, s53, 3\n\t"
      "s_add_u32 s53, s53, %1\n\t"
      "s_sub_u32 s53, s53, 9\n\t"
      "s_and_b32 s53, s53, 15\n\t"
      "s_cmp_lt_u32 %1, 64\n\t"
      "s_cselect_b32 s47, %1, s53\n\t"
      "s_and_b32 s53, s47, 3\n\t"
      "s_lshl_b32 s53, s53, 20\n\t"
      "s_and_b32 s54, s47, 4\n\t"
      "s_lshl_b32 s54, s54, 17\n\t"
      "s_add_u32 s53, s53, s54\n\t"
      "s_lshl_b32 s54, s11, 17\n\t"
      "s_add_u32 s53, s53, s54\n\t"
      "s_lshr_b32 s55, %1, 3\n\t"
      "s_lshr_b32 s54, s47, 3\n\t"
      "s_lshl_b32 s54, s54, 7\n\t"
      "v_add_u32 v98, s53, v11\n\t"
      "s_cmp_lt_u32 %1, 64\n\t"
      "s_cselect_b64 s[8:9], -1, 0\n\t"
      "s_mov_b64 exec, s[8:9]\n\t"
      "s_waitcnt lgkmcnt(0)\n\t"
      "s_add_u32 s54, s54, s7\n\t"
      "s_and_b32 s54, s54, 0x1fff\n\t"
      "v_add_u32 v98, s54, v98\n\t"
      "v_add_u32 v99, 0x10000, v98\n\t"
      "global_load_dwordx4 v[32:35], v98, s[84:85]\n\t"
      "global_load_dwordx4 v[36:39], v99, s[84:85]\n\t"
      "s_mov_b64 exec, -1\n\t"
      "global_load_dword v28, v12, s[90:91] sc0 sc1\n\t"
      "s_lshr_b32 s98, s95, s98\n\t"
      "s_and_b32 s98, s98, 7\n\t"
      "s_lshl_b32 s53, s55, 3\n\t"
      "s_add_u32 s53, s53, s98\n\t"
      "s_lshl_b32 s53, s53, 7\n\t"
      "s_add_u32 s53, s53, 393216\n\t"
      "v_mov_b32 v40, s53\n\t"
      "s_cmp_eq_u32 s11, 1\n\t"
      "s_cselect_b64 exec, 1, 0\n\t"
      "global_atomic_add v42, v40, v41, s[90:91] sc0 sc1\n\t"
      "s_mov_b64 exec, -1\n\t"
      "s_lshl_b32 s53, s97, 9\n\t"
      "s_add_u32 s12, s86, s53\n\t"
      "s_addc_u32 s13, s87, 0\n\t"
      "s_lshl_b32 s54, s94, 13\n\t"
      "s_add_u32 s53, s53, s54\n\t"
      "s_sub_u32 s14, 0x10000000, s53\n\t"
      "s_mov_b32 s15, 0x20000\n\t"
      "s_cmp_lg_u32 s8, 0\n\t"
      "s_cbranch_scc1 NOW_%=\n\t"
      "s_lshl_b32 s10, s98, 3\n\t"
      "s_lshr_b64 s[34:35], s[4:5], s10\n\t"
      "s_and_b32 s10, s34, 63\n\t"
      "s_lshl_b32 s49, s10, 22\n\t"
      "s_xor_b32 s53, s55, 17\n\t"
      "s_xor_b32 s54, s11, 3\n\t"
      "s_or_b32 s33, s53, s54\n\t"
      "s_cbranch_scc0 INV_%=\n\t"
      "INVB_%=:\n\t"
      "s_cmp_eq_u32 s55, 16\n\t"
      "s_cselect_b64 exec, -1, 0\n\t"
      "s_lshl_b32 s53, s11, 13\n\t"
      "v_lshl_add_u32 v98, v10, 7, s53\n\t"
      "v_min_u32 v98, END_%=-CODE_%=, v98\n\t"
      "global_load_dword v43, v98, s[92:93]\n\t"
      "s_mov_b64 exec, -1\n\t"
      "NOW_%=:\n\t"
      "s_sub_u32 s53, %1, 250\n\t"
      "s_cmp_lt_u32 s53, 4\n\t"
      "s_cselect_b32 s6, 6, 0\n\t"
      "s_xor_b32 s6, s6, %1\n\t"
      "s_and_b32 s53, %1, 1\n\t"
      "s_lshl_b32 s53, s53, 21\n\t"
      "s_and_b32 s54, s6, 2\n\t"
      "s_lshl_b32 s54, s54, 18\n\t"
      "s_add_u32 s53, s53, s54\n\t"
      "s_and_b32 s54, s11, 2\n\t"
      "s_lshl_b32 s54, s54, 19\n\t"
      "s_add_u32 s53, s53, s54\n\t"
      "s_and_b32 s54, s11, 1\n\t"
      "s_lshl_b32 s54, s54, 18\n\t"
      "s_add_u32 s53, s53, s54\n\t"
      "s_lshr_b32 s54, s6, 2\n\t"
      "s_lshl_b32 s54, s54, 7\n\t"
      "s_add_u32 s54, s54, s7\n\t"
      "s_and_b32 s54, s54, 0x1fff\n\t"
      "s_add_u32 s53, s53, s54\n\t"
      "v_add_u32 v11, s53, v11\n\t"
      "v_lshlrev_b32 v97, 4, v97\n\t"
      "v_lshl_add_u32 v17, v96, 4, v97\n\t"
      "s_mov_b32 s46, s98\n\t"
      "s_cmp_ge_u32 %1, 32\n\t"
      "s_cselect_b64 exec, -1, 0\n\t"
      "global_load_dwordx4 a[224:227], v11, s[84:85]\n\t"
      "v_add_u32 v11, 0x10000, v11\n\t"
      "global_load_dwordx4 a[228:231], v11, s[84:85]\n\t"
      "v_add_u32 v11, 0x10000, v11\n\t"
      "global_load_dwordx4 a[232:235], v11, s[84:85]\n\t"
      "v_add_u32 v11, 0x10000, v11\n\t"
      "global_load_dwordx4 a[236:239], v11, s[84:85]\n\t"
      "v_add_u32 v11, 0x10000, v11\n\t"
      "s_mov_b64 exec, -1\n\t"
      "s_cmp_eq_u32 s8, 0\n\t"
      "s_cbranch_scc1 SETUP_%=\n\t"
      "s_and_b32 s53, s47, 2\n\t"
      "s_lshl_b32 s33, s53, 20\n\t"
      "s_lshr_b32 s53, s47, 2\n\t"
      "s_lshl_b32 s53, s53, 14\n\t"
      "s_add_u32 s33, s33, s53\n\t"
      "s_and_b32 s53, s47, 1\n\t"
      "s_lshl_b32 s53, s53, 13\n\t"
      "s_add_u32 s33, s33, s53\n\t"
      "s_lshl_b32 s53, s11, 11\n\t"
      "s_add_u32 s33, s33, s53\n\t"
      "s_and_b32 s53, s47, 1\n\t"
      "s_mul_i32 s53, s53, 0\n\t"
      "s_xor_b32 s33, s33, s53\n\t"
      "s_and_b32 s53, s47, 2\n\t"
      "s_mul_i32 s53, s53, 3072\n\t"
      "s_xor_b32 s33, s33, s53\n\t"
      "s_lshl_b32 s55, s94, 1\n\t"
      "s_bfe_u32 s53, s47, 0x10001\n\t"
      "s_add_u32 s53, s53, s55\n\t"
      "s_lshl_b32 s53, s53, 5\n\t"
      "s_lshr_b32 s54, s47, 2\n\t"
      "s_lshl_b32 s54, s54, 1\n\t"
      "s_add_u32 s53, s53, s54\n\t"
      "s_and_b32 s54, s47, 1\n\t"
      "s_add_u32 s53, s53, s54\n\t"
      "s_lshl_b32 s53, s53, 7\n\t"
      "s_add_u32 s53, s53, 65536\n\t"
      "v_writelane_b32 v48, s16, 0\n\t"
      "v_writelane_b32 v49, s17, 0\n\t"
      "v_writelane_b32 v48, s18, 1\n\t"
      "v_writelane_b32 v49, s19, 1\n\t"
      "v_writelane_b32 v48, s20, 2\n\t"
      "v_writelane_b32 v49, s21, 2\n\t"
      "v_writelane_b32 v48, s22, 3\n\t"
      "v_writelane_b32 v49, s23, 3\n\t"
      "v_writelane_b32 v48, s24, 4\n\t"
      "v_writelane_b32 v49, s25, 4\n\t"
      "v_writelane_b32 v48, s26, 5\n\t"
      "v_writelane_b32 v49, s27, 5\n\t"
      "v_writelane_b32 v48, s28, 6\n\t"
      "v_writelane_b32 v49, s29, 6\n\t"
      "v_writelane_b32 v48, s30, 7\n\t"
      "v_writelane_b32 v49, s31, 7\n\t"
      "v_add_co_u32 v48, vcc, s53, v48\n\t"
      "v_addc_co_u32 v49, vcc, 0, v49, vcc\n\t"
      "s_waitcnt vmcnt(5)\n\t"
      "v_readfirstlane_b32 s53, v28\n\t"
      "s_nop 3\n\t"
      "s_not_b32 s53, s53\n\t"
      "s_and_b32 s53, s53, 1\n\t"
      "s_lshl_b32 s53, s53, 22\n\t"
      "s_add_u32 s33, s33, s53\n\t"
      "s_add_u32 s33, s33, 1048576\n\t"
      "s_add_u32 s32, s90, s33\n\t"
      "s_addc_u32 s33, s91, 0\n\t"
      "global_store_dwordx4 v17, v[32:35], s[32:33] sc0 sc1\n\t"
      "global_store_dwordx4 v17, v[36:39], s[32:33] offset:128 sc0 sc1\n\t"
      "SETUP_%=:\n\t"
      "s_lshr_b32 s53, s11, 1\n\t"
      "s_and_b32 s54, s11, 1\n\t"
      "s_mul_i32 s55, s53, 8192\n\t"
      "s_mul_i32 s10, s96, 6144\n\t"
      "s_xor_b32 s55, s55, s10\n\t"
      "v_add_u32 v1, s55, v30\n\t"
      "v_xor_b32 v16, 2048, v1\n\t"
      "v_lshrrev_b32 v96, 4, v10\n\t"
      "v_bfe_u32 v97, v10, 2, 2\n\t"
      "v_and_b32 v98, 3, v10\n\t"
      "v_lshlrev_b32 v99, 12, v96\n\t"
      "v_mul_u32_u24 v97, 544, v97\n\t"
      "v_lshlrev_b32 v98, 3, v98\n\t"
      "v_and_b32 v100, 1, v96\n\t"
      "v_lshlrev_b32 v100, 7, v100\n\t"
      "v_xor_b32 v97, v97, v98\n\t"
      "v_add_u32 v99, v99, v97\n\t"
      "s_lshl_b32 s55, s54, 8\n\t"
      "s_add_u32 s55, s55, 65536\n\t"
      "s_add_u32 s55, s55, %3\n\t"
      "v_xor_b32 v99, v99, v100\n\t"
      "v_add_u32 v99, s55, v99\n\t"
      "v_xor_b32 v18, 0, v99\n\t"
      "v_xor_b32 v19, 32, v99\n\t"
      "v_xor_b32 v20, 64, v99\n\t"
      "v_xor_b32 v21, 96, v99\n\t"
      "v_xor_b32 v22, 128, v99\n\t"
      "v_xor_b32 v23, 160, v99\n\t"
      "v_xor_b32 v24, 192, v99\n\t"
      "v_xor_b32 v25, 224, v99\n\t"
      "v_and_b32 v97, 15, v10\n\t"
      "v_lshlrev_b32 v97, 13, v97\n\t"
      "v_lshrrev_b32 v100, 2, v100\n\t"
      "v_lshrrev_b32 v98, 5, v10\n\t"
      "v_lshlrev_b32 v98, 4, v98\n\t"
      "v_add3_u32 v98, v97, v100, v98\n\t"
      "s_lshl_b32 s53, s53, 20\n\t"
      "s_lshl_b32 s55, s54, 8\n\t"
      "s_add_u32 s55, s55, s53\n\t"
      "v_add_u32 v26, s55, v98\n\t"
      "s_lshl_b32 s53, s11, 12\n\t"
      "s_add_u32 s53, s53, TAIL_%=-CODE_%=\n\t"
      "v_lshl_add_u32 v15, v10, 6, s53\n\t"
      "v_min_u32 v15, END_%=-CODE_%=, v15\n\t"
      "s_mov_b32 s48, 0\n\t"
      "s_mov_b32 s50, 28\n\t"
      "s_add_u32 s53, %3, 65520\n\t"
      "v_mov_b32 v14, s53\n\t"
      "s_cmp_eq_u32 s11, 1\n\t"
      "s_cselect_b64 s[56:57], -1, 0\n\t"
      "s_cmp_eq_u32 s11, 0\n\t"
      "s_cselect_b64 s[58:59], 0xff, 0\n\t"
      "s_and_b32 s54, %1, 1\n\t"
      "s_lshl_b32 s54, s54, 21\n\t"
      "s_lshr_b32 s55, %1, 1\n\t"
      "s_lshl_b32 s55, s55, 14\n\t"
      "s_add_u32 s6, s54, s55\n\t"
      "s_lshr_b32 s53, s11, 1\n\t"
      "s_mul_i32 s53, s53, 8192\n\t"
      "s_and_b32 s54, %1, 1\n\t"
      "s_mul_i32 s54, s54, 6144\n\t"
      "s_xor_b32 s53, s53, s54\n\t"
      "s_and_b32 s54, s11, 1\n\t"
      "s_mul_i32 s54, s54, 4096\n\t"
      "s_xor_b32 s53, s53, s54\n\t"
      "v_xor_b32 v17, s53, v17\n\t"
      "s_add_u32 s40, s90, 32768\n\t"
      "s_addc_u32 s41, s91, 0\n\t"
      "s_lshl_b32 s53, %1, 7\n\t"
      "s_add_u32 s42, s90, s53\n\t"
      "s_addc_u32 s43, s91, 0\n\t"
      "s_cmp_lg_u32 s8, 0\n\t"
      "s_cbranch_scc1 NOWE_%=\n\t"
      "s_add_u32 m0, s99, 0\n\t"
      "s_add_u32 s14, s12, s49\n\t"
      "s_addc_u32 s15, s13, 0\n\t"
      "s_nop 0\n\t"
      "global_load_lds_dwordx4 v6, s[14:15] offset:0 \n\t"
      "global_load_lds_dwordx4 v7, s[14:15] offset:1024 \n\t"
      "global_load_lds_dwordx4 v8, s[14:15] offset:2048 \n\t"
      "global_load_lds_dwordx4 v9, s[14:15] offset:3072 \n\t"
      "s_add_u32 s49, s49, 0x200000\n\t"
      "s_and_b32 s49, s49, 0xfffffff\n\t"
      "s_add_u32 m0, s99, 16384\n\t"
      "s_add_u32 s14, s12, s49\n\t"
      "s_addc_u32 s15, s13, 0\n\t"
      "s_nop 0\n\t"
      "global_load_lds_dwordx4 v6, s[14:15] offset:0 \n\t"
      "global_load_lds_dwordx4 v7, s[14:15] offset:1024 \n\t"
      "global_load_lds_dwordx4 v8, s[14:15] offset:2048 \n\t"
      "global_load_lds_dwordx4 v9, s[14:15] offset:3072 \n\t"
      "s_add_u32 s49, s49, 0x200000\n\t"
      "s_and_b32 s49, s49, 0xfffffff\n\t"
      "s_add_u32 m0, s99, 32768\n\t"
      "s_add_u32 s14, s12, s49\n\t"
      "s_addc_u32 s15, s13, 0\n\t"
      "s_nop 0\n\t"
      "global_load_lds_dwordx4 v6, s[14:15] offset:0 \n\t"
      "global_load_lds_dwordx4 v7, s[14:15] offset:1024 \n\t"
      "global_load_lds_dwordx4 v8, s[14:15] offset:2048 \n\t"
      "global_load_lds_dwordx4 v9, s[14:15] offset:3072 \n\t"
      "s_add_u32 s49, s49, 0x200000\n\t"
      "s_and_b32 s49, s49, 0xfffffff\n\t"
      "s_add_u32 m0, s99, 49152\n\t"
      "s_add_u32 s14, s12, s49\n\t"
      "s_addc_u32 s15, s13, 0\n\t"
      "s_nop 0\n\t"
      "global_load_lds_dwordx4 v6, s[14:15] offset:0 \n\t"
      "global_load_lds_dwordx4 v7, s[14:15] offset:1024 \n\t"
      "global_load_lds_dwordx4 v8, s[14:15] offset:2048 \n\t"
      "global_load_lds_dwordx4 v9, s[14:15] offset:3072 \n\t"
      "s_add_u32 s49, s49, 0x200000\n\t"
      "s_and_b32 s49, s49, 0xfffffff\n\t"
      "s_waitcnt vmcnt(21)\n\t"
      "NOWE_%=:\n\t"
      "v_add_u32 v28, 1, v28\n\t"
      "s_nop 1\n\t"
      "v_readfirstlane_b32 s52, v28\n\t"
      "s_nop 3\n\t"
      "s_and_b32 s53, s52, 1\n\t"
      "s_lshl_b32 s53, s53, 22\n\t"
      "s_add_u32 s53, s53, 1048576\n\t"
      "s_add_u32 s6, s6, s53\n\t"
      "s_add_u32 s86, s90, s6\n\t"
      "s_addc_u32 s87, s91, 0\n\t"
      "s_cmp_lg_u32 s8, 0\n\t"
      "s_cbranch_scc1 NOST1_%=\n\t"
      "s_branch NOST2_%=\n\t"
      "NOST1_%=:\n\t"
      "s_waitcnt vmcnt(6)\n\t"
      "NOST2_%=:\n\t"
      "s_cmp_lg_u32 s11, 1\n\t"
      "s_cbranch_scc1 SRC_%=\n\t"
      "s_mov_b64 exec, 1\n\t"
      "s_mov_b32 s54, 8\n\t"
      "CLAIML_%=:\n\t"
      "v_readfirstlane_b32 s53, v42\n\t"
      "s_nop 3\n\t"
      "s_add_u32 s53, s53, 1\n\t"
      "s_cmp_eq_u32 s53, s52\n\t"
      "s_cbranch_scc1 CLAIMED_%=\n\t"
      "global_atomic_sub v40, v41, s[90:91] sc1\n\t"
      "s_sub_u32 s54, s54, 1\n\t"
      "s_cbranch_scc1 CLAIMX_%=\n\t"
      "s_add_u32 s98, s98, 1\n\t"
      "s_and_b32 s98, s98, 7\n\t"
      "s_and_b32 s53, %1, 0xf8\n\t"
      "s_add_u32 s53, s53, s98\n\t"
      "s_lshl_b32 s53, s53, 7\n\t"
      "s_add_u32 s53, s53, 393216\n\t"
      "v_mov_b32 v40, s53\n\t"
      "global_atomic_add v42, v40, v41, s[90:91] sc0 sc1\n\t"
      "s_waitcnt vmcnt(0)\n\t"
      "s_branch CLAIML_%=\n\t"
      "CLAIMX_%=:\n\t"
      "global_store_dword v27, v28, s[40:41] sc1\n\t"
      "CLAIMED_%=:\n\t"
      "v_mov_b32 v29, s98\n\t"
      "ds_write_b32 v14, v29\n\t"
      "s_waitcnt lgkmcnt(0)\n\t"
      "SRC_%=:\n\t"
      "s_and_b64 exec, s[58:59], s[8:9]\n\t"
      "s_cmp_eq_u32 s8, 0\n\t"
      "s_cbranch_scc1 NOST3_%=\n\t"
      "s_waitcnt vmcnt(0)\n\t"
      "NOST3_%=:\n\t"
      "s_barrier\n\t"
      "global_store_dword v[48:49], v28, off sc0 sc1\n\t"
      "s_mov_b64 exec, -1\n\t"
      "ds_read_b32 v29, v14\n\t"
      "v_mov_b32 v2, v28\n\t"
      "v_mov_b32 v3, v28\n\t"
      "s_waitcnt lgkmcnt(0)\n\t"
      "v_readfirstlane_b32 s98, v29\n\t"
      "s_nop 3\n\t"
      "s_and_b32 s98, s98, 7\n\t"
      "s_lshl_b32 m0, s98, 1\n\t"
      "s_lshl_b32 s55, s98, 1\n\t"
      "s_add_u32 s55, s55, s96\n\t"
      "s_lshl_b32 s53, s55, 21\n\t"
      "s_lshl_b32 s54, s97, 9\n\t"
      "s_add_u32 s53, s53, s54\n\t"
      "s_add_u32 s44, s88, s53\n\t"
      "s_addc_u32 s45, s89, 0\n\t"
      "s_movrels_b64 s[88:89], s[16:17]\n\t"
      "s_and_b32 s53, s52, 1\n\t"
      "s_lshl_b32 s53, s53, 22\n\t"
      "s_add_u32 s53, s53, 1048576\n\t"
      "s_lshl_b32 s54, s96, 21\n\t"
      "s_add_u32 s54, s54, s53\n\t"
      "s_add_u32 s36, s88, s54\n\t"
      "s_addc_u32 s37, s89, 0\n\t"
      "s_xor_b32 s53, s46, s98\n\t"
      "s_or_b32 s53, s53, s8\n\t"
      "s_cbranch_scc0 NOX_%=\n\t"
      "s_lshl_b32 s10, s98, 3\n\t"
      "s_lshr_b64 s[34:35], s[4:5], s10\n\t"
      "s_and_b32 s10, s34, 63\n\t"
      "s_lshl_b32 s49, s10, 22\n\t"
      "s_add_u32 m0, s99, 0\n\t"
      "s_add_u32 s14, s12, s49\n\t"
      "s_addc_u32 s15, s13, 0\n\t"
      "s_nop 0\n\t"
      "global_load_lds_dwordx4 v6, s[14:15] offset:0 \n\t"
      "global_load_lds_dwordx4 v7, s[14:15] offset:1024 \n\t"
      "global_load_lds_dwordx4 v8, s[14:15] offset:2048 \n\t"
      "global_load_lds_dwordx4 v9, s[14:15] offset:3072 \n\t"
      "s_add_u32 s49, s49, 0x200000\n\t"
      "s_and_b32 s49, s49, 0xfffffff\n\t"
      "s_add_u32 m0, s99, 16384\n\t"
      "s_add_u32 s14, s12, s49\n\t"
      "s_addc_u32 s15, s13, 0\n\t"
      "s_nop 0\n\t"
      "global_load_lds_dwordx4 v6, s[14:15] offset:0 \n\t"
      "global_load_lds_dwordx4 v7, s[14:15] offset:1024 \n\t"
      "global_load_lds_dwordx4 v8, s[14:15] offset:2048 \n\t"
      "global_load_lds_dwordx4 v9, s[14:15] offset:3072 \n\t"
      "s_add_u32 s49, s49, 0x200000\n\t"
      "s_and_b32 s49, s49, 0xfffffff\n\t"
      "s_add_u32 m0, s99, 32768\n\t"
      "s_add_u32 s14, s12, s49\n\t"
      "s_addc_u32 s15, s13, 0\n\t"
      "s_nop 0\n\t"
      "global_load_lds_dwordx4 v6, s[14:15] offset:0 \n\t"
      "global_load_lds_dwordx4 v7, s[14:15] offset:1024 \n\t"
      "global_load_lds_dwordx4 v8, s[14:15] offset:2048 \n\t"
      "global_load_lds_dwordx4 v9, s[14:15] offset:3072 \n\t"
      "s_add_u32 s49, s49, 0x200000\n\t"
      "s_and_b32 s49, s49, 0xfffffff\n\t"
      "s_add_u32 m0, s99, 49152\n\t"
      "s_add_u32 s14, s12, s49\n\t"
      "s_addc_u32 s15, s13, 0\n\t"
      "s_nop 0\n\t"
      "global_load_lds_dwordx4 v6, s[14:15] offset:0 \n\t"
      "global_load_lds_dwordx4 v7, s[14:15] offset:1024 \n\t"
      "global_load_lds_dwordx4 v8, s[14:15] offset:2048 \n\t"
      "global_load_lds_dwordx4 v9, s[14:15] offset:3072 \n\t"
      "s_add_u32 s49, s49, 0x200000\n\t"
      "s_and_b32 s49, s49, 0xfffffff\n\t"
      "NOX_%=:\n\t"
      "s_and_b32 s53, %1, 1\n\t"
      "s_lshl_b32 s53, s53, 10\n\t"
      "s_lshr_b32 s54, %1, 1\n\t"
      "s_lshl_b32 s54, s54, 2\n\t"
      "s_add_u32 s53, s53, s54\n\t"
      "s_add_u32 s53, s53, 327680\n\t"
      "v_mov_b32 v4, s53\n\t"
      "s_cmp_eq_u32 s11, 1\n\t"
      "s_cbranch_scc1 W1C_%=\n\t"
      "s_waitcnt vmcnt(0)\n\t"
      "s_branch BP_%=\n\t"
      "INV_%=:\n\t"
      "s_waitcnt vmcnt(0)\n\t"
      "v_add_u32 v43, 1, v28\n\t"
      "s_mov_b64 exec, 1\n\t"
      "global_store_dword v31, v43, s[90:91] sc0 sc1\n\t"
      "s_mov_b64 exec, -1\n\t"
      "s_branch INVB_%=\n\t"
      "W1C_%=:\n\t"
      "s_lshl_b32 s53, s55, 12\n\t"
      "s_add_u32 s53, s53, 65536\n\t"
      "v_and_b32 v13, 31, v10\n\t"
      "v_lshl_add_u32 v13, v13, 7, s53\n\t"
      "v_mov_b32 v12, v13\n\t"
      "v_readfirstlane_b32 s54, v31\n\t"
      "s_lshl_b32 s53, s96, 10\n\t"
      "s_add_u32 s53, s53, 327680\n\t"
      "v_lshl_add_u32 v11, v10, 3, s53\n\t"
      "v_writelane_b32 v12, s54, 6\n\t"
      "s_mov_b64 exec, 127\n\t"
      "global_load_dword v44, v12, s[90:91] sc0 sc1\n\t"
      "s_sleep 4\n\t"
      "global_load_dword v45, v12, s[90:91] sc0 sc1\n\t"
      "s_sleep 4\n\t"
      "s_mov_b32 s54, 12\n\t"
      "FAST_%=:\n\t"
      "global_load_dword v46, v12, s[90:91] sc0 sc1\n\t"
      "s_waitcnt vmcnt(2)\n\t"
      "v_cmp_gt_u32 vcc, s52, v44\n\t"
      "s_nop 4\n\t"
      "s_cbranch_vccz POLLOK1_%=\n\t"
      "s_sleep 4\n\t"
      "global_load_dword v44, v12, s[90:91] sc0 sc1\n\t"
      "s_waitcnt vmcnt(2)\n\t"
      "v_cmp_gt_u32 vcc, s52, v45\n\t"
      "s_nop 4\n\t"
      "s_cbranch_vccz POLLOK1_%=\n\t"
      "s_sleep 4\n\t"
      "global_load_dword v45, v12, s[90:91] sc0 sc1\n\t"
      "s_waitcnt vmcnt(2)\n\t"
      "v_cmp_gt_u32 vcc, s52, v46\n\t"
      "s_nop 4\n\t"
      "s_cbranch_vccz POLLOK1_%=\n\t"
      "s_sleep 4\n\t"
      "s_sub_u32 s54, s54, 1\n\t"
      "s_cbranch_scc0 FAST_%=\n\t"
      "s_memrealtime s[34:35]\n\t"
      "s_waitcnt lgkmcnt(0)\n\t"
      "POLL1_%=:\n\t"
      "global_load_dword v2, v12, s[90:91] sc0 sc1\n\t"
      "global_load_dword v29, v27, s[40:41] sc1\n\t"
      "s_waitcnt vmcnt(1)\n\t"
      "v_cmp_gt_u32 vcc, s52, v2\n\t"
      "s_nop 4\n\t"
      "s_cbranch_vccz POLLOK1_%=\n\t"
      "s_memrealtime s[54:55]\n\t"
      "s_waitcnt vmcnt(0) lgkmcnt(0)\n\t"
      "v_readfirstlane_b32 s53, v29\n\t"
      "s_nop 3\n\t"
      "s_cmp_eq_u32 s53, s52\n\t"
      "s_cbranch_scc1 POLLTMO1_%=\n\t"
      "s_sub_u32 s54, s54, s34\n\t"
      "s_subb_u32 s55, s55, s35\n\t"
      "s_cmp_lg_u32 s55, 0\n\t"
      "s_cbranch_scc1 POLLTMO1_%=\n\t"
      "s_cmp_gt_u32 s54, 50000000\n\t"
      "s_cbranch_scc1 POLLTMO1_%=\n\t"
      "s_and_b32 s53, vcc_lo, 63\n\t"
      "s_cbranch_scc1 POLLMORE1_%=\n\t"
      "s_cmp_gt_u32 s54, 500\n\t"
      "s_cbranch_scc0 POLLMORE1_%=\n\t"
      "buffer_inv sc0 sc1\n\t"
      "s_waitcnt vmcnt(0)\n\t"
      "s_branch POLLOK1_%=\n\t"
      "POLLMORE1_%=:\n\t"
      "s_sleep 1\n\t"
      "s_branch POLL1_%=\n\t"
      "POLLTMO1_%=:\n\t"
      "global_store_dword v27, v28, s[40:41] sc1\n\t"
      "v_mov_b32 v2, v28\n\t"
      "v_mov_b32 v3, v28\n\t"
      "POLLOK1_%=:\n\t"
      "s_mov_b64 exec, -1\n\t"
      "BP_%=:\n\t"
      "s_barrier\n\t"
      "v_xor_b32 v14, 4096, v1\n\t"
      "v_xor_b32 v31, 6144, v1\n\t"
      "global_load_dwordx4 a[0:3], v1, s[36:37] offset:0 \n\t"
      "global_load_dwordx4 a[4:7], v1, s[36:37] offset:1024 \n\t"
      "global_load_dwordx4 a[8:11], v16, s[36:37] offset:0 \n\t"
      "global_load_dwordx4 a[12:15], v16, s[36:37] offset:1024 \n\t"
      "global_load_dwordx4 a[16:19], v14, s[36:37] offset:0 \n\t"
      "global_load_dwordx4 a[20:23], v14, s[36:37] offset:1024 \n\t"
      "global_load_dwordx4 a[24:27], v31, s[36:37] offset:0 \n\t"
      "global_load_dwordx4 a[28:31], v31, s[36:37] offset:1024 \n\t"
      "s_add_u32 s36, s36, 16384\n\t"
      "s_addc_u32 s37, s37, 0\n\t"
      "s_cmp_ge_u32 %1, 32\n\t"
      "s_cselect_b64 exec, -1, 0\n\t"
      "global_store_dwordx4 v17, a[224:227], s[86:87] offset:0 sc0 sc1\n\t"
      "global_store_dwordx4 v17, a[228:231], s[86:87] offset:128 sc0 sc1\n\t"
      "v_xor_b32 v17, 2048, v17\n\t"
      "global_store_dwordx4 v17, a[232:235], s[86:87] offset:0 sc0 sc1\n\t"
      "global_store_dwordx4 v17, a[236:239], s[86:87] offset:128 sc0 sc1\n\t"
      "s_mov_b64 exec, -1\n\t"
      "ds_read_b64_tr_b16 a[64:65], v18 offset:0\n\t"
      "ds_read_b64_tr_b16 a[66:67], v18 offset:2048\n\t"
      "s_waitcnt lgkmcnt(8)\n\t"
      "ds_read_b64_tr_b16 a[72:73], v19 offset:0\n\t"
      "ds_read_b64_tr_b16 a[74:75], v19 offset:2048\n\t"
      "s_waitcnt lgkmcnt(8)\n\t"
      "ds_read_b64_tr_b16 a[80:81], v20 offset:0\n\t"
      "ds_read_b64_tr_b16 a[82:83], v20 offset:2048\n\t"
      "s_waitcnt lgkmcnt(8)\n\t"
      "ds_read_b64_tr_b16 a[88:89], v21 offset:0\n\t"
      "ds_read_b64_tr_b16 a[90:91], v21 offset:2048\n\t"
      "s_waitcnt lgkmcnt(8)\n\t"
      "ds_read_b64_tr_b16 a[96:97], v22 offset:0\n\t"
      "ds_read_b64_tr_b16 a[98:99], v22 offset:2048\n\t"
      "s_waitcnt lgkmcnt(8)\n\t"
      "ds_read_b64_tr_b16 a[104:105], v23 offset:0\n\t"
      "ds_read_b64_tr_b16 a[106:107], v23 offset:2048\n\t"
      "s_waitcnt lgkmcnt(8)\n\t"
      "ds_read_b64_tr_b16 a[112:113], v24 offset:0\n\t"
      "ds_read_b64_tr_b16 a[114:115], v24 offset:2048\n\t"
      "s_waitcnt lgkmcnt(8)\n\t"
      "ds_read_b64_tr_b16 a[120:121], v25 offset:0\n\t"
      "ds_read_b64_tr_b16 a[122:123], v25 offset:2048\n\t"
      "s_waitcnt lgkmcnt(8)\n\t"
      "ds_read_b64_tr_b16 a[68:69], v18 offset:16384\n\t"
      "ds_read_b64_tr_b16 a[70:71], v18 offset:18432\n\t"
      "s_waitcnt lgkmcnt(8)\n\t"
      "ds_read_b64_tr_b16 a[76:77], v19 offset:16384\n\t"
      "ds_read_b64_tr_b16 a[78:79], v19 offset:18432\n\t"
      "s_waitcnt lgkmcnt(8)\n\t"
      "ds_read_b64_tr_b16 a[84:85], v20 offset:16384\n\t"
      "ds_read_b64_tr_b16 a[86:87], v20 offset:18432\n\t"
      "s_waitcnt lgkmcnt(8)\n\t"
      "ds_read_b64_tr_b16 a[92:93], v21 offset:16384\n\t"
      "ds_read_b64_tr_b16 a[94:95], v21 offset:18432\n\t"
      "s_waitcnt lgkmcnt(8)\n\t"
      "ds_read_b64_tr_b16 a[100:101], v22 offset:16384\n\t"
      "ds_read_b64_tr_b16 a[102:103], v22 offset:18432\n\t"
      "s_waitcnt lgkmcnt(8)\n\t"
      "ds_read_b64_tr_b16 a[108:109], v23 offset:16384\n\t"
      "ds_read_b64_tr_b16 a[110:111], v23 offset:18432\n\t"
      "s_waitcnt lgkmcnt(8)\n\t"
      "ds_read_b64_tr_b16 a[116:117], v24 offset:16384\n\t"
      "ds_read_b64_tr_b16 a[118:119], v24 offset:18432\n\t"
      "s_waitcnt lgkmcnt(8)\n\t"
      "ds_read_b64_tr_b16 a[124:125], v25 offset:16384\n\t"
      "ds_read_b64_tr_b16 a[126:127], v25 offset:18432\n\t"
      "s_waitcnt lgkmcnt(8)\n\t"
      "global_load_dwordx4 a[32:35], v1, s[36:37] offset:0 \n\t"
      "global_load_dwordx4 a[36:39], v1, s[36:37] offset:1024 \n\t"
      "global_load_dwordx4 a[40:43], v16, s[36:37] offset:0 \n\t"
      "global_load_dwordx4 a[44:47], v16, s[36:37] offset:1024 \n\t"
      "global_load_dwordx4 a[48:51], v14, s[36:37] offset:0 \n\t"
      "global_load_dwordx4 a[52:55], v14, s[36:37] offset:1024 \n\t"
      "global_load_dwordx4 a[56:59], v31, s[36:37] offset:0 \n\t"
      "global_load_dwordx4 a[60:63], v31, s[36:37] offset:1024 \n\t"
      "s_add_u32 s36, s36, 16384\n\t"
      "s_addc_u32 s37, s37, 0\n\t"
      "s_mov_b64 exec, s[56:57]\n\t"
      "global_load_dword v2, v13, s[90:91] sc0 sc1\n\t"
      "s_mov_b64 exec, -1\n\t"
      "s_waitcnt vmcnt(22) lgkmcnt(0)\n\t"
      "s_barrier\n\t"
      "s_waitcnt vmcnt(19)\n\t"
      "v_mfma_f32_16x16x32_bf16 v[32:35], a[64:67], a[0:3], 0\n\t"
      "s_add_u32 m0, s99, 0\n\t"
      "s_add_u32 s14, s12, s49\n\t"
      "s_addc_u32 s15, s13, 0\n\t"
      "v_mfma_f32_16x16x32_bf16 v[32:35], a[68:71], a[4:7], v[32:35]\n\t"
      "ds_read_b64_tr_b16 a[128:129], v18 offset:32768\n\t"
      "v_mfma_f32_16x16x32_bf16 v[36:39], a[72:75], a[0:3], 0\n\t"
      "global_load_lds_dwordx4 v6, s[14:15] offset:0 \n\t"
      "v_mfma_f32_16x16x32_bf16 v[36:39], a[76:79], a[4:7], v[36:39]\n\t"
      "v_mfma_f32_16x16x32_bf16 v[40:43], a[80:83], a[0:3], 0\n\t"
      "ds_read_b64_tr_b16 a[130:131], v18 offset:34816\n\t"
      "v_mfma_f32_16x16x32_bf16 v[40:43], a[84:87], a[4:7], v[40:43]\n\t"
      "v_mfma_f32_16x16x32_bf16 v[44:47], a[88:91], a[0:3], 0\n\t"
      "v_mfma_f32_16x16x32_bf16 v[44:47], a[92:95], a[4:7], v[44:47]\n\t"
      "ds_read_b64_tr_b16 a[136:137], v19 offset:32768\n\t"
      "v_mfma_f32_16x16x32_bf16 v[48:51], a[96:99], a[0:3], 0\n\t"
      "v_mfma_f32_16x16x32_bf16 v[48:51], a[100:103], a[4:7], v[48:51]\n\t"
      "v_mfma_f32_16x16x32_bf16 v[52:55], a[104:107], a[0:3], 0\n\t"
      "ds_read_b64_tr_b16 a[138:139], v19 offset:34816\n\t"
      "global_load_lds_dwordx4 v7, s[14:15] offset:1024 \n\t"
      "v_mfma_f32_16x16x32_bf16 v[52:55], a[108:111], a[4:7], v[52:55]\n\t"
      "v_mfma_f32_16x16x32_bf16 v[56:59], a[112:115], a[0:3], 0\n\t"
      "v_mfma_f32_16x16x32_bf16 v[56:59], a[116:119], a[4:7], v[56:59]\n\t"
      "ds_read_b64_tr_b16 a[144:145], v20 offset:32768\n\t"
      "v_mfma_f32_16x16x32_bf16 v[60:63], a[120:123], a[0:3], 0\n\t"
      "v_mfma_f32_16x16x32_bf16 v[60:63], a[124:127], a[4:7], v[60:63]\n\t"
      "s_waitcnt vmcnt(19)\n\t"
      "v_mfma_f32_16x16x32_bf16 v[64:67], a[64:67], a[8:11], 0\n\t"
      "ds_read_b64_tr_b16 a[146:147], v20 offset:34816\n\t"
      "global_load_dwordx4 a[0:3], v1, s[36:37] offset:0 \n\t"
      "v_mfma_f32_16x16x32_bf16 v[64:67], a[68:71], a[12:15], v[64:67]\n\t"
      "v_mfma_f32_16x16x32_bf16 v[68:71], a[72:75], a[8:11], 0\n\t"
      "global_load_dwordx4 a[4:7], v1, s[36:37] offset:1024 \n\t"
      "v_mfma_f32_16x16x32_bf16 v[68:71], a[76:79], a[12:15], v[68:71]\n\t"
      "ds_read_b64_tr_b16 a[152:153], v21 offset:32768\n\t"
      "v_mfma_f32_16x16x32_bf16 v[72:75], a[80:83], a[8:11], 0\n\t"
      "v_mfma_f32_16x16x32_bf16 v[72:75], a[84:87], a[12:15], v[72:75]\n\t"
      "v_mfma_f32_16x16x32_bf16 v[76:79], a[88:91], a[8:11], 0\n\t"
      "ds_read_b64_tr_b16 a[154:155], v21 offset:34816\n\t"
      "global_load_lds_dwordx4 v8, s[14:15] offset:2048 \n\t"
      "v_mfma_f32_16x16x32_bf16 v[76:79], a[92:95], a[12:15], v[76:79]\n\t"
      "v_mfma_f32_16x16x32_bf16 v[80:83], a[96:99], a[8:11], 0\n\t"
      "v_mfma_f32_16x16x32_bf16 v[80:83], a[100:103], a[12:15], v[80:83]\n\t"
      "ds_read_b64_tr_b16 a[160:161], v22 offset:32768\n\t"
      "v_mfma_f32_16x16x32_bf16 v[84:87], a[104:107], a[8:11], 0\n\t"
      "v_mfma_f32_16x16x32_bf16 v[84:87], a[108:111], a[12:15], v[84:87]\n\t"
      "v_mfma_f32_16x16x32_bf16 v[88:91], a[112:115], a[8:11], 0\n\t"
      "ds_read_b64_tr_b16 a[162:163], v22 offset:34816\n\t"
      "v_mfma_f32_16x16x32_bf16 v[88:91], a[116:119], a[12:15], v[88:91]\n\t"
      "v_mfma_f32_16x16x32_bf16 v[92:95], a[120:123], a[8:11], 0\n\t"
      "v_mfma_f32_16x16x32_bf16 v[92:95], a[124:127], a[12:15], v[92:95]\n\t"
      "ds_read_b64_tr_b16 a[168:169], v23 offset:32768\n\t"
      "s_waitcnt vmcnt(20)\n\t"
      "v_mfma_f32_16x16x32_bf16 v[96:99], a[64:67], a[16:19], 0\n\t"
      "global_load_dwordx4 a[8:11], v16, s[36:37] offset:0 \n\t"
      "v_mfma_f32_16x16x32_bf16 v[96:99], a[68:71], a[20:23], v[96:99]\n\t"
      "v_mfma_f32_16x16x32_bf16 v[100:103], a[72:75], a[16:19], 0\n\t"
      "ds_read_b64_tr_b16 a[170:171], v23 offset:34816\n\t"
      "global_load_lds_dwordx4 v9, s[14:15] offset:3072 \n\t"
      "v_mfma_f32_16x16x32_bf16 v[100:103], a[76:79], a[20:23], v[100:103]\n\t"
      "s_add_u32 s49, s49, 0x200000\n\t"
      "s_and_b32 s49, s49, 0xfffffff\n\t"
      "v_mfma_f32_16x16x32_bf16 v[104:107], a[80:83], a[16:19], 0\n\t"
      "global_load_dwordx4 a[12:15], v16, s[36:37] offset:1024 \n\t"
      "v_mfma_f32_16x16x32_bf16 v[104:107], a[84:87], a[20:23], v[104:107]\n\t"
      "ds_read_b64_tr_b16 a[176:177], v24 offset:32768\n\t"
      "v_mfma_f32_16x16x32_bf16 v[108:111], a[88:91], a[16:19], 0\n\t"
      "v_mfma_f32_16x16x32_bf16 v[108:111], a[92:95], a[20:23], v[108:111]\n\t"
      "v_mfma_f32_16x16x32_bf16 v[112:115], a[96:99], a[16:19], 0\n\t"
      "ds_read_b64_tr_b16 a[178:179], v24 offset:34816\n\t"
      "v_mfma_f32_16x16x32_bf16 v[112:115], a[100:103], a[20:23], v[112:115]\n\t"
      "v_mfma_f32_16x16x32_bf16 v[116:119], a[104:107], a[16:19], 0\n\t"
      "v_mfma_f32_16x16x32_bf16 v[116:119], a[108:111], a[20:23], v[116:119]\n\t"
      "ds_read_b64_tr_b16 a[184:185], v25 offset:32768\n\t"
      "v_mfma_f32_16x16x32_bf16 v[120:123], a[112:115], a[16:19], 0\n\t"
      "v_mfma_f32_16x16x32_bf16 v[120:123], a[116:119], a[20:23], v[120:123]\n\t"
      "v_mfma_f32_16x16x32_bf16 v[124:127], a[120:123], a[16:19], 0\n\t"
      "ds_read_b64_tr_b16 a[186:187], v25 offset:34816\n\t"
      "v_mfma_f32_16x16x32_bf16 v[124:127], a[124:127], a[20:23], v[124:127]\n\t"
      "s_waitcnt vmcnt(8)\n\t"
      "v_cmp_gt_u32 vcc, s52, v2\n\t"
      "s_waitcnt vmcnt(21)\n\t"
      "v_mfma_f32_16x16x32_bf16 a[192:195], a[64:67], a[24:27], 0\n\t"
      "global_load_dwordx4 a[16:19], v14, s[36:37] offset:0 \n\t"
      "v_mfma_f32_16x16x32_bf16 a[192:195], a[68:71], a[28:31], a[192:195]\n\t"
      "v_mfma_f32_16x16x32_bf16 a[196:199], a[72:75], a[24:27], 0\n\t"
      "global_load_dwordx4 a[20:23], v14, s[36:37] offset:1024 \n\t"
      "v_mfma_f32_16x16x32_bf16 a[196:199], a[76:79], a[28:31], a[196:199]\n\t"
      "s_cbranch_vccnz SLOW2_%=\n\t"
      "BACK2_%=:\n\t"
      "s_waitcnt vmcnt(31) lgkmcnt(0)\n\t"
      "s_barrier\n\t"
      "v_mfma_f32_16x16x32_bf16 a[200:203], a[80:83], a[24:27], 0\n\t"
      "v_mfma_f32_16x16x32_bf16 a[200:203], a[84:87], a[28:31], a[200:203]\n\t"
      "ds_read_b64_tr_b16 a[132:133], v18 offset:49152\n\t"
      "v_mfma_f32_16x16x32_bf16 a[204:207], a[88:91], a[24:27], 0\n\t"
      "s_add_u32 m0, s99, 16384\n\t"
      "s_add_u32 s14, s12, s49\n\t"
      "s_addc_u32 s15, s13, 0\n\t"
      "v_mfma_f32_16x16x32_bf16 a[204:207], a[92:95], a[28:31], a[204:207]\n\t"
      "ds_read_b64_tr_b16 a[134:135], v18 offset:51200\n\t"
      "v_mfma_f32_16x16x32_bf16 a[208:211], a[96:99], a[24:27], 0\n\t"
      "global_load_lds_dwordx4 v6, s[14:15] offset:0 \n\t"
      "v_mfma_f32_16x16x32_bf16 a[208:211], a[100:103], a[28:31], a[208:211]\n\t"
      "v_mfma_f32_16x16x32_bf16 a[212:215], a[104:107], a[24:27], 0\n\t"
      "v_mfma_f32_16x16x32_bf16 a[212:215], a[108:111], a[28:31], a[212:215]\n\t"
      "ds_read_b64_tr_b16 a[140:141], v19 offset:49152\n\t"
      "v_mfma_f32_16x16x32_bf16 a[216:219], a[112:115], a[24:27], 0\n\t"
      "global_load_lds_dwordx4 v7, s[14:15] offset:1024 \n\t"
      "v_mfma_f32_16x16x32_bf16 a[216:219], a[116:119], a[28:31], a[216:219]\n\t"
      "ds_read_b64_tr_b16 a[142:143], v19 offset:51200\n\t"
      "v_mfma_f32_16x16x32_bf16 a[220:223], a[120:123], a[24:27], 0\n\t"
      "v_mfma_f32_16x16x32_bf16 a[220:223], a[124:127], a[28:31], a[220:223]\n\t"
      "s_waitcnt vmcnt(19)\n\t"
      "v_mfma_f32_16x16x32_bf16 v[128:131], a[64:67], a[32:35], 0\n\t"
      "global_load_lds_dwordx4 v8, s[14:15] offset:2048 \n\t"
      "v_mfma_f32_16x16x32_bf16 v[128:131], a[68:71], a[36:39], v[128:131]\n\t"
      "ds_read_b64_tr_b16 a[148:149], v20 offset:49152\n\t"
      "v_mfma_f32_16x16x32_bf16 v[132:135], a[72:75], a[32:35], 0\n\t"
      "global_load_dwordx4 a[24:27], v31, s[36:37] offset:0 \n\t"
      "v_mfma_f32_16x16x32_bf16 v[132:135], a[76:79], a[36:39], v[132:135]\n\t"
      "ds_read_b64_tr_b16 a[150:151], v20 offset:51200\n\t"
      "v_mfma_f32_16x16x32_bf16 v[136:139], a[80:83], a[32:35], 0\n\t"
      "global_load_dwordx4 a[28:31], v31, s[36:37] offset:1024 \n\t"
      "v_mfma_f32_16x16x32_bf16 v[136:139], a[84:87], a[36:39], v[136:139]\n\t"
      "s_add_u32 s36, s36, 16384\n\t"
      "s_addc_u32 s37, s37, 0\n\t"
      "v_mfma_f32_16x16x32_bf16 v[140:143], a[88:91], a[32:35], 0\n\t"
      "v_mfma_f32_16x16x32_bf16 v[140:143], a[92:95], a[36:39], v[140:143]\n\t"
      "ds_read_b64_tr_b16 a[156:157], v21 offset:49152\n\t"
      "v_mfma_f32_16x16x32_bf16 v[144:147], a[96:99], a[32:35], 0\n\t"
      "global_load_lds_dwordx4 v9, s[14:15] offset:3072 \n\t"
      "v_mfma_f32_16x16x32_bf16 v[144:147], a[100:103], a[36:39], v[144:147]\n\t"
      "ds_read_b64_tr_b16 a[158:159], v21 offset:51200\n\t"
      "s_add_u32 s49, s49, 0x200000\n\t"
      "s_and_b32 s49, s49, 0xfffffff\n\t"
      "v_mfma_f32_16x16x32_bf16 v[148:151], a[104:107], a[32:35], 0\n\t"
      "s_add_u32 m0, s99, 32768\n\t"
      "s_add_u32 s14, s12, s49\n\t"
      "s_addc_u32 s15, s13, 0\n\t"
      "v_mfma_f32_16x16x32_bf16 v[148:151], a[108:111], a[36:39], v[148:151]\n\t"
      "v_mfma_f32_16x16x32_bf16 v[152:155], a[112:115], a[32:35], 0\n\t"
      "global_load_lds_dwordx4 v6, s[14:15] offset:0 \n\t"
      "v_mfma_f32_16x16x32_bf16 v[152:155], a[116:119], a[36:39], v[152:155]\n\t"
      "ds_read_b64_tr_b16 a[164:165], v22 offset:49152\n\t"
      "v_mfma_f32_16x16x32_bf16 v[156:159], a[120:123], a[32:35], 0\n\t"
      "v_mfma_f32_16x16x32_bf16 v[156:159], a[124:127], a[36:39], v[156:159]\n\t"
      "ds_read_b64_tr_b16 a[166:167], v22 offset:51200\n\t"
      "s_waitcnt vmcnt(22)\n\t"
      "v_mfma_f32_16x16x32_bf16 v[160:163], a[64:67], a[40:43], 0\n\t"
      "global_load_dwordx4 a[32:35], v1, s[36:37] offset:0 \n\t"
      "v_mfma_f32_16x16x32_bf16 v[160:163], a[68:71], a[44:47], v[160:163]\n\t"
      "v_mfma_f32_16x16x32_bf16 v[164:167], a[72:75], a[40:43], 0\n\t"
      "global_load_dwordx4 a[36:39], v1, s[36:37] offset:1024 \n\t"
      "v_mfma_f32_16x16x32_bf16 v[164:167], a[76:79], a[44:47], v[164:167]\n\t"
      "ds_read_b64_tr_b16 a[172:173], v23 offset:49152\n\t"
      "v_mfma_f32_16x16x32_bf16 v[168:171], a[80:83], a[40:43], 0\n\t"
      "global_load_lds_dwordx4 v7, s[14:15] offset:1024 \n\t"
      "v_mfma_f32_16x16x32_bf16 v[168:171], a[84:87], a[44:47], v[168:171]\n\t"
      "ds_read_b64_tr_b16 a[174:175], v23 offset:51200\n\t"
      "v_mfma_f32_16x16x32_bf16 v[172:175], a[88:91], a[40:43], 0\n\t"
      "v_mfma_f32_16x16x32_bf16 v[172:175], a[92:95], a[44:47], v[172:175]\n\t"
      "v_mfma_f32_16x16x32_bf16 v[176:179], a[96:99], a[40:43], 0\n\t"
      "global_load_lds_dwordx4 v8, s[14:15] offset:2048 \n\t"
      "v_mfma_f32_16x16x32_bf16 v[176:179], a[100:103], a[44:47], v[176:179]\n\t"
      "ds_read_b64_tr_b16 a[180:181], v24 offset:49152\n\t"
      "v_mfma_f32_16x16x32_bf16 v[180:183], a[104:107], a[40:43], 0\n\t"
      "v_mfma_f32_16x16x32_bf16 v[180:183], a[108:111], a[44:47], v[180:183]\n\t"
      "ds_read_b64_tr_b16 a[182:183], v24 offset:51200\n\t"
      "v_mfma_f32_16x16x32_bf16 v[184:187], a[112:115], a[40:43], 0\n\t"
      "v_mfma_f32_16x16x32_bf16 v[184:187], a[116:119], a[44:47], v[184:187]\n\t"
      "v_mfma_f32_16x16x32_bf16 v[188:191], a[120:123], a[40:43], 0\n\t"
      "v_mfma_f32_16x16x32_bf16 v[188:191], a[124:127], a[44:47], v[188:191]\n\t"
      "ds_read_b64_tr_b16 a[188:189], v25 offset:49152\n\t"
      "s_waitcnt vmcnt(24)\n\t"
      "v_mfma_f32_16x16x32_bf16 v[192:195], a[64:67], a[48:51], 0\n\t"
      "global_load_lds_dwordx4 v9, s[14:15] offset:3072 \n\t"
      "v_mfma_f32_16x16x32_bf16 v[192:195], a[68:71], a[52:55], v[192:195]\n\t"
      "ds_read_b64_tr_b16 a[190:191], v25 offset:51200\n\t"
      "s_add_u32 s49, s49, 0x200000\n\t"
      "s_and_b32 s49, s49, 0xfffffff\n\t"
      "v_mfma_f32_16x16x32_bf16 v[196:199], a[72:75], a[48:51], 0\n\t"
      "global_load_dwordx4 a[40:43], v16, s[36:37] offset:0 \n\t"
      "v_mfma_f32_16x16x32_bf16 v[196:199], a[76:79], a[52:55], v[196:199]\n\t"
      "v_mfma_f32_16x16x32_bf16 v[200:203], a[80:83], a[48:51], 0\n\t"
      "global_load_dwordx4 a[44:47], v16, s[36:37] offset:1024 \n\t"
      "v_mfma_f32_16x16x32_bf16 v[200:203], a[84:87], a[52:55], v[200:203]\n\t"
      "v_mfma_f32_16x16x32_bf16 v[204:207], a[88:91], a[48:51], 0\n\t"
      "v_mfma_f32_16x16x32_bf16 v[204:207], a[92:95], a[52:55], v[204:207]\n\t"
      "v_mfma_f32_16x16x32_bf16 v[208:211], a[96:99], a[48:51], 0\n\t"
      "v_mfma_f32_16x16x32_bf16 v[208:211], a[100:103], a[52:55], v[208:211]\n\t"
      "v_mfma_f32_16x16x32_bf16 v[212:215], a[104:107], a[48:51], 0\n\t"
      "v_mfma_f32_16x16x32_bf16 v[212:215], a[108:111], a[52:55], v[212:215]\n\t"
      "v_mfma_f32_16x16x32_bf16 v[216:219], a[112:115], a[48:51], 0\n\t"
      "v_mfma_f32_16x16x32_bf16 v[216:219], a[116:119], a[52:55], v[216:219]\n\t"
      "v_mfma_f32_16x16x32_bf16 v[220:223], a[120:123], a[48:51], 0\n\t"
      "v_mfma_f32_16x16x32_bf16 v[220:223], a[124:127], a[52:55], v[220:223]\n\t"
      "s_waitcnt vmcnt(25)\n\t"
      "v_mfma_f32_16x16x32_bf16 v[224:227], a[64:67], a[56:59], 0\n\t"
      "global_load_dwordx4 a[48:51], v14, s[36:37] offset:0 \n\t"
      "v_mfma_f32_16x16x32_bf16 v[224:227], a[68:71], a[60:63], v[224:227]\n\t"
      "v_mfma_f32_16x16x32_bf16 v[228:231], a[72:75], a[56:59], 0\n\t"
      "global_load_dwordx4 a[52:55], v14, s[36:37] offset:1024 \n\t"
      "v_mfma_f32_16x16x32_bf16 v[228:231], a[76:79], a[60:63], v[228:231]\n\t"
      "s_waitcnt vmcnt(19) lgkmcnt(0)\n\t"
      "s_barrier\n\t"
      "v_mfma_f32_16x16x32_bf16 v[232:235], a[80:83], a[56:59], 0\n\t"
      "v_mfma_f32_16x16x32_bf16 v[232:235], a[84:87], a[60:63], v[232:235]\n\t"
      "ds_read_b64_tr_b16 a[64:65], v18 offset:0\n\t"
      "s_and_b64 exec, s[58:59], 1\n\t"
      "global_store_dword v4, v28, s[90:91] sc0 sc1\n\t"
      "s_mov_b64 exec, -1\n\t"
      "v_mfma_f32_16x16x32_bf16 v[236:239], a[88:91], a[56:59], 0\n\t"
      "s_add_u32 m0, s99, 49152\n\t"
      "s_add_u32 s14, s12, s49\n\t"
      "s_addc_u32 s15, s13, 0\n\t"
      "v_mfma_f32_16x16x32_bf16 v[236:239], a[92:95], a[60:63], v[236:239]\n\t"
      "ds_read_b64_tr_b16 a[66:67], v18 offset:2048\n\t"
      "v_mfma_f32_16x16x32_bf16 v[240:243], a[96:99], a[56:59], 0\n\t"
      "global_load_lds_dwordx4 v6, s[14:15] offset:0 \n\t"
      "v_mfma_f32_16x16x32_bf16 v[240:243], a[100:103], a[60:63], v[240:243]\n\t"
      "v_mfma_f32_16x16x32_bf16 v[244:247], a[104:107], a[56:59], 0\n\t"
      "v_mfma_f32_16x16x32_bf16 v[244:247], a[108:111], a[60:63], v[244:247]\n\t"
      "ds_read_b64_tr_b16 a[72:73], v19 offset:0\n\t"
      "v_mfma_f32_16x16x32_bf16 v[248:251], a[112:115], a[56:59], 0\n\t"
      "global_load_lds_dwordx4 v7, s[14:15] offset:1024 \n\t"
      "v_mfma_f32_16x16x32_bf16 v[248:251], a[116:119], a[60:63], v[248:251]\n\t"
      "ds_read_b64_tr_b16 a[74:75], v19 offset:2048\n\t"
      "v_mfma_f32_16x16x32_bf16 v[252:255], a[120:123], a[56:59], 0\n\t"
      "v_mfma_f32_16x16x32_bf16 v[252:255], a[124:127], a[60:63], v[252:255]\n\t"
      "s_waitcnt vmcnt(25)\n\t"
      "v_mfma_f32_16x16x32_bf16 v[32:35], a[128:131], a[0:3], v[32:35]\n\t"
      "global_load_dwordx4 a[56:59], v31, s[36:37] offset:0 \n\t"
      "v_mfma_f32_16x16x32_bf16 v[32:35], a[132:135], a[4:7], v[32:35]\n\t"
      "ds_read_b64_tr_b16 a[80:81], v20 offset:0\n\t"
      "v_mfma_f32_16x16x32_bf16 v[36:39], a[136:139], a[0:3], v[36:39]\n\t"
      "global_load_dwordx4 a[60:63], v31, s[36:37] offset:1024 \n\t"
      "v_mfma_f32_16x16x32_bf16 v[36:39], a[140:143], a[4:7], v[36:39]\n\t"
      "ds_read_b64_tr_b16 a[82:83], v20 offset:2048\n\t"
      "s_add_u32 s36, s36, 16384\n\t"
      "s_addc_u32 s37, s37, 0\n\t"
      "v_mfma_f32_16x16x32_bf16 v[40:43], a[144:147], a[0:3], v[40:43]\n\t"
      "v_mfma_f32_16x16x32_bf16 v[40:43], a[148:151], a[4:7], v[40:43]\n\t"
      "v_mfma_f32_16x16x32_bf16 v[44:47], a[152:155], a[0:3], v[44:47]\n\t"
      "v_mfma_f32_16x16x32_bf16 v[44:47], a[156:159], a[4:7], v[44:47]\n\t"
      "ds_read_b64_tr_b16 a[88:89], v21 offset:0\n\t"
      "v_mfma_f32_16x16x32_bf16 v[48:51], a[160:163], a[0:3], v[48:51]\n\t"
      "v_mfma_f32_16x16x32_bf16 v[48:51], a[164:167], a[4:7], v[48:51]\n\t"
      "ds_read_b64_tr_b16 a[90:91], v21 offset:2048\n\t"
      "v_mfma_f32_16x16x32_bf16 v[52:55], a[168:171], a[0:3], v[52:55]\n\t"
      "global_load_lds_dwordx4 v8, s[14:15] offset:2048 \n\t"
      "v_mfma_f32_16x16x32_bf16 v[52:55], a[172:175], a[4:7], v[52:55]\n\t"
      "v_mfma_f32_16x16x32_bf16 v[56:59], a[176:179], a[0:3], v[56:59]\n\t"
      "v_mfma_f32_16x16x32_bf16 v[56:59], a[180:183], a[4:7], v[56:59]\n\t"
      "ds_read_b64_tr_b16 a[96:97], v22 offset:0\n\t"
      "v_mfma_f32_16x16x32_bf16 v[60:63], a[184:187], a[0:3], v[60:63]\n\t"
      "v_mfma_f32_16x16x32_bf16 v[60:63], a[188:191], a[4:7], v[60:63]\n\t"
      "ds_read_b64_tr_b16 a[98:99], v22 offset:2048\n\t"
      "s_waitcnt vmcnt(24)\n\t"
      "v_mfma_f32_16x16x32_bf16 v[64:67], a[128:131], a[8:11], v[64:67]\n\t"
      "global_load_dwordx4 a[0:3], v1, s[36:37] offset:0 \n\t"
      "v_mfma_f32_16x16x32_bf16 v[64:67], a[132:135], a[12:15], v[64:67]\n\t"
      "v_mfma_f32_16x16x32_bf16 v[68:71], a[136:139], a[8:11], v[68:71]\n\t"
      "global_load_dwordx4 a[4:7], v1, s[36:37] offset:1024 \n\t"
      "v_mfma_f32_16x16x32_bf16 v[68:71], a[140:143], a[12:15], v[68:71]\n\t"
      "ds_read_b64_tr_b16 a[104:105], v23 offset:0\n\t"
      "v_mfma_f32_16x16x32_bf16 v[72:75], a[144:147], a[8:11], v[72:75]\n\t"
      "v_mfma_f32_16x16x32_bf16 v[72:75], a[148:151], a[12:15], v[72:75]\n\t"
      "ds_read_b64_tr_b16 a[106:107], v23 offset:2048\n\t"
      "v_mfma_f32_16x16x32_bf16 v[76:79], a[152:155], a[8:11], v[76:79]\n\t"
      "v_mfma_f32_16x16x32_bf16 v[76:79], a[156:159], a[12:15], v[76:79]\n\t"
      "v_mfma_f32_16x16x32_bf16 v[80:83], a[160:163], a[8:11], v[80:83]\n\t"
      "v_mfma_f32_16x16x32_bf16 v[80:83], a[164:167], a[12:15], v[80:83]\n\t"
      "ds_read_b64_tr_b16 a[112:113], v24 offset:0\n\t"
      "v_mfma_f32_16x16x32_bf16 v[84:87], a[168:171], a[8:11], v[84:87]\n\t"
      "global_load_lds_dwordx4 v9, s[14:15] offset:3072 \n\t"
      "v_mfma_f32_16x16x32_bf16 v[84:87], a[172:175], a[12:15], v[84:87]\n\t"
      "ds_read_b64_tr_b16 a[114:115], v24 offset:2048\n\t"
      "v_mfma_f32_16x16x32_bf16 v[88:91], a[176:179], a[8:11], v[88:91]\n\t"
      "s_add_u32 s49, s49, 0x200000\n\t"
      "v_mfma_f32_16x16x32_bf16 v[88:91], a[180:183], a[12:15], v[88:91]\n\t"
      "v_mfma_f32_16x16x32_bf16 v[92:95], a[184:187], a[8:11], v[92:95]\n\t"
      "s_and_b32 s49, s49, 0xfffffff\n\t"
      "v_mfma_f32_16x16x32_bf16 v[92:95], a[188:191], a[12:15], v[92:95]\n\t"
      "ds_read_b64_tr_b16 a[120:121], v25 offset:0\n\t"
      "s_waitcnt vmcnt(25)\n\t"
      "v_mfma_f32_16x16x32_bf16 v[96:99], a[128:131], a[16:19], v[96:99]\n\t"
      "global_load_dwordx4 a[8:11], v16, s[36:37] offset:0 \n\t"
      "v_mfma_f32_16x16x32_bf16 v[96:99], a[132:135], a[20:23], v[96:99]\n\t"
      "ds_read_b64_tr_b16 a[122:123], v25 offset:2048\n\t"
      "v_mfma_f32_16x16x32_bf16 v[100:103], a[136:139], a[16:19], v[100:103]\n\t"
      "global_load_dwordx4 a[12:15], v16, s[36:37] offset:1024 \n\t"
      "v_mfma_f32_16x16x32_bf16 v[100:103], a[140:143], a[20:23], v[100:103]\n\t"
      "v_mfma_f32_16x16x32_bf16 v[104:107], a[144:147], a[16:19], v[104:107]\n\t"
      "v_mfma_f32_16x16x32_bf16 v[104:107], a[148:151], a[20:23], v[104:107]\n\t"
      "v_mfma_f32_16x16x32_bf16 v[108:111], a[152:155], a[16:19], v[108:111]\n\t"
      "v_mfma_f32_16x16x32_bf16 v[108:111], a[156:159], a[20:23], v[108:111]\n\t"
      "v_mfma_f32_16x16x32_bf16 v[112:115], a[160:163], a[16:19], v[112:115]\n\t"
      "v_mfma_f32_16x16x32_bf16 v[112:115], a[164:167], a[20:23], v[112:115]\n\t"
      "v_mfma_f32_16x16x32_bf16 v[116:119], a[168:171], a[16:19], v[116:119]\n\t"
      "v_mfma_f32_16x16x32_bf16 v[116:119], a[172:175], a[20:23], v[116:119]\n\t"
      "v_mfma_f32_16x16x32_bf16 v[120:123], a[176:179], a[16:19], v[120:123]\n\t"
      "v_mfma_f32_16x16x32_bf16 v[120:123], a[180:183], a[20:23], v[120:123]\n\t"
      "v_mfma_f32_16x16x32_bf16 v[124:127], a[184:187], a[16:19], v[124:127]\n\t"
      "v_mfma_f32_16x16x32_bf16 v[124:127], a[188:191], a[20:23], v[124:127]\n\t"
      "s_waitcnt vmcnt(22)\n\t"
      "v_mfma_f32_16x16x32_bf16 a[192:195], a[128:131], a[24:27], a[192:195]\n\t"
      "global_load_dwordx4 a[16:19], v14, s[36:37] offset:0 \n\t"
      "v_mfma_f32_16x16x32_bf16 a[192:195], a[132:135], a[28:31], a[192:195]\n\t"
      "v_mfma_f32_16x16x32_bf16 a[196:199], a[136:139], a[24:27], a[196:199]\n\t"
      "global_load_dwordx4 a[20:23], v14, s[36:37] offset:1024 \n\t"
      "v_mfma_f32_16x16x32_bf16 a[196:199], a[140:143], a[28:31], a[196:199]\n\t"
      "s_waitcnt vmcnt(23) lgkmcnt(0)\n\t"
      "s_barrier\n\t"
      "v_mfma_f32_16x16x32_bf16 a[200:203], a[144:147], a[24:27], a[200:203]\n\t"
      "v_mfma_f32_16x16x32_bf16 a[200:203], a[148:151], a[28:31], a[200:203]\n\t"
      "ds_read_b64_tr_b16 a[68:69], v18 offset:16384\n\t"
      "v_mfma_f32_16x16x32_bf16 a[204:207], a[152:155], a[24:27], a[204:207]\n\t"
      "s_add_u32 m0, s99, 0\n\t"
      "s_add_u32 s14, s12, s49\n\t"
      "s_addc_u32 s15, s13, 0\n\t"
      "v_mfma_f32_16x16x32_bf16 a[204:207], a[156:159], a[28:31], a[204:207]\n\t"
      "ds_read_b64_tr_b16 a[70:71], v18 offset:18432\n\t"
      "v_mfma_f32_16x16x32_bf16 a[208:211], a[160:163], a[24:27], a[208:211]\n\t"
      "global_load_lds_dwordx4 v6, s[14:15] offset:0 \n\t"
      "v_mfma_f32_16x16x32_bf16 a[208:211], a[164:167], a[28:31], a[208:211]\n\t"
      "v_mfma_f32_16x16x32_bf16 a[212:215], a[168:171], a[24:27], a[212:215]\n\t"
      "v_mfma_f32_16x16x32_bf16 a[212:215], a[172:175], a[28:31], a[212:215]\n\t"
      "ds_read_b64_tr_b16 a[76:77], v19 offset:16384\n\t"
      "v_mfma_f32_16x16x32_bf16 a[216:219], a[176:179], a[24:27], a[216:219]\n\t"
      "global_load_lds_dwordx4 v7, s[14:15] offset:1024 \n\t"
      "v_mfma_f32_16x16x32_bf16 a[216:219], a[180:183], a[28:31], a[216:219]\n\t"
      "ds_read_b64_tr_b16 a[78:79], v19 offset:18432\n\t"
      "v_mfma_f32_16x16x32_bf16 a[220:223], a[184:187], a[24:27], a[220:223]\n\t"
      "v_mfma_f32_16x16x32_bf16 a[220:223], a[188:191], a[28:31], a[220:223]\n\t"
      "s_waitcnt vmcnt(22)\n\t"
      "v_mfma_f32_16x16x32_bf16 v[128:131], a[128:131], a[32:35], v[128:131]\n\t"
      "global_load_dwordx4 a[24:27], v31, s[36:37] offset:0 \n\t"
      "v_mfma_f32_16x16x32_bf16 v[128:131], a[132:135], a[36:39], v[128:131]\n\t"
      "ds_read_b64_tr_b16 a[84:85], v20 offset:16384\n\t"
      "v_mfma_f32_16x16x32_bf16 v[132:135], a[136:139], a[32:35], v[132:135]\n\t"
      "global_load_dwordx4 a[28:31], v31, s[36:37] offset:1024 \n\t"
      "v_mfma_f32_16x16x32_bf16 v[132:135], a[140:143], a[36:39], v[132:135]\n\t"
      "ds_read_b64_tr_b16 a[86:87], v20 offset:18432\n\t"
      "s_add_u32 s36, s36, 16384\n\t"
      "s_addc_u32 s37, s37, 0\n\t"
      "v_mfma_f32_16x16x32_bf16 v[136:139], a[144:147], a[32:35], v[136:139]\n\t"
      "v_mfma_f32_16x16x32_bf16 v[136:139], a[148:151], a[36:39], v[136:139]\n\t"
      "v_mfma_f32_16x16x32_bf16 v[140:143], a[152:155], a[32:35], v[140:143]\n\t"
      "v_mfma_f32_16x16x32_bf16 v[140:143], a[156:159], a[36:39], v[140:143]\n\t"
      "ds_read_b64_tr_b16 a[92:93], v21 offset:16384\n\t"
      "v_mfma_f32_16x16x32_bf16 v[144:147], a[160:163], a[32:35], v[144:147]\n\t"
      "v_mfma_f32_16x16x32_bf16 v[144:147], a[164:167], a[36:39], v[144:147]\n\t"
      "ds_read_b64_tr_b16 a[94:95], v21 offset:18432\n\t"
      "v_mfma_f32_16x16x32_bf16 v[148:151], a[168:171], a[32:35], v[148:151]\n\t"
      "global_load_lds_dwordx4 v8, s[14:15] offset:2048 \n\t"
      "v_mfma_f32_16x16x32_bf16 v[148:151], a[172:175], a[36:39], v[148:151]\n\t"
      "v_mfma_f32_16x16x32_bf16 v[152:155], a[176:179], a[32:35], v[152:155]\n\t"
      "v_mfma_f32_16x16x32_bf16 v[152:155], a[180:183], a[36:39], v[152:155]\n\t"
      "ds_read_b64_tr_b16 a[100:101], v22 offset:16384\n\t"
      "v_mfma_f32_16x16x32_bf16 v[156:159], a[184:187], a[32:35], v[156:159]\n\t"
      "v_mfma_f32_16x16x32_bf16 v[156:159], a[188:191], a[36:39], v[156:159]\n\t"
      "ds_read_b64_tr_b16 a[102:103], v22 offset:18432\n\t"
      "s_waitcnt vmcnt(20)\n\t"
      "v_mfma_f32_16x16x32_bf16 v[160:163], a[128:131], a[40:43], v[160:163]\n\t"
      "global_load_dwordx4 a[32:35], v1, s[36:37] offset:0 \n\t"
      "v_mfma_f32_16x16x32_bf16 v[160:163], a[132:135], a[44:47], v[160:163]\n\t"
      "v_mfma_f32_16x16x32_bf16 v[164:167], a[136:139], a[40:43], v[164:167]\n\t"
      "global_load_dwordx4 a[36:39], v1, s[36:37] offset:1024 \n\t"
      "v_mfma_f32_16x16x32_bf16 v[164:167], a[140:143], a[44:47], v[164:167]\n\t"
      "ds_read_b64_tr_b16 a[108:109], v23 offset:16384\n\t"
      "v_mfma_f32_16x16x32_bf16 v[168:171], a[144:147], a[40:43], v[168:171]\n\t"
      "v_mfma_f32_16x16x32_bf16 v[168:171], a[148:151], a[44:47], v[168:171]\n\t"
      "ds_read_b64_tr_b16 a[110:111], v23 offset:18432\n\t"
      "v_mfma_f32_16x16x32_bf16 v[172:175], a[152:155], a[40:43], v[172:175]\n\t"
      "v_mfma_f32_16x16x32_bf16 v[172:175], a[156:159], a[44:47], v[172:175]\n\t"
      "v_mfma_f32_16x16x32_bf16 v[176:179], a[160:163], a[40:43], v[176:179]\n\t"
      "v_mfma_f32_16x16x32_bf16 v[176:179], a[164:167], a[44:47], v[176:179]\n\t"
      "ds_read_b64_tr_b16 a[116:117], v24 offset:16384\n\t"
      "v_mfma_f32_16x16x32_bf16 v[180:183], a[168:171], a[40:43], v[180:183]\n\t"
      "global_load_lds_dwordx4 v9, s[14:15] offset:3072 \n\t"
      "v_mfma_f32_16x16x32_bf16 v[180:183], a[172:175], a[44:47], v[180:183]\n\t"
      "ds_read_b64_tr_b16 a[118:119], v24 offset:18432\n\t"
      "v_mfma_f32_16x16x32_bf16 v[184:187], a[176:179], a[40:43], v[184:187]\n\t"
      "s_add_u32 s49, s49, 0x200000\n\t"
      "v_mfma_f32_16x16x32_bf16 v[184:187], a[180:183], a[44:47], v[184:187]\n\t"
      "v_mfma_f32_16x16x32_bf16 v[188:191], a[184:187], a[40:43], v[188:191]\n\t"
      "s_and_b32 s49, s49, 0xfffffff\n\t"
      "v_mfma_f32_16x16x32_bf16 v[188:191], a[188:191], a[44:47], v[188:191]\n\t"
      "ds_read_b64_tr_b16 a[124:125], v25 offset:16384\n\t"
      "s_waitcnt vmcnt(21)\n\t"
      "v_mfma_f32_16x16x32_bf16 v[192:195], a[128:131], a[48:51], v[192:195]\n\t"
      "global_load_dwordx4 a[40:43], v16, s[36:37] offset:0 \n\t"
      "v_mfma_f32_16x16x32_bf16 v[192:195], a[132:135], a[52:55], v[192:195]\n\t"
      "ds_read_b64_tr_b16 a[126:127], v25 offset:18432\n\t"
      "v_mfma_f32_16x16x32_bf16 v[196:199], a[136:139], a[48:51], v[196:199]\n\t"
      "global_load_dwordx4 a[44:47], v16, s[36:37] offset:1024 \n\t"
      "v_mfma_f32_16x16x32_bf16 v[196:199], a[140:143], a[52:55], v[196:199]\n\t"
      "v_mfma_f32_16x16x32_bf16 v[200:203], a[144:147], a[48:51], v[200:203]\n\t"
      "v_mfma_f32_16x16x32_bf16 v[200:203], a[148:151], a[52:55], v[200:203]\n\t"
      "v_mfma_f32_16x16x32_bf16 v[204:207], a[152:155], a[48:51], v[204:207]\n\t"
      "v_mfma_f32_16x16x32_bf16 v[204:207], a[156:159], a[52:55], v[204:207]\n\t"
      "v_mfma_f32_16x16x32_bf16 v[208:211], a[160:163], a[48:51], v[208:211]\n\t"
      "v_mfma_f32_16x16x32_bf16 v[208:211], a[164:167], a[52:55], v[208:211]\n\t"
      "v_mfma_f32_16x16x32_bf16 v[212:215], a[168:171], a[48:51], v[212:215]\n\t"
      "v_mfma_f32_16x16x32_bf16 v[212:215], a[172:175], a[52:55], v[212:215]\n\t"
      "v_mfma_f32_16x16x32_bf16 v[216:219], a[176:179], a[48:51], v[216:219]\n\t"
      "v_mfma_f32_16x16x32_bf16 v[216:219], a[180:183], a[52:55], v[216:219]\n\t"
      "v_mfma_f32_16x16x32_bf16 v[220:223], a[184:187], a[48:51], v[220:223]\n\t"
      "v_mfma_f32_16x16x32_bf16 v[220:223], a[188:191], a[52:55], v[220:223]\n\t"
      "s_waitcnt vmcnt(18)\n\t"
      "v_mfma_f32_16x16x32_bf16 v[224:227], a[128:131], a[56:59], v[224:227]\n\t"
      "global_load_dwordx4 a[48:51], v14, s[36:37] offset:0 \n\t"
      "v_mfma_f32_16x16x32_bf16 v[224:227], a[132:135], a[60:63], v[224:227]\n\t"
      "v_mfma_f32_16x16x32_bf16 v[228:231], a[136:139], a[56:59], v[228:231]\n\t"
      "global_load_dwordx4 a[52:55], v14, s[36:37] offset:1024 \n\t"
      "v_mfma_f32_16x16x32_bf16 v[228:231], a[140:143], a[60:63], v[228:231]\n\t"
      "s_waitcnt vmcnt(29) lgkmcnt(0)\n\t"
      "s_barrier\n\t"
      "v_mfma_f32_16x16x32_bf16 v[232:235], a[144:147], a[56:59], v[232:235]\n\t"
      "v_mfma_f32_16x16x32_bf16 v[232:235], a[148:151], a[60:63], v[232:235]\n\t"
      "ds_read_b64_tr_b16 a[128:129], v18 offset:32768\n\t"
      "v_mfma_f32_16x16x32_bf16 v[236:239], a[152:155], a[56:59], v[236:239]\n\t"
      "s_add_u32 m0, s99, 16384\n\t"
      "s_add_u32 s14, s12, s49\n\t"
      "s_addc_u32 s15, s13, 0\n\t"
      "v_mfma_f32_16x16x32_bf16 v[236:239], a[156:159], a[60:63], v[236:239]\n\t"
      "ds_read_b64_tr_b16 a[130:131], v18 offset:34816\n\t"
      "v_mfma_f32_16x16x32_bf16 v[240:243], a[160:163], a[56:59], v[240:243]\n\t"
      "global_load_lds_dwordx4 v6, s[14:15] offset:0 \n\t"
      "v_mfma_f32_16x16x32_bf16 v[240:243], a[164:167], a[60:63], v[240:243]\n\t"
      "v_mfma_f32_16x16x32_bf16 v[244:247], a[168:171], a[56:59], v[244:247]\n\t"
      "v_mfma_f32_16x16x32_bf16 v[244:247], a[172:175], a[60:63], v[244:247]\n\t"
      "ds_read_b64_tr_b16 a[136:137], v19 offset:32768\n\t"
      "v_mfma_f32_16x16x32_bf16 v[248:251], a[176:179], a[56:59], v[248:251]\n\t"
      "global_load_lds_dwordx4 v7, s[14:15] offset:1024 \n\t"
      "v_mfma_f32_16x16x32_bf16 v[248:251], a[180:183], a[60:63], v[248:251]\n\t"
      "ds_read_b64_tr_b16 a[138:139], v19 offset:34816\n\t"
      "v_mfma_f32_16x16x32_bf16 v[252:255], a[184:187], a[56:59], v[252:255]\n\t"
      "v_mfma_f32_16x16x32_bf16 v[252:255], a[188:191], a[60:63], v[252:255]\n\t"
      "s_waitcnt vmcnt(19)\n\t"
      "v_mfma_f32_16x16x32_bf16 v[32:35], a[64:67], a[0:3], v[32:35]\n\t"
      "global_load_dwordx4 a[56:59], v31, s[36:37] offset:0 \n\t"
      "v_mfma_f32_16x16x32_bf16 v[32:35], a[68:71], a[4:7], v[32:35]\n\t"
      "ds_read_b64_tr_b16 a[144:145], v20 offset:32768\n\t"
      "v_mfma_f32_16x16x32_bf16 v[36:39], a[72:75], a[0:3], v[36:39]\n\t"
      "global_load_dwordx4 a[60:63], v31, s[36:37] offset:1024 \n\t"
      "v_mfma_f32_16x16x32_bf16 v[36:39], a[76:79], a[4:7], v[36:39]\n\t"
      "ds_read_b64_tr_b16 a[146:147], v20 offset:34816\n\t"
      "s_add_u32 s36, s36, 16384\n\t"
      "s_addc_u32 s37, s37, 0\n\t"
      "v_mfma_f32_16x16x32_bf16 v[40:43], a[80:83], a[0:3], v[40:43]\n\t"
      "v_mfma_f32_16x16x32_bf16 v[40:43], a[84:87], a[4:7], v[40:43]\n\t"
      "v_mfma_f32_16x16x32_bf16 v[44:47], a[88:91], a[0:3], v[44:47]\n\t"
      "v_mfma_f32_16x16x32_bf16 v[44:47], a[92:95], a[4:7], v[44:47]\n\t"
      "ds_read_b64_tr_b16 a[152:153], v21 offset:32768\n\t"
      "v_mfma_f32_16x16x32_bf16 v[48:51], a[96:99], a[0:3], v[48:51]\n\t"
      "v_mfma_f32_16x16x32_bf16 v[48:51], a[100:103], a[4:7], v[48:51]\n\t"
      "ds_read_b64_tr_b16 a[154:155], v21 offset:34816\n\t"
      "v_mfma_f32_16x16x32_bf16 v[52:55], a[104:107], a[0:3], v[52:55]\n\t"
      "global_load_lds_dwordx4 v8, s[14:15] offset:2048 \n\t"
      "v_mfma_f32_16x16x32_bf16 v[52:55], a[108:111], a[4:7], v[52:55]\n\t"
      "v_mfma_f32_16x16x32_bf16 v[56:59], a[112:115], a[0:3], v[56:59]\n\t"
      "v_mfma_f32_16x16x32_bf16 v[56:59], a[116:119], a[4:7], v[56:59]\n\t"
      "ds_read_b64_tr_b16 a[160:161], v22 offset:32768\n\t"
      "v_mfma_f32_16x16x32_bf16 v[60:63], a[120:123], a[0:3], v[60:63]\n\t"
      "v_mfma_f32_16x16x32_bf16 v[60:63], a[124:127], a[4:7], v[60:63]\n\t"
      "ds_read_b64_tr_b16 a[162:163], v22 offset:34816\n\t"
      "s_waitcnt vmcnt(19)\n\t"
      "v_mfma_f32_16x16x32_bf16 v[64:67], a[64:67], a[8:11], v[64:67]\n\t"
      "global_load_dwordx4 a[0:3], v1, s[36:37] offset:0 \n\t"
      "v_mfma_f32_16x16x32_bf16 v[64:67], a[68:71], a[12:15], v[64:67]\n\t"
      "v_mfma_f32_16x16x32_bf16 v[68:71], a[72:75], a[8:11], v[68:71]\n\t"
      "global_load_dwordx4 a[4:7], v1, s[36:37] offset:1024 \n\t"
      "v_mfma_f32_16x16x32_bf16 v[68:71], a[76:79], a[12:15], v[68:71]\n\t"
      "ds_read_b64_tr_b16 a[168:169], v23 offset:32768\n\t"
      "v_mfma_f32_16x16x32_bf16 v[72:75], a[80:83], a[8:11], v[72:75]\n\t"
      "v_mfma_f32_16x16x32_bf16 v[72:75], a[84:87], a[12:15], v[72:75]\n\t"
      "ds_read_b64_tr_b16 a[170:171], v23 offset:34816\n\t"
      "v_mfma_f32_16x16x32_bf16 v[76:79], a[88:91], a[8:11], v[76:79]\n\t"
      "v_mfma_f32_16x16x32_bf16 v[76:79], a[92:95], a[12:15], v[76:79]\n\t"
      "v_mfma_f32_16x16x32_bf16 v[80:83], a[96:99], a[8:11], v[80:83]\n\t"
      "v_mfma_f32_16x16x32_bf16 v[80:83], a[100:103], a[12:15], v[80:83]\n\t"
      "ds_read_b64_tr_b16 a[176:177], v24 offset:32768\n\t"
      "v_mfma_f32_16x16x32_bf16 v[84:87], a[104:107], a[8:11], v[84:87]\n\t"
      "global_load_lds_dwordx4 v9, s[14:15] offset:3072 \n\t"
      "v_mfma_f32_16x16x32_bf16 v[84:87], a[108:111], a[12:15], v[84:87]\n\t"
      "ds_read_b64_tr_b16 a[178:179], v24 offset:34816\n\t"
      "v_mfma_f32_16x16x32_bf16 v[88:91], a[112:115], a[8:11], v[88:91]\n\t"
      "s_add_u32 s49, s49, 0x200000\n\t"
      "v_mfma_f32_16x16x32_bf16 v[88:91], a[116:119], a[12:15], v[88:91]\n\t"
      "v_mfma_f32_16x16x32_bf16 v[92:95], a[120:123], a[8:11], v[92:95]\n\t"
      "s_and_b32 s49, s49, 0xfffffff\n\t"
      "v_mfma_f32_16x16x32_bf16 v[92:95], a[124:127], a[12:15], v[92:95]\n\t"
      "ds_read_b64_tr_b16 a[184:185], v25 offset:32768\n\t"
      "s_waitcnt vmcnt(20)\n\t"
      "v_mfma_f32_16x16x32_bf16 v[96:99], a[64:67], a[16:19], v[96:99]\n\t"
      "global_load_dwordx4 a[8:11], v16, s[36:37] offset:0 \n\t"
      "v_mfma_f32_16x16x32_bf16 v[96:99], a[68:71], a[20:23], v[96:99]\n\t"
      "ds_read_b64_tr_b16 a[186:187], v25 offset:34816\n\t"
      "v_mfma_f32_16x16x32_bf16 v[100:103], a[72:75], a[16:19], v[100:103]\n\t"
      "global_load_dwordx4 a[12:15], v16, s[36:37] offset:1024 \n\t"
      "v_mfma_f32_16x16x32_bf16 v[100:103], a[76:79], a[20:23], v[100:103]\n\t"
      "v_mfma_f32_16x16x32_bf16 v[104:107], a[80:83], a[16:19], v[104:107]\n\t"
      "v_mfma_f32_16x16x32_bf16 v[104:107], a[84:87], a[20:23], v[104:107]\n\t"
      "v_mfma_f32_16x16x32_bf16 v[108:111], a[88:91], a[16:19], v[108:111]\n\t"
      "v_mfma_f32_16x16x32_bf16 v[108:111], a[92:95], a[20:23], v[108:111]\n\t"
      "v_mfma_f32_16x16x32_bf16 v[112:115], a[96:99], a[16:19], v[112:115]\n\t"
      "v_mfma_f32_16x16x32_bf16 v[112:115], a[100:103], a[20:23], v[112:115]\n\t"
      "v_mfma_f32_16x16x32_bf16 v[116:119], a[104:107], a[16:19], v[116:119]\n\t"
      "v_mfma_f32_16x16x32_bf16 v[116:119], a[108:111], a[20:23], v[116:119]\n\t"
      "v_mfma_f32_16x16x32_bf16 v[120:123], a[112:115], a[16:19], v[120:123]\n\t"
      "v_mfma_f32_16x16x32_bf16 v[120:123], a[116:119], a[20:23], v[120:123]\n\t"
      "v_mfma_f32_16x16x32_bf16 v[124:127], a[120:123], a[16:19], v[124:127]\n\t"
      "v_mfma_f32_16x16x32_bf16 v[124:127], a[124:127], a[20:23], v[124:127]\n\t"
      "s_waitcnt vmcnt(18)\n\t"
      "v_mfma_f32_16x16x32_bf16 a[192:195], a[64:67], a[24:27], a[192:195]\n\t"
      "global_load_dwordx4 a[16:19], v14, s[36:37] offset:0 \n\t"
      "v_mfma_f32_16x16x32_bf16 a[192:195], a[68:71], a[28:31], a[192:195]\n\t"
      "v_mfma_f32_16x16x32_bf16 a[196:199], a[72:75], a[24:27], a[196:199]\n\t"
      "global_load_dwordx4 a[20:23], v14, s[36:37] offset:1024 \n\t"
      "v_mfma_f32_16x16x32_bf16 a[196:199], a[76:79], a[28:31], a[196:199]\n\t"
      "s_waitcnt vmcnt(28) lgkmcnt(0)\n\t"
      "s_barrier\n\t"
      "v_mfma_f32_16x16x32_bf16 a[200:203], a[80:83], a[24:27], a[200:203]\n\t"
      "v_mfma_f32_16x16x32_bf16 a[200:203], a[84:87], a[28:31], a[200:203]\n\t"
      "ds_read_b64_tr_b16 a[132:133], v18 offset:49152\n\t"
      "v_mfma_f32_16x16x32_bf16 a[204:207], a[88:91], a[24:27], a[204:207]\n\t"
      "s_add_u32 m0, s99, 32768\n\t"
      "s_add_u32 s14, s12, s49\n\t"
      "s_addc_u32 s15, s13, 0\n\t"
      "v_mfma_f32_16x16x32_bf16 a[204:207], a[92:95], a[28:31], a[204:207]\n\t"
      "ds_read_b64_tr_b16 a[134:135], v18 offset:51200\n\t"
      "v_mfma_f32_16x16x32_bf16 a[208:211], a[96:99], a[24:27], a[208:211]\n\t"
      "global_load_lds_dwordx4 v6, s[14:15] offset:0 \n\t"
      "v_mfma_f32_16x16x32_bf16 a[208:211], a[100:103], a[28:31], a[208:211]\n\t"
      "v_mfma_f32_16x16x32_bf16 a[212:215], a[104:107], a[24:27], a[212:215]\n\t"
      "v_mfma_f32_16x16x32_bf16 a[212:215], a[108:111], a[28:31], a[212:215]\n\t"
      "ds_read_b64_tr_b16 a[140:141], v19 offset:49152\n\t"
      "v_mfma_f32_16x16x32_bf16 a[216:219], a[112:115], a[24:27], a[216:219]\n\t"
      "global_load_lds_dwordx4 v7, s[14:15] offset:1024 \n\t"
      "v_mfma_f32_16x16x32_bf16 a[216:219], a[116:119], a[28:31], a[216:219]\n\t"
      "ds_read_b64_tr_b16 a[142:143], v19 offset:51200\n\t"
      "v_mfma_f32_16x16x32_bf16 a[220:223], a[120:123], a[24:27], a[220:223]\n\t"
      "v_mfma_f32_16x16x32_bf16 a[220:223], a[124:127], a[28:31], a[220:223]\n\t"
      "s_waitcnt vmcnt(19)\n\t"
      "v_mfma_f32_16x16x32_bf16 v[128:131], a[64:67], a[32:35], v[128:131]\n\t"
      "global_load_dwordx4 a[24:27], v31, s[36:37] offset:0 \n\t"
      "v_mfma_f32_16x16x32_bf16 v[128:131], a[68:71], a[36:39], v[128:131]\n\t"
      "ds_read_b64_tr_b16 a[148:149], v20 offset:49152\n\t"
      "v_mfma_f32_16x16x32_bf16 v[132:135], a[72:75], a[32:35], v[132:135]\n\t"
      "global_load_dwordx4 a[28:31], v31, s[36:37] offset:1024 \n\t"
      "v_mfma_f32_16x16x32_bf16 v[132:135], a[76:79], a[36:39], v[132:135]\n\t"
      "ds_read_b64_tr_b16 a[150:151], v20 offset:51200\n\t"
      "s_add_u32 s36, s36, 16384\n\t"
      "s_addc_u32 s37, s37, 0\n\t"
      "v_mfma_f32_16x16x32_bf16 v[136:139], a[80:83], a[32:35], v[136:139]\n\t"
      "v_mfma_f32_16x16x32_bf16 v[136:139], a[84:87], a[36:39], v[136:139]\n\t"
      "v_mfma_f32_16x16x32_bf16 v[140:143], a[88:91], a[32:35], v[140:143]\n\t"
      "v_mfma_f32_16x16x32_bf16 v[140:143], a[92:95], a[36:39], v[140:143]\n\t"
      "ds_read_b64_tr_b16 a[156:157], v21 offset:49152\n\t"
      "v_mfma_f32_16x16x32_bf16 v[144:147], a[96:99], a[32:35], v[144:147]\n\t"
      "v_mfma_f32_16x16x32_bf16 v[144:147], a[100:103], a[36:39], v[144:147]\n\t"
      "ds_read_b64_tr_b16 a[158:159], v21 offset:51200\n\t"
      "v_mfma_f32_16x16x32_bf16 v[148:151], a[104:107], a[32:35], v[148:151]\n\t"
      "global_load_lds_dwordx4 v8, s[14:15] offset:2048 \n\t"
      "v_mfma_f32_16x16x32_bf16 v[148:151], a[108:111], a[36:39], v[148:151]\n\t"
      "v_mfma_f32_16x16x32_bf16 v[152:155], a[112:115], a[32:35], v[152:155]\n\t"
      "v_mfma_f32_16x16x32_bf16 v[152:155], a[116:119], a[36:39], v[152:155]\n\t"
      "ds_read_b64_tr_b16 a[164:165], v22 offset:49152\n\t"
      "v_mfma_f32_16x16x32_bf16 v[156:159], a[120:123], a[32:35], v[156:159]\n\t"
      "v_mfma_f32_16x16x32_bf16 v[156:159], a[124:127], a[36:39], v[156:159]\n\t"
      "ds_read_b64_tr_b16 a[166:167], v22 offset:51200\n\t"
      "s_waitcnt vmcnt(19)\n\t"
      "v_mfma_f32_16x16x32_bf16 v[160:163], a[64:67], a[40:43], v[160:163]\n\t"
      "global_load_dwordx4 a[32:35], v1, s[36:37] offset:0 \n\t"
      "v_mfma_f32_16x16x32_bf16 v[160:163], a[68:71], a[44:47], v[160:163]\n\t"
      "v_mfma_f32_16x16x32_bf16 v[164:167], a[72:75], a[40:43], v[164:167]\n\t"
      "global_load_dwordx4 a[36:39], v1, s[36:37] offset:1024 \n\t"
      "v_mfma_f32_16x16x32_bf16 v[164:167], a[76:79], a[44:47], v[164:167]\n\t"
      "ds_read_b64_tr_b16 a[172:173], v23 offset:49152\n\t"
      "v_mfma_f32_16x16x32_bf16 v[168:171], a[80:83], a[40:43], v[168:171]\n\t"
      "v_mfma_f32_16x16x32_bf16 v[168:171], a[84:87], a[44:47], v[168:171]\n\t"
      "ds_read_b64_tr_b16 a[174:175], v23 offset:51200\n\t"
      "v_mfma_f32_16x16x32_bf16 v[172:175], a[88:91], a[40:43], v[172:175]\n\t"
      "v_mfma_f32_16x16x32_bf16 v[172:175], a[92:95], a[44:47], v[172:175]\n\t"
      "v_mfma_f32_16x16x32_bf16 v[176:179], a[96:99], a[40:43], v[176:179]\n\t"
      "v_mfma_f32_16x16x32_bf16 v[176:179], a[100:103], a[44:47], v[176:179]\n\t"
      "ds_read_b64_tr_b16 a[180:181], v24 offset:49152\n\t"
      "v_mfma_f32_16x16x32_bf16 v[180:183], a[104:107], a[40:43], v[180:183]\n\t"
      "global_load_lds_dwordx4 v9, s[14:15] offset:3072 \n\t"
      "v_mfma_f32_16x16x32_bf16 v[180:183], a[108:111], a[44:47], v[180:183]\n\t"
      "ds_read_b64_tr_b16 a[182:183], v24 offset:51200\n\t"
      "v_mfma_f32_16x16x32_bf16 v[184:187], a[112:115], a[40:43], v[184:187]\n\t"
      "s_add_u32 s49, s49, 0x200000\n\t"
      "v_mfma_f32_16x16x32_bf16 v[184:187], a[116:119], a[44:47], v[184:187]\n\t"
      "v_mfma_f32_16x16x32_bf16 v[188:191], a[120:123], a[40:43], v[188:191]\n\t"
      "s_and_b32 s49, s49, 0xfffffff\n\t"
      "v_mfma_f32_16x16x32_bf16 v[188:191], a[124:127], a[44:47], v[188:191]\n\t"
      "ds_read_b64_tr_b16 a[188:189], v25 offset:49152\n\t"
      "s_waitcnt vmcnt(20)\n\t"
      "v_mfma_f32_16x16x32_bf16 v[192:195], a[64:67], a[48:51], v[192:195]\n\t"
      "global_load_dwordx4 a[40:43], v16, s[36:37] offset:0 \n\t"
      "v_mfma_f32_16x16x32_bf16 v[192:195], a[68:71], a[52:55], v[192:195]\n\t"
      "ds_read_b64_tr_b16 a[190:191], v25 offset:51200\n\t"
      "v_mfma_f32_16x16x32_bf16 v[196:199], a[72:75], a[48:51], v[196:199]\n\t"
      "global_load_dwordx4 a[44:47], v16, s[36:37] offset:1024 \n\t"
      "v_mfma_f32_16x16x32_bf16 v[196:199], a[76:79], a[52:55], v[196:199]\n\t"
      "v_mfma_f32_16x16x32_bf16 v[200:203], a[80:83], a[48:51], v[200:203]\n\t"
      "v_mfma_f32_16x16x32_bf16 v[200:203], a[84:87], a[52:55], v[200:203]\n\t"
      "v_mfma_f32_16x16x32_bf16 v[204:207], a[88:91], a[48:51], v[204:207]\n\t"
      "v_mfma_f32_16x16x32_bf16 v[204:207], a[92:95], a[52:55], v[204:207]\n\t"
      "v_mfma_f32_16x16x32_bf16 v[208:211], a[96:99], a[48:51], v[208:211]\n\t"
      "v_mfma_f32_16x16x32_bf16 v[208:211], a[100:103], a[52:55], v[208:211]\n\t"
      "v_mfma_f32_16x16x32_bf16 v[212:215], a[104:107], a[48:51], v[212:215]\n\t"
      "v_mfma_f32_16x16x32_bf16 v[212:215], a[108:111], a[52:55], v[212:215]\n\t"
      "v_mfma_f32_16x16x32_bf16 v[216:219], a[112:115], a[48:51], v[216:219]\n\t"
      "v_mfma_f32_16x16x32_bf16 v[216:219], a[116:119], a[52:55], v[216:219]\n\t"
      "v_mfma_f32_16x16x32_bf16 v[220:223], a[120:123], a[48:51], v[220:223]\n\t"
      "v_mfma_f32_16x16x32_bf16 v[220:223], a[124:127], a[52:55], v[220:223]\n\t"
      "s_waitcnt vmcnt(18)\n\t"
      "v_mfma_f32_16x16x32_bf16 v[224:227], a[64:67], a[56:59], v[224:227]\n\t"
      "global_load_dwordx4 a[48:51], v14, s[36:37] offset:0 \n\t"
      "v_mfma_f32_16x16x32_bf16 v[224:227], a[68:71], a[60:63], v[224:227]\n\t"
      "v_mfma_f32_16x16x32_bf16 v[228:231], a[72:75], a[56:59], v[228:231]\n\t"
      "global_load_dwordx4 a[52:55], v14, s[36:37] offset:1024 \n\t"
      "v_mfma_f32_16x16x32_bf16 v[228:231], a[76:79], a[60:63], v[228:231]\n\t"
      "s_waitcnt vmcnt(28) lgkmcnt(0)\n\t"
      "s_barrier\n\t"
      "v_mfma_f32_16x16x32_bf16 v[232:235], a[80:83], a[56:59], v[232:235]\n\t"
      "v_mfma_f32_16x16x32_bf16 v[232:235], a[84:87], a[60:63], v[232:235]\n\t"
      "ds_read_b64_tr_b16 a[64:65], v18 offset:0\n\t"
      "v_mfma_f32_16x16x32_bf16 v[236:239], a[88:91], a[56:59], v[236:239]\n\t"
      "s_add_u32 m0, s99, 49152\n\t"
      "s_add_u32 s14, s12, s49\n\t"
      "s_addc_u32 s15, s13, 0\n\t"
      "v_mfma_f32_16x16x32_bf16 v[236:239], a[92:95], a[60:63], v[236:239]\n\t"
      "ds_read_b64_tr_b16 a[66:67], v18 offset:2048\n\t"
      "v_mfma_f32_16x16x32_bf16 v[240:243], a[96:99], a[56:59], v[240:243]\n\t"
      "global_load_lds_dwordx4 v6, s[14:15] offset:0 \n\t"
      "v_mfma_f32_16x16x32_bf16 v[240:243], a[100:103], a[60:63], v[240:243]\n\t"
      "v_mfma_f32_16x16x32_bf16 v[244:247], a[104:107], a[56:59], v[244:247]\n\t"
      "v_mfma_f32_16x16x32_bf16 v[244:247], a[108:111], a[60:63], v[244:247]\n\t"
      "ds_read_b64_tr_b16 a[72:73], v19 offset:0\n\t"
      "v_mfma_f32_16x16x32_bf16 v[248:251], a[112:115], a[56:59], v[248:251]\n\t"
      "global_load_lds_dwordx4 v7, s[14:15] offset:1024 \n\t"
      "v_mfma_f32_16x16x32_bf16 v[248:251], a[116:119], a[60:63], v[248:251]\n\t"
      "ds_read_b64_tr_b16 a[74:75], v19 offset:2048\n\t"
      "v_mfma_f32_16x16x32_bf16 v[252:255], a[120:123], a[56:59], v[252:255]\n\t"
      "v_mfma_f32_16x16x32_bf16 v[252:255], a[124:127], a[60:63], v[252:255]\n\t"
      "s_waitcnt vmcnt(19)\n\t"
      "v_mfma_f32_16x16x32_bf16 v[32:35], a[128:131], a[0:3], v[32:35]\n\t"
      "global_load_dwordx4 a[56:59], v31, s[36:37] offset:0 \n\t"
      "v_mfma_f32_16x16x32_bf16 v[32:35], a[132:135], a[4:7], v[32:35]\n\t"
      "ds_read_b64_tr_b16 a[80:81], v20 offset:0\n\t"
      "v_mfma_f32_16x16x32_bf16 v[36:39], a[136:139], a[0:3], v[36:39]\n\t"
      "global_load_dwordx4 a[60:63], v31, s[36:37] offset:1024 \n\t"
      "v_mfma_f32_16x16x32_bf16 v[36:39], a[140:143], a[4:7], v[36:39]\n\t"
      "ds_read_b64_tr_b16 a[82:83], v20 offset:2048\n\t"
      "s_add_u32 s36, s36, 16384\n\t"
      "s_addc_u32 s37, s37, 0\n\t"
      "v_mfma_f32_16x16x32_bf16 v[40:43], a[144:147], a[0:3], v[40:43]\n\t"
      "v_mfma_f32_16x16x32_bf16 v[40:43], a[148:151], a[4:7], v[40:43]\n\t"
      "v_mfma_f32_16x16x32_bf16 v[44:47], a[152:155], a[0:3], v[44:47]\n\t"
      "v_mfma_f32_16x16x32_bf16 v[44:47], a[156:159], a[4:7], v[44:47]\n\t"
      "ds_read_b64_tr_b16 a[88:89], v21 offset:0\n\t"
      "v_mfma_f32_16x16x32_bf16 v[48:51], a[160:163], a[0:3], v[48:51]\n\t"
      "v_mfma_f32_16x16x32_bf16 v[48:51], a[164:167], a[4:7], v[48:51]\n\t"
      "ds_read_b64_tr_b16 a[90:91], v21 offset:2048\n\t"
      "v_mfma_f32_16x16x32_bf16 v[52:55], a[168:171], a[0:3], v[52:55]\n\t"
      "global_load_lds_dwordx4 v8, s[14:15] offset:2048 \n\t"
      "v_mfma_f32_16x16x32_bf16 v[52:55], a[172:175], a[4:7], v[52:55]\n\t"
      "v_mfma_f32_16x16x32_bf16 v[56:59], a[176:179], a[0:3], v[56:59]\n\t"
      "v_mfma_f32_16x16x32_bf16 v[56:59], a[180:183], a[4:7], v[56:59]\n\t"
      "ds_read_b64_tr_b16 a[96:97], v22 offset:0\n\t"
      "v_mfma_f32_16x16x32_bf16 v[60:63], a[184:187], a[0:3], v[60:63]\n\t"
      "v_mfma_f32_16x16x32_bf16 v[60:63], a[188:191], a[4:7], v[60:63]\n\t"
      "ds_read_b64_tr_b16 a[98:99], v22 offset:2048\n\t"
      "s_waitcnt vmcnt(19)\n\t"
      "v_mfma_f32_16x16x32_bf16 v[64:67], a[128:131], a[8:11], v[64:67]\n\t"
      "global_load_dwordx4 a[0:3], v1, s[36:37] offset:0 \n\t"
      "v_mfma_f32_16x16x32_bf16 v[64:67], a[132:135], a[12:15], v[64:67]\n\t"
      "v_mfma_f32_16x16x32_bf16 v[68:71], a[136:139], a[8:11], v[68:71]\n\t"
      "global_load_dwordx4 a[4:7], v1, s[36:37] offset:1024 \n\t"
      "v_mfma_f32_16x16x32_bf16 v[68:71], a[140:143], a[12:15], v[68:71]\n\t"
      "ds_read_b64_tr_b16 a[104:105], v23 offset:0\n\t"
      "v_mfma_f32_16x16x32_bf16 v[72:75], a[144:147], a[8:11], v[72:75]\n\t"
      "v_mfma_f32_16x16x32_bf16 v[72:75], a[148:151], a[12:15], v[72:75]\n\t"
      "ds_read_b64_tr_b16 a[106:107], v23 offset:2048\n\t"
      "v_mfma_f32_16x16x32_bf16 v[76:79], a[152:155], a[8:11], v[76:79]\n\t"
      "v_mfma_f32_16x16x32_bf16 v[76:79], a[156:159], a[12:15], v[76:79]\n\t"
      "v_mfma_f32_16x16x32_bf16 v[80:83], a[160:163], a[8:11], v[80:83]\n\t"
      "v_mfma_f32_16x16x32_bf16 v[80:83], a[164:167], a[12:15], v[80:83]\n\t"
      "ds_read_b64_tr_b16 a[112:113], v24 offset:0\n\t"
      "v_mfma_f32_16x16x32_bf16 v[84:87], a[168:171], a[8:11], v[84:87]\n\t"
      "global_load_lds_dwordx4 v9, s[14:15] offset:3072 \n\t"
      "v_mfma_f32_16x16x32_bf16 v[84:87], a[172:175], a[12:15], v[84:87]\n\t"
      "ds_read_b64_tr_b16 a[114:115], v24 offset:2048\n\t"
      "v_mfma_f32_16x16x32_bf16 v[88:91], a[176:179], a[8:11], v[88:91]\n\t"
      "s_add_u32 s49, s49, 0x200000\n\t"
      "v_mfma_f32_16x16x32_bf16 v[88:91], a[180:183], a[12:15], v[88:91]\n\t"
      "v_mfma_f32_16x16x32_bf16 v[92:95], a[184:187], a[8:11], v[92:95]\n\t"
      "s_and_b32 s49, s49, 0xfffffff\n\t"
      "v_mfma_f32_16x16x32_bf16 v[92:95], a[188:191], a[12:15], v[92:95]\n\t"
      "ds_read_b64_tr_b16 a[120:121], v25 offset:0\n\t"
      "s_waitcnt vmcnt(20)\n\t"
      "v_mfma_f32_16x16x32_bf16 v[96:99], a[128:131], a[16:19], v[96:99]\n\t"
      "global_load_dwordx4 a[8:11], v16, s[36:37] offset:0 \n\t"
      "v_mfma_f32_16x16x32_bf16 v[96:99], a[132:135], a[20:23], v[96:99]\n\t"
      "ds_read_b64_tr_b16 a[122:123], v25 offset:2048\n\t"
      "v_mfma_f32_16x16x32_bf16 v[100:103], a[136:139], a[16:19], v[100:103]\n\t"
      "global_load_dwordx4 a[12:15], v16, s[36:37] offset:1024 \n\t"
      "v_mfma_f32_16x16x32_bf16 v[100:103], a[140:143], a[20:23], v[100:103]\n\t"
      "v_mfma_f32_16x16x32_bf16 v[104:107], a[144:147], a[16:19], v[104:107]\n\t"
      "v_mfma_f32_16x16x32_bf16 v[104:107], a[148:151], a[20:23], v[104:107]\n\t"
      "v_mfma_f32_16x16x32_bf16 v[108:111], a[152:155], a[16:19], v[108:111]\n\t"
      "v_mfma_f32_16x16x32_bf16 v[108:111], a[156:159], a[20:23], v[108:111]\n\t"
      "v_mfma_f32_16x16x32_bf16 v[112:115], a[160:163], a[16:19], v[112:115]\n\t"
      "v_mfma_f32_16x16x32_bf16 v[112:115], a[164:167], a[20:23], v[112:115]\n\t"
      "v_mfma_f32_16x16x32_bf16 v[116:119], a[168:171], a[16:19], v[116:119]\n\t"
      "v_mfma_f32_16x16x32_bf16 v[116:119], a[172:175], a[20:23], v[116:119]\n\t"
      "v_mfma_f32_16x16x32_bf16 v[120:123], a[176:179], a[16:19], v[120:123]\n\t"
      "v_mfma_f32_16x16x32_bf16 v[120:123], a[180:183], a[20:23], v[120:123]\n\t"
      "v_mfma_f32_16x16x32_bf16 v[124:127], a[184:187], a[16:19], v[124:127]\n\t"
      "v_mfma_f32_16x16x32_bf16 v[124:127], a[188:191], a[20:23], v[124:127]\n\t"
      "s_waitcnt vmcnt(18)\n\t"
      "v_mfma_f32_16x16x32_bf16 a[192:195], a[128:131], a[24:27], a[192:195]\n\t"
      "global_load_dwordx4 a[16:19], v14, s[36:37] offset:0 \n\t"
      "v_mfma_f32_16x16x32_bf16 a[192:195], a[132:135], a[28:31], a[192:195]\n\t"
      "v_mfma_f32_16x16x32_bf16 a[196:199], a[136:139], a[24:27], a[196:199]\n\t"
      "global_load_dwordx4 a[20:23], v14, s[36:37] offset:1024 \n\t"
      "v_mfma_f32_16x16x32_bf16 a[196:199], a[140:143], a[28:31], a[196:199]\n\t"
      "s_waitcnt vmcnt(28) lgkmcnt(0)\n\t"
      "s_barrier\n\t"
      "v_mfma_f32_16x16x32_bf16 a[200:203], a[144:147], a[24:27], a[200:203]\n\t"
      "v_mfma_f32_16x16x32_bf16 a[200:203], a[148:151], a[28:31], a[200:203]\n\t"
      "ds_read_b64_tr_b16 a[68:69], v18 offset:16384\n\t"
      "v_mfma_f32_16x16x32_bf16 a[204:207], a[152:155], a[24:27], a[204:207]\n\t"
      "s_add_u32 m0, s99, 0\n\t"
      "s_add_u32 s14, s12, s49\n\t"
      "s_addc_u32 s15, s13, 0\n\t"
      "v_mfma_f32_16x16x32_bf16 a[204:207], a[156:159], a[28:31], a[204:207]\n\t"
      "ds_read_b64_tr_b16 a[70:71], v18 offset:18432\n\t"
      "v_mfma_f32_16x16x32_bf16 a[208:211], a[160:163], a[24:27], a[208:211]\n\t"
      "global_load_lds_dwordx4 v6, s[14:15] offset:0 \n\t"
      "v_mfma_f32_16x16x32_bf16 a[208:211], a[164:167], a[28:31], a[208:211]\n\t"
      "v_mfma_f32_16x16x32_bf16 a[212:215], a[168:171], a[24:27], a[212:215]\n\t"
      "v_mfma_f32_16x16x32_bf16 a[212:215], a[172:175], a[28:31], a[212:215]\n\t"
      "ds_read_b64_tr_b16 a[76:77], v19 offset:16384\n\t"
      "v_mfma_f32_16x16x32_bf16 a[216:219], a[176:179], a[24:27], a[216:219]\n\t"
      "global_load_lds_dwordx4 v7, s[14:15] offset:1024 \n\t"
      "v_mfma_f32_16x16x32_bf16 a[216:219], a[180:183], a[28:31], a[216:219]\n\t"
      "ds_read_b64_tr_b16 a[78:79], v19 offset:18432\n\t"
      "v_mfma_f32_16x16x32_bf16 a[220:223], a[184:187], a[24:27], a[220:223]\n\t"
      "v_mfma_f32_16x16x32_bf16 a[220:223], a[188:191], a[28:31], a[220:223]\n\t"
      "s_waitcnt vmcnt(19)\n\t"
      "v_mfma_f32_16x16x32_bf16 v[128:131], a[128:131], a[32:35], v[128:131]\n\t"
      "global_load_dwordx4 a[24:27], v31, s[36:37] offset:0 \n\t"
      "v_mfma_f32_16x16x32_bf16 v[128:131], a[132:135], a[36:39], v[128:131]\n\t"
      "ds_read_b64_tr_b16 a[84:85], v20 offset:16384\n\t"
      "v_mfma_f32_16x16x32_bf16 v[132:135], a[136:139], a[32:35], v[132:135]\n\t"
      "global_load_dwordx4 a[28:31], v31, s[36:37] offset:1024 \n\t"
      "v_mfma_f32_16x16x32_bf16 v[132:135], a[140:143], a[36:39], v[132:135]\n\t"
      "ds_read_b64_tr_b16 a[86:87], v20 offset:18432\n\t"
      "s_add_u32 s36, s36, 16384\n\t"
      "s_addc_u32 s37, s37, 0\n\t"
      "v_mfma_f32_16x16x32_bf16 v[136:139], a[144:147], a[32:35], v[136:139]\n\t"
      "v_mfma_f32_16x16x32_bf16 v[136:139], a[148:151], a[36:39], v[136:139]\n\t"
      "v_mfma_f32_16x16x32_bf16 v[140:143], a[152:155], a[32:35], v[140:143]\n\t"
      "v_mfma_f32_16x16x32_bf16 v[140:143], a[156:159], a[36:39], v[140:143]\n\t"
      "ds_read_b64_tr_b16 a[92:93], v21 offset:16384\n\t"
      "v_mfma_f32_16x16x32_bf16 v[144:147], a[160:163], a[32:35], v[144:147]\n\t"
      "v_mfma_f32_16x16x32_bf16 v[144:147], a[164:167], a[36:39], v[144:147]\n\t"
      "ds_read_b64_tr_b16 a[94:95], v21 offset:18432\n\t"
      "v_mfma_f32_16x16x32_bf16 v[148:151], a[168:171], a[32:35], v[148:151]\n\t"
      "global_load_lds_dwordx4 v8, s[14:15] offset:2048 \n\t"
      "v_mfma_f32_16x16x32_bf16 v[148:151], a[172:175], a[36:39], v[148:151]\n\t"
      "v_mfma_f32_16x16x32_bf16 v[152:155], a[176:179], a[32:35], v[152:155]\n\t"
      "v_mfma_f32_16x16x32_bf16 v[152:155], a[180:183], a[36:39], v[152:155]\n\t"
      "ds_read_b64_tr_b16 a[100:101], v22 offset:16384\n\t"
      "v_mfma_f32_16x16x32_bf16 v[156:159], a[184:187], a[32:35], v[156:159]\n\t"
      "v_mfma_f32_16x16x32_bf16 v[156:159], a[188:191], a[36:39], v[156:159]\n\t"
      "ds_read_b64_tr_b16 a[102:103], v22 offset:18432\n\t"
      "s_waitcnt vmcnt(19)\n\t"
      "v_mfma_f32_16x16x32_bf16 v[160:163], a[128:131], a[40:43], v[160:163]\n\t"
      "global_load_dwordx4 a[32:35], v1, s[36:37] offset:0 \n\t"
      "v_mfma_f32_16x16x32_bf16 v[160:163], a[132:135], a[44:47], v[160:163]\n\t"
      "v_mfma_f32_16x16x32_bf16 v[164:167], a[136:139], a[40:43], v[164:167]\n\t"
      "global_load_dwordx4 a[36:39], v1, s[36:37] offset:1024 \n\t"
      "v_mfma_f32_16x16x32_bf16 v[164:167], a[140:143], a[44:47], v[164:167]\n\t"
      "ds_read_b64_tr_b16 a[108:109], v23 offset:16384\n\t"
      "v_mfma_f32_16x16x32_bf16 v[168:171], a[144:147], a[40:43], v[168:171]\n\t"
      "v_mfma_f32_16x16x32_bf16 v[168:171], a[148:151], a[44:47], v[168:171]\n\t"
      "ds_read_b64_tr_b16 a[110:111], v23 offset:18432\n\t"
      "v_mfma_f32_16x16x32_bf16 v[172:175], a[152:155], a[40:43], v[172:175]\n\t"
      "v_mfma_f32_16x16x32_bf16 v[172:175], a[156:159], a[44:47], v[172:175]\n\t"
      "v_mfma_f32_16x16x32_bf16 v[176:179], a[160:163], a[40:43], v[176:179]\n\t"
      "v_mfma_f32_16x16x32_bf16 v[176:179], a[164:167], a[44:47], v[176:179]\n\t"
      "ds_read_b64_tr_b16 a[116:117], v24 offset:16384\n\t"
      "v_mfma_f32_16x16x32_bf16 v[180:183], a[168:171], a[40:43], v[180:183]\n\t"
      "global_load_lds_dwordx4 v9, s[14:15] offset:3072 \n\t"
      "v_mfma_f32_16x16x32_bf16 v[180:183], a[172:175], a[44:47], v[180:183]\n\t"
      "ds_read_b64_tr_b16 a[118:119], v24 offset:18432\n\t"
      "v_mfma_f32_16x16x32_bf16 v[184:187], a[176:179], a[40:43], v[184:187]\n\t"
      "s_add_u32 s49, s49, 0x200000\n\t"
      "v_mfma_f32_16x16x32_bf16 v[184:187], a[180:183], a[44:47], v[184:187]\n\t"
      "v_mfma_f32_16x16x32_bf16 v[188:191], a[184:187], a[40:43], v[188:191]\n\t"
      "s_and_b32 s49, s49, 0xfffffff\n\t"
      "v_mfma_f32_16x16x32_bf16 v[188:191], a[188:191], a[44:47], v[188:191]\n\t"
      "ds_read_b64_tr_b16 a[124:125], v25 offset:16384\n\t"
      "s_waitcnt vmcnt(20)\n\t"
      "v_mfma_f32_16x16x32_bf16 v[192:195], a[128:131], a[48:51], v[192:195]\n\t"
      "global_load_dwordx4 a[40:43], v16, s[36:37] offset:0 \n\t"
      "v_mfma_f32_16x16x32_bf16 v[192:195], a[132:135], a[52:55], v[192:195]\n\t"
      "ds_read_b64_tr_b16 a[126:127], v25 offset:18432\n\t"
      "v_mfma_f32_16x16x32_bf16 v[196:199], a[136:139], a[48:51], v[196:199]\n\t"
      "global_load_dwordx4 a[44:47], v16, s[36:37] offset:1024 \n\t"
      "v_mfma_f32_16x16x32_bf16 v[196:199], a[140:143], a[52:55], v[196:199]\n\t"
      "v_mfma_f32_16x16x32_bf16 v[200:203], a[144:147], a[48:51], v[200:203]\n\t"
      "v_mfma_f32_16x16x32_bf16 v[200:203], a[148:151], a[52:55], v[200:203]\n\t"
      "v_mfma_f32_16x16x32_bf16 v[204:207], a[152:155], a[48:51], v[204:207]\n\t"
      "v_mfma_f32_16x16x32_bf16 v[204:207], a[156:159], a[52:55], v[204:207]\n\t"
      "v_mfma_f32_16x16x32_bf16 v[208:211], a[160:163], a[48:51], v[208:211]\n\t"
      "v_mfma_f32_16x16x32_bf16 v[208:211], a[164:167], a[52:55], v[208:211]\n\t"
      "v_mfma_f32_16x16x32_bf16 v[212:215], a[168:171], a[48:51], v[212:215]\n\t"
      "v_mfma_f32_16x16x32_bf16 v[212:215], a[172:175], a[52:55], v[212:215]\n\t"
      "s_mov_b64 exec, s[56:57]\n\t"
      "global_load_dwordx2 v[2:3], v11, s[88:89]\n\t"
      "s_mov_b64 exec, -1\n\t"
      "v_mfma_f32_16x16x32_bf16 v[216:219], a[176:179], a[48:51], v[216:219]\n\t"
      "v_mfma_f32_16x16x32_bf16 v[216:219], a[180:183], a[52:55], v[216:219]\n\t"
      "v_mfma_f32_16x16x32_bf16 v[220:223], a[184:187], a[48:51], v[220:223]\n\t"
      "v_mfma_f32_16x16x32_bf16 v[220:223], a[188:191], a[52:55], v[220:223]\n\t"
      "s_waitcnt vmcnt(19)\n\t"
      "v_mfma_f32_16x16x32_bf16 v[224:227], a[128:131], a[56:59], v[224:227]\n\t"
      "global_load_dwordx4 a[48:51], v14, s[36:37] offset:0 \n\t"
      "v_mfma_f32_16x16x32_bf16 v[224:227], a[132:135], a[60:63], v[224:227]\n\t"
      "v_mfma_f32_16x16x32_bf16 v[228:231], a[136:139], a[56:59], v[228:231]\n\t"
      "global_load_dwordx4 a[52:55], v14, s[36:37] offset:1024 \n\t"
      "v_mfma_f32_16x16x32_bf16 v[228:231], a[140:143], a[60:63], v[228:231]\n\t"
      "s_waitcnt vmcnt(28) lgkmcnt(0)\n\t"
      "s_branch MID_%=\n\t"
      ".p2align 6\n\t"
      "LOOP_%=:\n\t"
      "v_mfma_f32_16x16x32_bf16 v[220:223], a[184:187], a[48:51], v[220:223]\n\t"
      "v_mfma_f32_16x16x32_bf16 v[220:223], a[188:191], a[52:55], v[220:223]\n\t"
      "s_waitcnt vmcnt(18)\n\t"
      "v_mfma_f32_16x16x32_bf16 v[224:227], a[128:131], a[56:59], v[224:227]\n\t"
      "global_load_dwordx4 a[48:51], v14, s[36:37] offset:0 \n\t"
      "v_mfma_f32_16x16x32_bf16 v[224:227], a[132:135], a[60:63], v[224:227]\n\t"
      "global_load_dwordx4 a[52:55], v14, s[36:37] offset:1024 \n\t"
      "v_mfma_f32_16x16x32_bf16 v[228:231], a[136:139], a[56:59], v[228:231]\n\t"
      "s_waitcnt vmcnt(28) lgkmcnt(0)\n\t"
      "v_mfma_f32_16x16x32_bf16 v[228:231], a[140:143], a[60:63], v[228:231]\n\t"
      "MID_%=:\n\t"
      "s_barrier\n\t"
      "v_mfma_f32_16x16x32_bf16 v[232:235], a[144:147], a[56:59], v[232:235]\n\t"
      "v_mfma_f32_16x16x32_bf16 v[232:235], a[148:151], a[60:63], v[232:235]\n\t"
      "ds_read_b64_tr_b16 a[128:129], v18 offset:32768\n\t"
      "v_mfma_f32_16x16x32_bf16 v[236:239], a[152:155], a[56:59], v[236:239]\n\t"
      "s_add_u32 m0, s99, 16384\n\t"
      "s_add_u32 s14, s12, s49\n\t"
      "s_addc_u32 s15, s13, 0\n\t"
      "v_mfma_f32_16x16x32_bf16 v[236:239], a[156:159], a[60:63], v[236:239]\n\t"
      "ds_read_b64_tr_b16 a[130:131], v18 offset:34816\n\t"
      "v_mfma_f32_16x16x32_bf16 v[240:243], a[160:163], a[56:59], v[240:243]\n\t"
      "global_load_lds_dwordx4 v6, s[14:15] offset:0 \n\t"
      "v_mfma_f32_16x16x32_bf16 v[240:243], a[164:167], a[60:63], v[240:243]\n\t"
      "v_mfma_f32_16x16x32_bf16 v[244:247], a[168:171], a[56:59], v[244:247]\n\t"
      "v_mfma_f32_16x16x32_bf16 v[244:247], a[172:175], a[60:63], v[244:247]\n\t"
      "ds_read_b64_tr_b16 a[136:137], v19 offset:32768\n\t"
      "v_mfma_f32_16x16x32_bf16 v[248:251], a[176:179], a[56:59], v[248:251]\n\t"
      "global_load_lds_dwordx4 v7, s[14:15] offset:1024 \n\t"
      "v_mfma_f32_16x16x32_bf16 v[248:251], a[180:183], a[60:63], v[248:251]\n\t"
      "ds_read_b64_tr_b16 a[138:139], v19 offset:34816\n\t"
      "v_mfma_f32_16x16x32_bf16 v[252:255], a[184:187], a[56:59], v[252:255]\n\t"
      "v_mfma_f32_16x16x32_bf16 v[252:255], a[188:191], a[60:63], v[252:255]\n\t"
      "s_waitcnt vmcnt(19)\n\t"
      "v_mfma_f32_16x16x32_bf16 v[32:35], a[64:67], a[0:3], v[32:35]\n\t"
      "global_load_dwordx4 a[56:59], v31, s[36:37] offset:0 \n\t"
      "v_mfma_f32_16x16x32_bf16 v[32:35], a[68:71], a[4:7], v[32:35]\n\t"
      "ds_read_b64_tr_b16 a[144:145], v20 offset:32768\n\t"
      "v_mfma_f32_16x16x32_bf16 v[36:39], a[72:75], a[0:3], v[36:39]\n\t"
      "v_mfma_f32_16x16x32_bf16 v[36:39], a[76:79], a[4:7], v[36:39]\n\t"
      "ds_read_b64_tr_b16 a[146:147], v20 offset:34816\n\t"
      "v_mfma_f32_16x16x32_bf16 v[40:43], a[80:83], a[0:3], v[40:43]\n\t"
      "global_load_dwordx4 a[60:63], v31, s[36:37] offset:1024 \n\t"
      "v_mfma_f32_16x16x32_bf16 v[40:43], a[84:87], a[4:7], v[40:43]\n\t"
      "v_mfma_f32_16x16x32_bf16 v[44:47], a[88:91], a[0:3], v[44:47]\n\t"
      "s_add_u32 s36, s36, 16384\n\t"
      "s_addc_u32 s37, s37, 0\n\t"
      "v_mfma_f32_16x16x32_bf16 v[44:47], a[92:95], a[4:7], v[44:47]\n\t"
      "ds_read_b64_tr_b16 a[152:153], v21 offset:32768\n\t"
      "v_mfma_f32_16x16x32_bf16 v[48:51], a[96:99], a[0:3], v[48:51]\n\t"
      "v_mfma_f32_16x16x32_bf16 v[48:51], a[100:103], a[4:7], v[48:51]\n\t"
      "ds_read_b64_tr_b16 a[154:155], v21 offset:34816\n\t"
      "v_mfma_f32_16x16x32_bf16 v[52:55], a[104:107], a[0:3], v[52:55]\n\t"
      "global_load_lds_dwordx4 v8, s[14:15] offset:2048 \n\t"
      "v_mfma_f32_16x16x32_bf16 v[52:55], a[108:111], a[4:7], v[52:55]\n\t"
      "v_mfma_f32_16x16x32_bf16 v[56:59], a[112:115], a[0:3], v[56:59]\n\t"
      "v_mfma_f32_16x16x32_bf16 v[56:59], a[116:119], a[4:7], v[56:59]\n\t"
      "ds_read_b64_tr_b16 a[160:161], v22 offset:32768\n\t"
      "v_mfma_f32_16x16x32_bf16 v[60:63], a[120:123], a[0:3], v[60:63]\n\t"
      "v_mfma_f32_16x16x32_bf16 v[60:63], a[124:127], a[4:7], v[60:63]\n\t"
      "ds_read_b64_tr_b16 a[162:163], v22 offset:34816\n\t"
      "s_waitcnt vmcnt(19)\n\t"
      "v_mfma_f32_16x16x32_bf16 v[64:67], a[64:67], a[8:11], v[64:67]\n\t"
      "global_load_dwordx4 a[0:3], v1, s[36:37] offset:0 \n\t"
      "v_mfma_f32_16x16x32_bf16 v[64:67], a[68:71], a[12:15], v[64:67]\n\t"
      "v_mfma_f32_16x16x32_bf16 v[68:71], a[72:75], a[8:11], v[68:71]\n\t"
      "v_mfma_f32_16x16x32_bf16 v[68:71], a[76:79], a[12:15], v[68:71]\n\t"
      "ds_read_b64_tr_b16 a[168:169], v23 offset:32768\n\t"
      "v_mfma_f32_16x16x32_bf16 v[72:75], a[80:83], a[8:11], v[72:75]\n\t"
      "global_load_dwordx4 a[4:7], v1, s[36:37] offset:1024 \n\t"
      "v_mfma_f32_16x16x32_bf16 v[72:75], a[84:87], a[12:15], v[72:75]\n\t"
      "ds_read_b64_tr_b16 a[170:171], v23 offset:34816\n\t"
      "v_mfma_f32_16x16x32_bf16 v[76:79], a[88:91], a[8:11], v[76:79]\n\t"
      "v_mfma_f32_16x16x32_bf16 v[76:79], a[92:95], a[12:15], v[76:79]\n\t"
      "v_mfma_f32_16x16x32_bf16 v[80:83], a[96:99], a[8:11], v[80:83]\n\t"
      "v_mfma_f32_16x16x32_bf16 v[80:83], a[100:103], a[12:15], v[80:83]\n\t"
      "ds_read_b64_tr_b16 a[176:177], v24 offset:32768\n\t"
      "v_mfma_f32_16x16x32_bf16 v[84:87], a[104:107], a[8:11], v[84:87]\n\t"
      "global_load_lds_dwordx4 v9, s[14:15] offset:3072 \n\t"
      "v_mfma_f32_16x16x32_bf16 v[84:87], a[108:111], a[12:15], v[84:87]\n\t"
      "ds_read_b64_tr_b16 a[178:179], v24 offset:34816\n\t"
      "v_mfma_f32_16x16x32_bf16 v[88:91], a[112:115], a[8:11], v[88:91]\n\t"
      "s_add_u32 s49, s49, 0x200000\n\t"
      "v_mfma_f32_16x16x32_bf16 v[88:91], a[116:119], a[12:15], v[88:91]\n\t"
      "v_mfma_f32_16x16x32_bf16 v[92:95], a[120:123], a[8:11], v[92:95]\n\t"
      "s_and_b32 s49, s49, 0xfffffff\n\t"
      "v_mfma_f32_16x16x32_bf16 v[92:95], a[124:127], a[12:15], v[92:95]\n\t"
      "ds_read_b64_tr_b16 a[184:185], v25 offset:32768\n\t"
      "s_waitcnt vmcnt(20)\n\t"
      "v_mfma_f32_16x16x32_bf16 v[96:99], a[64:67], a[16:19], v[96:99]\n\t"
      "global_load_dwordx4 a[8:11], v16, s[36:37] offset:0 \n\t"
      "v_mfma_f32_16x16x32_bf16 v[96:99], a[68:71], a[20:23], v[96:99]\n\t"
      "ds_read_b64_tr_b16 a[186:187], v25 offset:34816\n\t"
      "v_mfma_f32_16x16x32_bf16 v[100:103], a[72:75], a[16:19], v[100:103]\n\t"
      "v_mfma_f32_16x16x32_bf16 v[100:103], a[76:79], a[20:23], v[100:103]\n\t"
      "v_mfma_f32_16x16x32_bf16 v[104:107], a[80:83], a[16:19], v[104:107]\n\t"
      "global_load_dwordx4 a[12:15], v16, s[36:37] offset:1024 \n\t"
      "v_mfma_f32_16x16x32_bf16 v[104:107], a[84:87], a[20:23], v[104:107]\n\t"
      "s_cmp_eq_u32 s50, 0\n\t"
      "v_mfma_f32_16x16x32_bf16 v[108:111], a[88:91], a[16:19], v[108:111]\n\t"
      "v_mfma_f32_16x16x32_bf16 v[108:111], a[92:95], a[20:23], v[108:111]\n\t"
      "s_cselect_b64 exec, -1, 0\n\t"
      "v_mfma_f32_16x16x32_bf16 v[112:115], a[96:99], a[16:19], v[112:115]\n\t"
      "global_load_dword v5, v15, s[92:93]\n\t"
      "v_mfma_f32_16x16x32_bf16 v[112:115], a[100:103], a[20:23], v[112:115]\n\t"
      "global_load_dword v29, v27, s[40:41] sc1\n\t"
      "v_mfma_f32_16x16x32_bf16 v[116:119], a[104:107], a[16:19], v[116:119]\n\t"
      "s_mov_b64 exec, -1\n\t"
      "v_mfma_f32_16x16x32_bf16 v[116:119], a[108:111], a[20:23], v[116:119]\n\t"
      "v_mfma_f32_16x16x32_bf16 v[120:123], a[112:115], a[16:19], v[120:123]\n\t"
      "v_mfma_f32_16x16x32_bf16 v[120:123], a[116:119], a[20:23], v[120:123]\n\t"
      "v_mfma_f32_16x16x32_bf16 v[124:127], a[120:123], a[16:19], v[124:127]\n\t"
      "v_mfma_f32_16x16x32_bf16 v[124:127], a[124:127], a[20:23], v[124:127]\n\t"
      "s_waitcnt vmcnt(20)\n\t"
      "v_mfma_f32_16x16x32_bf16 a[192:195], a[64:67], a[24:27], a[192:195]\n\t"
      "global_load_dwordx4 a[16:19], v14, s[36:37] offset:0 \n\t"
      "v_mfma_f32_16x16x32_bf16 a[192:195], a[68:71], a[28:31], a[192:195]\n\t"
      "global_load_dwordx4 a[20:23], v14, s[36:37] offset:1024 \n\t"
      "v_mfma_f32_16x16x32_bf16 a[196:199], a[72:75], a[24:27], a[196:199]\n\t"
      "s_waitcnt vmcnt(30) lgkmcnt(0)\n\t"
      "v_mfma_f32_16x16x32_bf16 a[196:199], a[76:79], a[28:31], a[196:199]\n\t"
      "s_barrier\n\t"
      "v_mfma_f32_16x16x32_bf16 a[200:203], a[80:83], a[24:27], a[200:203]\n\t"
      "v_mfma_f32_16x16x32_bf16 a[200:203], a[84:87], a[28:31], a[200:203]\n\t"
      "ds_read_b64_tr_b16 a[132:133], v18 offset:49152\n\t"
      "v_mfma_f32_16x16x32_bf16 a[204:207], a[88:91], a[24:27], a[204:207]\n\t"
      "s_add_u32 m0, s99, 32768\n\t"
      "s_add_u32 s14, s12, s49\n\t"
      "s_addc_u32 s15, s13, 0\n\t"
      "v_mfma_f32_16x16x32_bf16 a[204:207], a[92:95], a[28:31], a[204:207]\n\t"
      "ds_read_b64_tr_b16 a[134:135], v18 offset:51200\n\t"
      "v_mfma_f32_16x16x32_bf16 a[208:211], a[96:99], a[24:27], a[208:211]\n\t"
      "global_load_lds_dwordx4 v6, s[14:15] offset:0 \n\t"
      "v_mfma_f32_16x16x32_bf16 a[208:211], a[100:103], a[28:31], a[208:211]\n\t"
      "v_mfma_f32_16x16x32_bf16 a[212:215], a[104:107], a[24:27], a[212:215]\n\t"
      "v_mfma_f32_16x16x32_bf16 a[212:215], a[108:111], a[28:31], a[212:215]\n\t"
      "ds_read_b64_tr_b16 a[140:141], v19 offset:49152\n\t"
      "v_mfma_f32_16x16x32_bf16 a[216:219], a[112:115], a[24:27], a[216:219]\n\t"
      "global_load_lds_dwordx4 v7, s[14:15] offset:1024 \n\t"
      "v_mfma_f32_16x16x32_bf16 a[216:219], a[116:119], a[28:31], a[216:219]\n\t"
      "ds_read_b64_tr_b16 a[142:143], v19 offset:51200\n\t"
      "v_mfma_f32_16x16x32_bf16 a[220:223], a[120:123], a[24:27], a[220:223]\n\t"
      "v_mfma_f32_16x16x32_bf16 a[220:223], a[124:127], a[28:31], a[220:223]\n\t"
      "s_waitcnt vmcnt(21)\n\t"
      "v_mfma_f32_16x16x32_bf16 v[128:131], a[64:67], a[32:35], v[128:131]\n\t"
      "global_load_dwordx4 a[24:27], v31, s[36:37] offset:0 \n\t"
      "v_mfma_f32_16x16x32_bf16 v[128:131], a[68:71], a[36:39], v[128:131]\n\t"
      "ds_read_b64_tr_b16 a[148:149], v20 offset:49152\n\t"
      "v_mfma_f32_16x16x32_bf16 v[132:135], a[72:75], a[32:35], v[132:135]\n\t"
      "v_mfma_f32_16x16x32_bf16 v[132:135], a[76:79], a[36:39], v[132:135]\n\t"
      "ds_read_b64_tr_b16 a[150:151], v20 offset:51200\n\t"
      "v_mfma_f32_16x16x32_bf16 v[136:139], a[80:83], a[32:35], v[136:139]\n\t"
      "global_load_dwordx4 a[28:31], v31, s[36:37] offset:1024 \n\t"
      "v_mfma_f32_16x16x32_bf16 v[136:139], a[84:87], a[36:39], v[136:139]\n\t"
      "v_mfma_f32_16x16x32_bf16 v[140:143], a[88:91], a[32:35], v[140:143]\n\t"
      "s_add_u32 s36, s36, 16384\n\t"
      "s_addc_u32 s37, s37, 0\n\t"
      "v_mfma_f32_16x16x32_bf16 v[140:143], a[92:95], a[36:39], v[140:143]\n\t"
      "ds_read_b64_tr_b16 a[156:157], v21 offset:49152\n\t"
      "v_mfma_f32_16x16x32_bf16 v[144:147], a[96:99], a[32:35], v[144:147]\n\t"
      "v_mfma_f32_16x16x32_bf16 v[144:147], a[100:103], a[36:39], v[144:147]\n\t"
      "ds_read_b64_tr_b16 a[158:159], v21 offset:51200\n\t"
      "v_mfma_f32_16x16x32_bf16 v[148:151], a[104:107], a[32:35], v[148:151]\n\t"
      "global_load_lds_dwordx4 v8, s[14:15] offset:2048 \n\t"
      "v_mfma_f32_16x16x32_bf16 v[148:151], a[108:111], a[36:39], v[148:151]\n\t"
      "v_mfma_f32_16x16x32_bf16 v[152:155], a[112:115], a[32:35], v[152:155]\n\t"
      "v_mfma_f32_16x16x32_bf16 v[152:155], a[116:119], a[36:39], v[152:155]\n\t"
      "ds_read_b64_tr_b16 a[164:165], v22 offset:49152\n\t"
      "v_mfma_f32_16x16x32_bf16 v[156:159], a[120:123], a[32:35], v[156:159]\n\t"
      "v_mfma_f32_16x16x32_bf16 v[156:159], a[124:127], a[36:39], v[156:159]\n\t"
      "ds_read_b64_tr_b16 a[166:167], v22 offset:51200\n\t"
      "s_waitcnt vmcnt(21)\n\t"
      "v_mfma_f32_16x16x32_bf16 v[160:163], a[64:67], a[40:43], v[160:163]\n\t"
      "global_load_dwordx4 a[32:35], v1, s[36:37] offset:0 \n\t"
      "v_mfma_f32_16x16x32_bf16 v[160:163], a[68:71], a[44:47], v[160:163]\n\t"
      "v_mfma_f32_16x16x32_bf16 v[164:167], a[72:75], a[40:43], v[164:167]\n\t"
      "v_mfma_f32_16x16x32_bf16 v[164:167], a[76:79], a[44:47], v[164:167]\n\t"
      "ds_read_b64_tr_b16 a[172:173], v23 offset:49152\n\t"
      "v_mfma_f32_16x16x32_bf16 v[168:171], a[80:83], a[40:43], v[168:171]\n\t"
      "global_load_dwordx4 a[36:39], v1, s[36:37] offset:1024 \n\t"
      "v_mfma_f32_16x16x32_bf16 v[168:171], a[84:87], a[44:47], v[168:171]\n\t"
      "ds_read_b64_tr_b16 a[174:175], v23 offset:51200\n\t"
      "v_mfma_f32_16x16x32_bf16 v[172:175], a[88:91], a[40:43], v[172:175]\n\t"
      "v_mfma_f32_16x16x32_bf16 v[172:175], a[92:95], a[44:47], v[172:175]\n\t"
      "v_mfma_f32_16x16x32_bf16 v[176:179], a[96:99], a[40:43], v[176:179]\n\t"
      "v_mfma_f32_16x16x32_bf16 v[176:179], a[100:103], a[44:47], v[176:179]\n\t"
      "ds_read_b64_tr_b16 a[180:181], v24 offset:49152\n\t"
      "v_mfma_f32_16x16x32_bf16 v[180:183], a[104:107], a[40:43], v[180:183]\n\t"
      "global_load_lds_dwordx4 v9, s[14:15] offset:3072 \n\t"
      "v_mfma_f32_16x16x32_bf16 v[180:183], a[108:111], a[44:47], v[180:183]\n\t"
      "ds_read_b64_tr_b16 a[182:183], v24 offset:51200\n\t"
      "v_mfma_f32_16x16x32_bf16 v[184:187], a[112:115], a[40:43], v[184:187]\n\t"
      "s_add_u32 s49, s49, 0x200000\n\t"
      "v_mfma_f32_16x16x32_bf16 v[184:187], a[116:119], a[44:47], v[184:187]\n\t"
      "v_mfma_f32_16x16x32_bf16 v[188:191], a[120:123], a[40:43], v[188:191]\n\t"
      "s_and_b32 s49, s49, 0xfffffff\n\t"
      "v_mfma_f32_16x16x32_bf16 v[188:191], a[124:127], a[44:47], v[188:191]\n\t"
      "ds_read_b64_tr_b16 a[188:189], v25 offset:49152\n\t"
      "s_waitcnt vmcnt(22)\n\t"
      "v_mfma_f32_16x16x32_bf16 v[192:195], a[64:67], a[48:51], v[192:195]\n\t"
      "global_load_dwordx4 a[40:43], v16, s[36:37] offset:0 \n\t"
      "v_mfma_f32_16x16x32_bf16 v[192:195], a[68:71], a[52:55], v[192:195]\n\t"
      "ds_read_b64_tr_b16 a[190:191], v25 offset:51200\n\t"
      "v_mfma_f32_16x16x32_bf16 v[196:199], a[72:75], a[48:51], v[196:199]\n\t"
      "v_mfma_f32_16x16x32_bf16 v[196:199], a[76:79], a[52:55], v[196:199]\n\t"
      "v_mfma_f32_16x16x32_bf16 v[200:203], a[80:83], a[48:51], v[200:203]\n\t"
      "global_load_dwordx4 a[44:47], v16, s[36:37] offset:1024 \n\t"
      "v_mfma_f32_16x16x32_bf16 v[200:203], a[84:87], a[52:55], v[200:203]\n\t"
      "v_mfma_f32_16x16x32_bf16 v[204:207], a[88:91], a[48:51], v[204:207]\n\t"
      "v_mfma_f32_16x16x32_bf16 v[204:207], a[92:95], a[52:55], v[204:207]\n\t"
      "v_mfma_f32_16x16x32_bf16 v[208:211], a[96:99], a[48:51], v[208:211]\n\t"
      "v_mfma_f32_16x16x32_bf16 v[208:211], a[100:103], a[52:55], v[208:211]\n\t"
      "v_mfma_f32_16x16x32_bf16 v[212:215], a[104:107], a[48:51], v[212:215]\n\t"
      "v_mfma_f32_16x16x32_bf16 v[212:215], a[108:111], a[52:55], v[212:215]\n\t"
      "v_mfma_f32_16x16x32_bf16 v[216:219], a[112:115], a[48:51], v[216:219]\n\t"
      "v_mfma_f32_16x16x32_bf16 v[216:219], a[116:119], a[52:55], v[216:219]\n\t"
      "v_mfma_f32_16x16x32_bf16 v[220:223], a[120:123], a[48:51], v[220:223]\n\t"
      "v_mfma_f32_16x16x32_bf16 v[220:223], a[124:127], a[52:55], v[220:223]\n\t"
      "s_waitcnt vmcnt(20)\n\t"
      "v_mfma_f32_16x16x32_bf16 v[224:227], a[64:67], a[56:59], v[224:227]\n\t"
      "global_load_dwordx4 a[48:51], v14, s[36:37] offset:0 \n\t"
      "v_mfma_f32_16x16x32_bf16 v[224:227], a[68:71], a[60:63], v[224:227]\n\t"
      "global_load_dwordx4 a[52:55], v14, s[36:37] offset:1024 \n\t"
      "v_mfma_f32_16x16x32_bf16 v[228:231], a[72:75], a[56:59], v[228:231]\n\t"
      "s_waitcnt vmcnt(30) lgkmcnt(0)\n\t"
      "v_mfma_f32_16x16x32_bf16 v[228:231], a[76:79], a[60:63], v[228:231]\n\t"
      "s_barrier\n\t"
      "v_mfma_f32_16x16x32_bf16 v[232:235], a[80:83], a[56:59], v[232:235]\n\t"
      "v_mfma_f32_16x16x32_bf16 v[232:235], a[84:87], a[60:63], v[232:235]\n\t"
      "ds_read_b64_tr_b16 a[64:65], v18 offset:0\n\t"
      "v_mfma_f32_16x16x32_bf16 v[236:239], a[88:91], a[56:59], v[236:239]\n\t"
      "s_add_u32 m0, s99, 49152\n\t"
      "s_add_u32 s14, s12, s49\n\t"
      "s_addc_u32 s15, s13, 0\n\t"
      "v_mfma_f32_16x16x32_bf16 v[236:239], a[92:95], a[60:63], v[236:239]\n\t"
      "ds_read_b64_tr_b16 a[66:67], v18 offset:2048\n\t"
      "v_mfma_f32_16x16x32_bf16 v[240:243], a[96:99], a[56:59], v[240:243]\n\t"
      "global_load_lds_dwordx4 v6, s[14:15] offset:0 \n\t"
      "v_mfma_f32_16x16x32_bf16 v[240:243], a[100:103], a[60:63], v[240:243]\n\t"
      "v_mfma_f32_16x16x32_bf16 v[244:247], a[104:107], a[56:59], v[244:247]\n\t"
      "v_mfma_f32_16x16x32_bf16 v[244:247], a[108:111], a[60:63], v[244:247]\n\t"
      "ds_read_b64_tr_b16 a[72:73], v19 offset:0\n\t"
      "v_mfma_f32_16x16x32_bf16 v[248:251], a[112:115], a[56:59], v[248:251]\n\t"
      "global_load_lds_dwordx4 v7, s[14:15] offset:1024 \n\t"
      "v_mfma_f32_16x16x32_bf16 v[248:251], a[116:119], a[60:63], v[248:251]\n\t"
      "ds_read_b64_tr_b16 a[74:75], v19 offset:2048\n\t"
      "v_mfma_f32_16x16x32_bf16 v[252:255], a[120:123], a[56:59], v[252:255]\n\t"
      "v_mfma_f32_16x16x32_bf16 v[252:255], a[124:127], a[60:63], v[252:255]\n\t"
      "s_waitcnt vmcnt(21)\n\t"
      "v_mfma_f32_16x16x32_bf16 v[32:35], a[128:131], a[0:3], v[32:35]\n\t"
      "global_load_dwordx4 a[56:59], v31, s[36:37] offset:0 \n\t"
      "v_mfma_f32_16x16x32_bf16 v[32:35], a[132:135], a[4:7], v[32:35]\n\t"
      "ds_read_b64_tr_b16 a[80:81], v20 offset:0\n\t"
      "v_mfma_f32_16x16x32_bf16 v[36:39], a[136:139], a[0:3], v[36:39]\n\t"
      "v_mfma_f32_16x16x32_bf16 v[36:39], a[140:143], a[4:7], v[36:39]\n\t"
      "ds_read_b64_tr_b16 a[82:83], v20 offset:2048\n\t"
      "v_mfma_f32_16x16x32_bf16 v[40:43], a[144:147], a[0:3], v[40:43]\n\t"
      "global_load_dwordx4 a[60:63], v31, s[36:37] offset:1024 \n\t"
      "v_mfma_f32_16x16x32_bf16 v[40:43], a[148:151], a[4:7], v[40:43]\n\t"
      "v_mfma_f32_16x16x32_bf16 v[44:47], a[152:155], a[0:3], v[44:47]\n\t"
      "s_add_u32 s36, s36, 16384\n\t"
      "s_addc_u32 s37, s37, 0\n\t"
      "v_mfma_f32_16x16x32_bf16 v[44:47], a[156:159], a[4:7], v[44:47]\n\t"
      "ds_read_b64_tr_b16 a[88:89], v21 offset:0\n\t"
      "v_mfma_f32_16x16x32_bf16 v[48:51], a[160:163], a[0:3], v[48:51]\n\t"
      "v_mfma_f32_16x16x32_bf16 v[48:51], a[164:167], a[4:7], v[48:51]\n\t"
      "ds_read_b64_tr_b16 a[90:91], v21 offset:2048\n\t"
      "v_mfma_f32_16x16x32_bf16 v[52:55], a[168:171], a[0:3], v[52:55]\n\t"
      "global_load_lds_dwordx4 v8, s[14:15] offset:2048 \n\t"
      "v_mfma_f32_16x16x32_bf16 v[52:55], a[172:175], a[4:7], v[52:55]\n\t"
      "v_mfma_f32_16x16x32_bf16 v[56:59], a[176:179], a[0:3], v[56:59]\n\t"
      "v_mfma_f32_16x16x32_bf16 v[56:59], a[180:183], a[4:7], v[56:59]\n\t"
      "ds_read_b64_tr_b16 a[96:97], v22 offset:0\n\t"
      "v_mfma_f32_16x16x32_bf16 v[60:63], a[184:187], a[0:3], v[60:63]\n\t"
      "v_mfma_f32_16x16x32_bf16 v[60:63], a[188:191], a[4:7], v[60:63]\n\t"
      "ds_read_b64_tr_b16 a[98:99], v22 offset:2048\n\t"
      "s_waitcnt vmcnt(21)\n\t"
      "v_mfma_f32_16x16x32_bf16 v[64:67], a[128:131], a[8:11], v[64:67]\n\t"
      "global_load_dwordx4 a[0:3], v1, s[36:37] offset:0 \n\t"
      "v_mfma_f32_16x16x32_bf16 v[64:67], a[132:135], a[12:15], v[64:67]\n\t"
      "v_mfma_f32_16x16x32_bf16 v[68:71], a[136:139], a[8:11], v[68:71]\n\t"
      "v_mfma_f32_16x16x32_bf16 v[68:71], a[140:143], a[12:15], v[68:71]\n\t"
      "ds_read_b64_tr_b16 a[104:105], v23 offset:0\n\t"
      "v_mfma_f32_16x16x32_bf16 v[72:75], a[144:147], a[8:11], v[72:75]\n\t"
      "global_load_dwordx4 a[4:7], v1, s[36:37] offset:1024 \n\t"
      "v_mfma_f32_16x16x32_bf16 v[72:75], a[148:151], a[12:15], v[72:75]\n\t"
      "ds_read_b64_tr_b16 a[106:107], v23 offset:2048\n\t"
      "v_mfma_f32_16x16x32_bf16 v[76:79], a[152:155], a[8:11], v[76:79]\n\t"
      "v_mfma_f32_16x16x32_bf16 v[76:79], a[156:159], a[12:15], v[76:79]\n\t"
      "v_mfma_f32_16x16x32_bf16 v[80:83], a[160:163], a[8:11], v[80:83]\n\t"
      "v_mfma_f32_16x16x32_bf16 v[80:83], a[164:167], a[12:15], v[80:83]\n\t"
      "ds_read_b64_tr_b16 a[112:113], v24 offset:0\n\t"
      "v_mfma_f32_16x16x32_bf16 v[84:87], a[168:171], a[8:11], v[84:87]\n\t"
      "global_load_lds_dwordx4 v9, s[14:15] offset:3072 \n\t"
      "v_mfma_f32_16x16x32_bf16 v[84:87], a[172:175], a[12:15], v[84:87]\n\t"
      "ds_read_b64_tr_b16 a[114:115], v24 offset:2048\n\t"
      "v_mfma_f32_16x16x32_bf16 v[88:91], a[176:179], a[8:11], v[88:91]\n\t"
      "s_add_u32 s49, s49, 0x200000\n\t"
      "v_mfma_f32_16x16x32_bf16 v[88:91], a[180:183], a[12:15], v[88:91]\n\t"
      "v_mfma_f32_16x16x32_bf16 v[92:95], a[184:187], a[8:11], v[92:95]\n\t"
      "s_and_b32 s49, s49, 0xfffffff\n\t"
      "v_mfma_f32_16x16x32_bf16 v[92:95], a[188:191], a[12:15], v[92:95]\n\t"
      "ds_read_b64_tr_b16 a[120:121], v25 offset:0\n\t"
      "s_waitcnt vmcnt(20)\n\t"
      "v_mfma_f32_16x16x32_bf16 v[96:99], a[128:131], a[16:19], v[96:99]\n\t"
      "global_load_dwordx4 a[8:11], v16, s[36:37] offset:0 \n\t"
      "v_mfma_f32_16x16x32_bf16 v[96:99], a[132:135], a[20:23], v[96:99]\n\t"
      "ds_read_b64_tr_b16 a[122:123], v25 offset:2048\n\t"
      "v_mfma_f32_16x16x32_bf16 v[100:103], a[136:139], a[16:19], v[100:103]\n\t"
      "v_mfma_f32_16x16x32_bf16 v[100:103], a[140:143], a[20:23], v[100:103]\n\t"
      "v_mfma_f32_16x16x32_bf16 v[104:107], a[144:147], a[16:19], v[104:107]\n\t"
      "global_load_dwordx4 a[12:15], v16, s[36:37] offset:1024 \n\t"
      "v_mfma_f32_16x16x32_bf16 v[104:107], a[148:151], a[20:23], v[104:107]\n\t"
      "v_mfma_f32_16x16x32_bf16 v[108:111], a[152:155], a[16:19], v[108:111]\n\t"
      "v_mfma_f32_16x16x32_bf16 v[108:111], a[156:159], a[20:23], v[108:111]\n\t"
      "v_mfma_f32_16x16x32_bf16 v[112:115], a[160:163], a[16:19], v[112:115]\n\t"
      "v_mfma_f32_16x16x32_bf16 v[112:115], a[164:167], a[20:23], v[112:115]\n\t"
      "v_mfma_f32_16x16x32_bf16 v[116:119], a[168:171], a[16:19], v[116:119]\n\t"
      "v_mfma_f32_16x16x32_bf16 v[116:119], a[172:175], a[20:23], v[116:119]\n\t"
      "v_mfma_f32_16x16x32_bf16 v[120:123], a[176:179], a[16:19], v[120:123]\n\t"
      "v_mfma_f32_16x16x32_bf16 v[120:123], a[180:183], a[20:23], v[120:123]\n\t"
      "v_mfma_f32_16x16x32_bf16 v[124:127], a[184:187], a[16:19], v[124:127]\n\t"
      "v_mfma_f32_16x16x32_bf16 v[124:127], a[188:191], a[20:23], v[124:127]\n\t"
      "s_waitcnt vmcnt(18)\n\t"
      "v_mfma_f32_16x16x32_bf16 a[192:195], a[128:131], a[24:27], a[192:195]\n\t"
      "global_load_dwordx4 a[16:19], v14, s[36:37] offset:0 \n\t"
      "v_mfma_f32_16x16x32_bf16 a[192:195], a[132:135], a[28:31], a[192:195]\n\t"
      "global_load_dwordx4 a[20:23], v14, s[36:37] offset:1024 \n\t"
      "v_mfma_f32_16x16x32_bf16 a[196:199], a[136:139], a[24:27], a[196:199]\n\t"
      "s_waitcnt vmcnt(30) lgkmcnt(0)\n\t"
      "v_mfma_f32_16x16x32_bf16 a[196:199], a[140:143], a[28:31], a[196:199]\n\t"
      "s_barrier\n\t"
      "v_mfma_f32_16x16x32_bf16 a[200:203], a[144:147], a[24:27], a[200:203]\n\t"
      "v_mfma_f32_16x16x32_bf16 a[200:203], a[148:151], a[28:31], a[200:203]\n\t"
      "ds_read_b64_tr_b16 a[68:69], v18 offset:16384\n\t"
      "v_mfma_f32_16x16x32_bf16 a[204:207], a[152:155], a[24:27], a[204:207]\n\t"
      "s_add_u32 m0, s99, 0\n\t"
      "s_add_u32 s14, s12, s49\n\t"
      "s_addc_u32 s15, s13, 0\n\t"
      "v_mfma_f32_16x16x32_bf16 a[204:207], a[156:159], a[28:31], a[204:207]\n\t"
      "ds_read_b64_tr_b16 a[70:71], v18 offset:18432\n\t"
      "v_mfma_f32_16x16x32_bf16 a[208:211], a[160:163], a[24:27], a[208:211]\n\t"
      "global_load_lds_dwordx4 v6, s[14:15] offset:0 \n\t"
      "v_mfma_f32_16x16x32_bf16 a[208:211], a[164:167], a[28:31], a[208:211]\n\t"
      "v_mfma_f32_16x16x32_bf16 a[212:215], a[168:171], a[24:27], a[212:215]\n\t"
      "v_mfma_f32_16x16x32_bf16 a[212:215], a[172:175], a[28:31], a[212:215]\n\t"
      "ds_read_b64_tr_b16 a[76:77], v19 offset:16384\n\t"
      "v_mfma_f32_16x16x32_bf16 a[216:219], a[176:179], a[24:27], a[216:219]\n\t"
      "global_load_lds_dwordx4 v7, s[14:15] offset:1024 \n\t"
      "v_mfma_f32_16x16x32_bf16 a[216:219], a[180:183], a[28:31], a[216:219]\n\t"
      "ds_read_b64_tr_b16 a[78:79], v19 offset:18432\n\t"
      "v_mfma_f32_16x16x32_bf16 a[220:223], a[184:187], a[24:27], a[220:223]\n\t"
      "v_mfma_f32_16x16x32_bf16 a[220:223], a[188:191], a[28:31], a[220:223]\n\t"
      "s_waitcnt vmcnt(19)\n\t"
      "v_mfma_f32_16x16x32_bf16 v[128:131], a[128:131], a[32:35], v[128:131]\n\t"
      "global_load_dwordx4 a[24:27], v31, s[36:37] offset:0 \n\t"
      "v_mfma_f32_16x16x32_bf16 v[128:131], a[132:135], a[36:39], v[128:131]\n\t"
      "ds_read_b64_tr_b16 a[84:85], v20 offset:16384\n\t"
      "v_mfma_f32_16x16x32_bf16 v[132:135], a[136:139], a[32:35], v[132:135]\n\t"
      "v_mfma_f32_16x16x32_bf16 v[132:135], a[140:143], a[36:39], v[132:135]\n\t"
      "ds_read_b64_tr_b16 a[86:87], v20 offset:18432\n\t"
      "v_mfma_f32_16x16x32_bf16 v[136:139], a[144:147], a[32:35], v[136:139]\n\t"
      "global_load_dwordx4 a[28:31], v31, s[36:37] offset:1024 \n\t"
      "v_mfma_f32_16x16x32_bf16 v[136:139], a[148:151], a[36:39], v[136:139]\n\t"
      "v_mfma_f32_16x16x32_bf16 v[140:143], a[152:155], a[32:35], v[140:143]\n\t"
      "s_add_u32 s36, s36, 16384\n\t"
      "s_addc_u32 s37, s37, 0\n\t"
      "v_mfma_f32_16x16x32_bf16 v[140:143], a[156:159], a[36:39], v[140:143]\n\t"
      "ds_read_b64_tr_b16 a[92:93], v21 offset:16384\n\t"
      "v_mfma_f32_16x16x32_bf16 v[144:147], a[160:163], a[32:35], v[144:147]\n\t"
      "v_mfma_f32_16x16x32_bf16 v[144:147], a[164:167], a[36:39], v[144:147]\n\t"
      "ds_read_b64_tr_b16 a[94:95], v21 offset:18432\n\t"
      "v_mfma_f32_16x16x32_bf16 v[148:151], a[168:171], a[32:35], v[148:151]\n\t"
      "global_load_lds_dwordx4 v8, s[14:15] offset:2048 \n\t"
      "v_mfma_f32_16x16x32_bf16 v[148:151], a[172:175], a[36:39], v[148:151]\n\t"
      "v_mfma_f32_16x16x32_bf16 v[152:155], a[176:179], a[32:35], v[152:155]\n\t"
      "v_mfma_f32_16x16x32_bf16 v[152:155], a[180:183], a[36:39], v[152:155]\n\t"
      "ds_read_b64_tr_b16 a[100:101], v22 offset:16384\n\t"
      "v_mfma_f32_16x16x32_bf16 v[156:159], a[184:187], a[32:35], v[156:159]\n\t"
      "v_mfma_f32_16x16x32_bf16 v[156:159], a[188:191], a[36:39], v[156:159]\n\t"
      "ds_read_b64_tr_b16 a[102:103], v22 offset:18432\n\t"
      "s_waitcnt vmcnt(19)\n\t"
      "v_mfma_f32_16x16x32_bf16 v[160:163], a[128:131], a[40:43], v[160:163]\n\t"
      "global_load_dwordx4 a[32:35], v1, s[36:37] offset:0 \n\t"
      "v_mfma_f32_16x16x32_bf16 v[160:163], a[132:135], a[44:47], v[160:163]\n\t"
      "v_mfma_f32_16x16x32_bf16 v[164:167], a[136:139], a[40:43], v[164:167]\n\t"
      "v_mfma_f32_16x16x32_bf16 v[164:167], a[140:143], a[44:47], v[164:167]\n\t"
      "ds_read_b64_tr_b16 a[108:109], v23 offset:16384\n\t"
      "v_mfma_f32_16x16x32_bf16 v[168:171], a[144:147], a[40:43], v[168:171]\n\t"
      "global_load_dwordx4 a[36:39], v1, s[36:37] offset:1024 \n\t"
      "v_mfma_f32_16x16x32_bf16 v[168:171], a[148:151], a[44:47], v[168:171]\n\t"
      "ds_read_b64_tr_b16 a[110:111], v23 offset:18432\n\t"
      "v_mfma_f32_16x16x32_bf16 v[172:175], a[152:155], a[40:43], v[172:175]\n\t"
      "v_mfma_f32_16x16x32_bf16 v[172:175], a[156:159], a[44:47], v[172:175]\n\t"
      "v_mfma_f32_16x16x32_bf16 v[176:179], a[160:163], a[40:43], v[176:179]\n\t"
      "v_mfma_f32_16x16x32_bf16 v[176:179], a[164:167], a[44:47], v[176:179]\n\t"
      "ds_read_b64_tr_b16 a[116:117], v24 offset:16384\n\t"
      "v_mfma_f32_16x16x32_bf16 v[180:183], a[168:171], a[40:43], v[180:183]\n\t"
      "global_load_lds_dwordx4 v9, s[14:15] offset:3072 \n\t"
      "v_mfma_f32_16x16x32_bf16 v[180:183], a[172:175], a[44:47], v[180:183]\n\t"
      "ds_read_b64_tr_b16 a[118:119], v24 offset:18432\n\t"
      "v_mfma_f32_16x16x32_bf16 v[184:187], a[176:179], a[40:43], v[184:187]\n\t"
      "s_add_u32 s49, s49, 0x200000\n\t"
      "v_mfma_f32_16x16x32_bf16 v[184:187], a[180:183], a[44:47], v[184:187]\n\t"
      "v_mfma_f32_16x16x32_bf16 v[188:191], a[184:187], a[40:43], v[188:191]\n\t"
      "s_and_b32 s49, s49, 0xfffffff\n\t"
      "v_mfma_f32_16x16x32_bf16 v[188:191], a[188:191], a[44:47], v[188:191]\n\t"
      "ds_read_b64_tr_b16 a[124:125], v25 offset:16384\n\t"
      "s_waitcnt vmcnt(20)\n\t"
      "v_mfma_f32_16x16x32_bf16 v[192:195], a[128:131], a[48:51], v[192:195]\n\t"
      "global_load_dwordx4 a[40:43], v16, s[36:37] offset:0 \n\t"
      "v_mfma_f32_16x16x32_bf16 v[192:195], a[132:135], a[52:55], v[192:195]\n\t"
      "ds_read_b64_tr_b16 a[126:127], v25 offset:18432\n\t"
      "v_mfma_f32_16x16x32_bf16 v[196:199], a[136:139], a[48:51], v[196:199]\n\t"
      "v_mfma_f32_16x16x32_bf16 v[196:199], a[140:143], a[52:55], v[196:199]\n\t"
      "v_mfma_f32_16x16x32_bf16 v[200:203], a[144:147], a[48:51], v[200:203]\n\t"
      "global_load_dwordx4 a[44:47], v16, s[36:37] offset:1024 \n\t"
      "v_mfma_f32_16x16x32_bf16 v[200:203], a[148:151], a[52:55], v[200:203]\n\t"
      "v_min_u32 v2, v2, v3\n\t"
      "v_mfma_f32_16x16x32_bf16 v[204:207], a[152:155], a[48:51], v[204:207]\n\t"
      "v_mfma_f32_16x16x32_bf16 v[204:207], a[156:159], a[52:55], v[204:207]\n\t"
      "v_cmp_gt_u32 vcc, s52, v2\n\t"
      "v_mfma_f32_16x16x32_bf16 v[208:211], a[160:163], a[48:51], v[208:211]\n\t"
      "v_mfma_f32_16x16x32_bf16 v[208:211], a[164:167], a[52:55], v[208:211]\n\t"
      "v_mfma_f32_16x16x32_bf16 v[212:215], a[168:171], a[48:51], v[212:215]\n\t"
      "v_mfma_f32_16x16x32_bf16 v[212:215], a[172:175], a[52:55], v[212:215]\n\t"
      "s_cbranch_vccnz SLOW3_%=\n\t"
      "BACK3_%=:\n\t"
      "v_mfma_f32_16x16x32_bf16 v[216:219], a[176:179], a[48:51], v[216:219]\n\t"
      "s_sub_u32 s50, s50, 1\n\t"
      "v_mfma_f32_16x16x32_bf16 v[216:219], a[180:183], a[52:55], v[216:219]\n\t"
      "s_cbranch_scc0 LOOP_%=\n\t"
      "v_mfma_f32_16x16x32_bf16 v[220:223], a[184:187], a[48:51], v[220:223]\n\t"
      "v_mfma_f32_16x16x32_bf16 v[220:223], a[188:191], a[52:55], v[220:223]\n\t"
      "s_waitcnt vmcnt(18)\n\t"
      "v_mfma_f32_16x16x32_bf16 v[224:227], a[128:131], a[56:59], v[224:227]\n\t"
      "global_load_dwordx4 a[48:51], v14, s[36:37] offset:0 \n\t"
      "v_mfma_f32_16x16x32_bf16 v[224:227], a[132:135], a[60:63], v[224:227]\n\t"
      "global_load_dwordx4 a[52:55], v14, s[36:37] offset:1024 \n\t"
      "v_mfma_f32_16x16x32_bf16 v[228:231], a[136:139], a[56:59], v[228:231]\n\t"
      "s_waitcnt vmcnt(28) lgkmcnt(0)\n\t"
      "v_mfma_f32_16x16x32_bf16 v[228:231], a[140:143], a[60:63], v[228:231]\n\t"
      "TAIL_%=:\n\t"
      "s_waitcnt vmcnt(16) lgkmcnt(0)\n\t"
      "s_barrier\n\t"
      "v_mfma_f32_16x16x32_bf16 v[232:235], a[144:147], a[56:59], v[232:235]\n\t"
      "v_mfma_f32_16x16x32_bf16 v[232:235], a[148:151], a[60:63], v[232:235]\n\t"
      "ds_read_b64_tr_b16 a[128:129], v18 offset:32768\n\t"
      "v_mfma_f32_16x16x32_bf16 v[236:239], a[152:155], a[56:59], v[236:239]\n\t"
      "v_mfma_f32_16x16x32_bf16 v[236:239], a[156:159], a[60:63], v[236:239]\n\t"
      "ds_read_b64_tr_b16 a[130:131], v18 offset:34816\n\t"
      "v_mfma_f32_16x16x32_bf16 v[240:243], a[160:163], a[56:59], v[240:243]\n\t"
      "v_mfma_f32_16x16x32_bf16 v[240:243], a[164:167], a[60:63], v[240:243]\n\t"
      "ds_read_b64_tr_b16 a[136:137], v19 offset:32768\n\t"
      "v_mfma_f32_16x16x32_bf16 v[244:247], a[168:171], a[56:59], v[244:247]\n\t"
      "s_add_u32 s16, s44, 0\n\t"
      "s_addc_u32 s17, s45, 0\n\t"
      "v_mfma_f32_16x16x32_bf16 v[244:247], a[172:175], a[60:63], v[244:247]\n\t"
      "ds_read_b64_tr_b16 a[138:139], v19 offset:34816\n\t"
      "v_mfma_f32_16x16x32_bf16 v[248:251], a[176:179], a[56:59], v[248:251]\n\t"
      "v_mfma_f32_16x16x32_bf16 v[248:251], a[180:183], a[60:63], v[248:251]\n\t"
      "ds_read_b64_tr_b16 a[144:145], v20 offset:32768\n\t"
      "s_add_u32 s18, s44, 131072\n\t"
      "s_addc_u32 s19, s45, 0\n\t"
      "v_mfma_f32_16x16x32_bf16 v[252:255], a[184:187], a[56:59], v[252:255]\n\t"
      "v_mfma_f32_16x16x32_bf16 v[252:255], a[188:191], a[60:63], v[252:255]\n\t"
      "ds_read_b64_tr_b16 a[146:147], v20 offset:34816\n\t"
      "s_waitcnt vmcnt(17)\n\t"
      "v_mfma_f32_16x16x32_bf16 v[32:35], a[64:67], a[0:3], v[32:35]\n\t"
      "global_load_dwordx4 a[56:59], v31, s[36:37] offset:0 \n\t"
      "v_mfma_f32_16x16x32_bf16 v[32:35], a[68:71], a[4:7], v[32:35]\n\t"
      "ds_read_b64_tr_b16 a[152:153], v21 offset:32768\n\t"
      "v_mfma_f32_16x16x32_bf16 v[36:39], a[72:75], a[0:3], v[36:39]\n\t"
      "v_mfma_f32_16x16x32_bf16 v[36:39], a[76:79], a[4:7], v[36:39]\n\t"
      "ds_read_b64_tr_b16 a[154:155], v21 offset:34816\n\t"
      "s_add_u32 s20, s44, 262144\n\t"
      "s_addc_u32 s21, s45, 0\n\t"
      "v_mfma_f32_16x16x32_bf16 v[40:43], a[80:83], a[0:3], v[40:43]\n\t"
      "global_load_dwordx4 a[60:63], v31, s[36:37] offset:1024 \n\t"
      "v_mfma_f32_16x16x32_bf16 v[40:43], a[84:87], a[4:7], v[40:43]\n\t"
      "ds_read_b64_tr_b16 a[160:161], v22 offset:32768\n\t"
      "v_mfma_f32_16x16x32_bf16 v[44:47], a[88:91], a[0:3], v[44:47]\n\t"
      "s_add_u32 s36, s36, 16384\n\t"
      "s_addc_u32 s37, s37, 0\n\t"
      "s_add_u32 s22, s44, 393216\n\t"
      "s_addc_u32 s23, s45, 0\n\t"
      "v_mfma_f32_16x16x32_bf16 v[44:47], a[92:95], a[4:7], v[44:47]\n\t"
      "ds_read_b64_tr_b16 a[162:163], v22 offset:34816\n\t"
      "v_mfma_f32_16x16x32_bf16 v[48:51], a[96:99], a[0:3], v[48:51]\n\t"
      "v_mfma_f32_16x16x32_bf16 v[48:51], a[100:103], a[4:7], v[48:51]\n\t"
      "ds_read_b64_tr_b16 a[168:169], v23 offset:32768\n\t"
      "s_add_u32 s24, s44, 524288\n\t"
      "s_addc_u32 s25, s45, 0\n\t"
      "v_mfma_f32_16x16x32_bf16 v[52:55], a[104:107], a[0:3], v[52:55]\n\t"
      "v_mfma_f32_16x16x32_bf16 v[52:55], a[108:111], a[4:7], v[52:55]\n\t"
      "ds_read_b64_tr_b16 a[170:171], v23 offset:34816\n\t"
      "v_mfma_f32_16x16x32_bf16 v[56:59], a[112:115], a[0:3], v[56:59]\n\t"
      "s_add_u32 s26, s44, 655360\n\t"
      "s_addc_u32 s27, s45, 0\n\t"
      "v_mfma_f32_16x16x32_bf16 v[56:59], a[116:119], a[4:7], v[56:59]\n\t"
      "ds_read_b64_tr_b16 a[176:177], v24 offset:32768\n\t"
      "v_mfma_f32_16x16x32_bf16 v[60:63], a[120:123], a[0:3], v[60:63]\n\t"
      "v_mfma_f32_16x16x32_bf16 v[60:63], a[124:127], a[4:7], v[60:63]\n\t"
      "ds_read_b64_tr_b16 a[178:179], v24 offset:34816\n\t"
      "s_waitcnt vmcnt(16)\n\t"
      "v_mfma_f32_16x16x32_bf16 v[64:67], a[64:67], a[8:11], v[64:67]\n\t"
      "global_load_dwordx4 a[0:3], v1, s[36:37] offset:0 \n\t"
      "v_mfma_f32_16x16x32_bf16 v[64:67], a[68:71], a[12:15], v[64:67]\n\t"
      "ds_read_b64_tr_b16 a[184:185], v25 offset:32768\n\t"
      "v_mfma_f32_16x16x32_bf16 v[68:71], a[72:75], a[8:11], v[68:71]\n\t"
      "s_add_u32 s28, s44, 786432\n\t"
      "s_addc_u32 s29, s45, 0\n\t"
      "v_mfma_f32_16x16x32_bf16 v[68:71], a[76:79], a[12:15], v[68:71]\n\t"
      "ds_read_b64_tr_b16 a[186:187], v25 offset:34816\n\t"
      "v_mfma_f32_16x16x32_bf16 v[72:75], a[80:83], a[8:11], v[72:75]\n\t"
      "global_load_dwordx4 a[4:7], v1, s[36:37] offset:1024 \n\t"
      "v_mfma_f32_16x16x32_bf16 v[72:75], a[84:87], a[12:15], v[72:75]\n\t"
      "ds_read_b64_tr_b16 a[132:133], v18 offset:49152\n\t"
      "s_add_u32 s30, s44, 917504\n\t"
      "s_addc_u32 s31, s45, 0\n\t"
      "v_mfma_f32_16x16x32_bf16 v[76:79], a[88:91], a[8:11], v[76:79]\n\t"
      "v_mfma_f32_16x16x32_bf16 v[76:79], a[92:95], a[12:15], v[76:79]\n\t"
      "ds_read_b64_tr_b16 a[134:135], v18 offset:51200\n\t"
      "v_mfma_f32_16x16x32_bf16 v[80:83], a[96:99], a[8:11], v[80:83]\n\t"
      "v_mfma_f32_16x16x32_bf16 v[80:83], a[100:103], a[12:15], v[80:83]\n\t"
      "ds_read_b64_tr_b16 a[140:141], v19 offset:49152\n\t"
      "v_mfma_f32_16x16x32_bf16 v[84:87], a[104:107], a[8:11], v[84:87]\n\t"
      "v_mfma_f32_16x16x32_bf16 v[84:87], a[108:111], a[12:15], v[84:87]\n\t"
      "ds_read_b64_tr_b16 a[142:143], v19 offset:51200\n\t"
      "v_mfma_f32_16x16x32_bf16 v[88:91], a[112:115], a[8:11], v[88:91]\n\t"
      "global_store_dword v27, v28, s[42:43] sc0 sc1\n\t"
      "v_mfma_f32_16x16x32_bf16 v[88:91], a[116:119], a[12:15], v[88:91]\n\t"
      "ds_read_b64_tr_b16 a[148:149], v20 offset:49152\n\t"
      "v_mfma_f32_16x16x32_bf16 v[92:95], a[120:123], a[8:11], v[92:95]\n\t"
      "v_mfma_f32_16x16x32_bf16 v[92:95], a[124:127], a[12:15], v[92:95]\n\t"
      "ds_read_b64_tr_b16 a[150:151], v20 offset:51200\n\t"
      "s_waitcnt vmcnt(17)\n\t"
      "v_mfma_f32_16x16x32_bf16 v[96:99], a[64:67], a[16:19], v[96:99]\n\t"
      "global_load_dwordx4 a[8:11], v16, s[36:37] offset:0 \n\t"
      "v_readfirstlane_b32 s53, v29\n\t"
      "v_mfma_f32_16x16x32_bf16 v[96:99], a[68:71], a[20:23], v[96:99]\n\t"
      "ds_read_b64_tr_b16 a[156:157], v21 offset:49152\n\t"
      "v_mfma_f32_16x16x32_bf16 v[100:103], a[72:75], a[16:19], v[100:103]\n\t"
      "v_mfma_f32_16x16x32_bf16 v[100:103], a[76:79], a[20:23], v[100:103]\n\t"
      "ds_read_b64_tr_b16 a[158:159], v21 offset:51200\n\t"
      "v_mfma_f32_16x16x32_bf16 v[104:107], a[80:83], a[16:19], v[104:107]\n\t"
      "global_load_dwordx4 a[12:15], v16, s[36:37] offset:1024 \n\t"
      "v_mfma_f32_16x16x32_bf16 v[104:107], a[84:87], a[20:23], v[104:107]\n\t"
      "ds_read_b64_tr_b16 a[164:165], v22 offset:49152\n\t"
      "v_mfma_f32_16x16x32_bf16 v[108:111], a[88:91], a[16:19], v[108:111]\n\t"
      "s_cmp_eq_u32 s53, s52\n\t"
      "v_mfma_f32_16x16x32_bf16 v[108:111], a[92:95], a[20:23], v[108:111]\n\t"
      "ds_read_b64_tr_b16 a[166:167], v22 offset:51200\n\t"
      "s_cselect_b32 s59, 0x7fc00000, 1.0\n\t"
      "v_mfma_f32_16x16x32_bf16 v[112:115], a[96:99], a[16:19], v[112:115]\n\t"
      "v_mfma_f32_16x16x32_bf16 v[112:115], a[100:103], a[20:23], v[112:115]\n\t"
      "ds_read_b64_tr_b16 a[172:173], v23 offset:49152\n\t"
      "s_mov_b32 s58, 0x6abcedc7\n\t"
      "v_mfma_f32_16x16x32_bf16 v[116:119], a[104:107], a[16:19], v[116:119]\n\t"
      "v_mfma_f32_16x16x32_bf16 v[116:119], a[108:111], a[20:23], v[116:119]\n\t"
      "ds_read_b64_tr_b16 a[174:175], v23 offset:51200\n\t"
      "v_mfma_f32_16x16x32_bf16 v[120:123], a[112:115], a[16:19], v[120:123]\n\t"
      "ds_read_b64_tr_b16 a[180:181], v24 offset:49152\n\t"
      "s_mov_b32 s54, 0xbfb8aa3b\n\t"
      "v_mfma_f32_16x16x32_bf16 v[120:123], a[116:119], a[20:23], v[120:123]\n\t"
      "ds_read_b64_tr_b16 a[182:183], v24 offset:51200\n\t"
      "v_mfma_f32_16x16x32_bf16 v[124:127], a[120:123], a[16:19], v[124:127]\n\t"
      "ds_read_b64_tr_b16 a[188:189], v25 offset:49152\n\t"
      "v_mfma_f32_16x16x32_bf16 v[124:127], a[124:127], a[20:23], v[124:127]\n\t"
      "ds_read_b64_tr_b16 a[190:191], v25 offset:51200\n\t"
      "s_mov_b32 s55, 0xc2ad1f97\n\t"
      "s_waitcnt vmcnt(15)\n\t"
      "v_mfma_f32_16x16x32_bf16 a[192:195], a[64:67], a[24:27], a[192:195]\n\t"
      "global_load_dwordx4 a[16:19], v14, s[36:37] offset:0 \n\t"
      "v_mfma_f32_16x16x32_bf16 a[192:195], a[68:71], a[28:31], a[192:195]\n\t"
      "global_load_dwordx4 a[20:23], v14, s[36:37] offset:1024 \n\t"
      "v_mfma_f32_16x16x32_bf16 a[196:199], a[72:75], a[24:27], a[196:199]\n\t"
      "s_mov_b32 s57, 0\n\t"
      "v_mfma_f32_16x16x32_bf16 a[196:199], a[76:79], a[28:31], a[196:199]\n\t"
      "v_mov_b32 v30, s55\n\t"
      "s_waitcnt lgkmcnt(0)\n\t"
      "v_mfma_f32_16x16x32_bf16 a[200:203], a[80:83], a[24:27], a[200:203]\n\t"
      "v_mfma_f32_16x16x32_bf16 a[200:203], a[84:87], a[28:31], a[200:203]\n\t"
      "v_mfma_f32_16x16x32_bf16 a[204:207], a[88:91], a[24:27], a[204:207]\n\t"
      "v_mfma_f32_16x16x32_bf16 a[204:207], a[92:95], a[28:31], a[204:207]\n\t"
      "v_mfma_f32_16x16x32_bf16 a[208:211], a[96:99], a[24:27], a[208:211]\n\t"
      "v_mfma_f32_16x16x32_bf16 a[208:211], a[100:103], a[28:31], a[208:211]\n\t"
      "v_mfma_f32_16x16x32_bf16 a[212:215], a[104:107], a[24:27], a[212:215]\n\t"
      "v_mfma_f32_16x16x32_bf16 a[212:215], a[108:111], a[28:31], a[212:215]\n\t"
      "v_mfma_f32_16x16x32_bf16 a[216:219], a[112:115], a[24:27], a[216:219]\n\t"
      "v_mfma_f32_16x16x32_bf16 a[216:219], a[116:119], a[28:31], a[216:219]\n\t"
      "v_mfma_f32_16x16x32_bf16 a[220:223], a[120:123], a[24:27], a[220:223]\n\t"
      "v_mfma_f32_16x16x32_bf16 a[220:223], a[124:127], a[28:31], a[220:223]\n\t"
      "s_waitcnt vmcnt(14)\n\t"
      "v_mfma_f32_16x16x32_bf16 v[32:35], a[128:131], a[32:35], v[32:35]\n\t"
      "global_load_dwordx4 a[24:27], v31, s[36:37] offset:0 \n\t"
      "v_mfma_f32_16x16x32_bf16 v[32:35], a[132:135], a[36:39], v[32:35]\n\t"
      "v_mfma_f32_16x16x32_bf16 v[36:39], a[136:139], a[32:35], v[36:39]\n\t"
      "v_mfma_f32_16x16x32_bf16 v[36:39], a[140:143], a[36:39], v[36:39]\n\t"
      "v_mfma_f32_16x16x32_bf16 v[40:43], a[144:147], a[32:35], v[40:43]\n\t"
      "global_load_dwordx4 a[28:31], v31, s[36:37] offset:1024 \n\t"
      "v_mfma_f32_16x16x32_bf16 v[40:43], a[148:151], a[36:39], v[40:43]\n\t"
      "v_mfma_f32_16x16x32_bf16 v[44:47], a[152:155], a[32:35], v[44:47]\n\t"
      "s_add_u32 s36, s36, 16384\n\t"
      "s_addc_u32 s37, s37, 0\n\t"
      "v_mfma_f32_16x16x32_bf16 v[44:47], a[156:159], a[36:39], v[44:47]\n\t"
      "v_mfma_f32_16x16x32_bf16 v[48:51], a[160:163], a[32:35], v[48:51]\n\t"
      "v_fma_f32 v18, v32, s54, v30\n\t"
      "v_fma_f32 v19, v33, s54, v30\n\t"
      "v_mfma_f32_16x16x32_bf16 v[48:51], a[164:167], a[36:39], v[48:51]\n\t"
      "v_fma_f32 v20, v34, s54, v30\n\t"
      "v_fma_f32 v21, v35, s54, v30\n\t"
      "v_mfma_f32_16x16x32_bf16 v[52:55], a[168:171], a[32:35], v[52:55]\n\t"
      "v_fma_f32 v22, v36, s54, v30\n\t"
      "v_fma_f32 v23, v37, s54, v30\n\t"
      "v_mfma_f32_16x16x32_bf16 v[52:55], a[172:175], a[36:39], v[52:55]\n\t"
      "v_fma_f32 v24, v38, s54, v30\n\t"
      "v_fma_f32 v25, v39, s54, v30\n\t"
      "v_mfma_f32_16x16x32_bf16 v[56:59], a[176:179], a[32:35], v[56:59]\n\t"
      "v_exp_f32 v18, v18 clamp\n\t"
      "v_mfma_f32_16x16x32_bf16 v[56:59], a[180:183], a[36:39], v[56:59]\n\t"
      "v_exp_f32 v19, v19 clamp\n\t"
      "v_mfma_f32_16x16x32_bf16 v[60:63], a[184:187], a[32:35], v[60:63]\n\t"
      "v_exp_f32 v20, v20 clamp\n\t"
      "v_mfma_f32_16x16x32_bf16 v[60:63], a[188:191], a[36:39], v[60:63]\n\t"
      "v_exp_f32 v21, v21 clamp\n\t"
      "s_waitcnt vmcnt(13)\n\t"
      "v_mfma_f32_16x16x32_bf16 v[64:67], a[128:131], a[40:43], v[64:67]\n\t"
      "global_load_dwordx4 a[32:35], v1, s[36:37] offset:0 \n\t"
      "v_exp_f32 v22, v22 clamp\n\t"
      "v_mfma_f32_16x16x32_bf16 v[64:67], a[132:135], a[44:47], v[64:67]\n\t"
      "v_exp_f32 v23, v23 clamp\n\t"
      "v_exp_f32 v24, v24 clamp\n\t"
      "v_exp_f32 v25, v25 clamp\n\t"
      "v_pk_fma_f32 v[18:19], v[18:19], s[58:59], s[58:59] op_sel:[0,0,1] op_sel_hi:[1,0,1]\n\t"
      "v_pk_fma_f32 v[20:21], v[20:21], s[58:59], s[58:59] op_sel:[0,0,1] op_sel_hi:[1,0,1]\n\t"
      "v_pk_fma_f32 v[22:23], v[22:23], s[58:59], s[58:59] op_sel:[0,0,1] op_sel_hi:[1,0,1]\n\t"
      "v_pk_fma_f32 v[24:25], v[24:25], s[58:59], s[58:59] op_sel:[0,0,1] op_sel_hi:[1,0,1]\n\t"
      "v_rcp_f32 v18, v18\n\t"
      "v_rcp_f32 v19, v19\n\t"
      "v_rcp_f32 v20, v20\n\t"
      "v_pk_mul_f32 v[32:33], v[32:33], v[18:19]\n\t"
      "v_rcp_f32 v21, v21\n\t"
      "v_pk_fma_f32 v[2:3], v[40:41], s[54:55], s[54:55] op_sel:[0,0,1] op_sel_hi:[1,0,1]\n\t"
      "v_rcp_f32 v22, v22\n\t"
      "v_pk_mul_f32 v[34:35], v[34:35], v[20:21]\n\t"
      "v_rcp_f32 v23, v23\n\t"
      "v_pk_fma_f32 v[4:5], v[42:43], s[54:55], s[54:55] op_sel:[0,0,1] op_sel_hi:[1,0,1]\n\t"
      "v_rcp_f32 v24, v24\n\t"
      "v_pk_mul_f32 v[36:37], v[36:37], v[22:23]\n\t"
      "v_pk_fma_f32 v[6:7], v[44:45], s[54:55], s[54:55] op_sel:[0,0,1] op_sel_hi:[1,0,1]\n\t"
      "v_rcp_f32 v25, v25\n\t"
      "v_pk_fma_f32 v[8:9], v[46:47], s[54:55], s[54:55] op_sel:[0,0,1] op_sel_hi:[1,0,1]\n\t"
      "v_pk_mul_f32 v[38:39], v[38:39], v[24:25]\n\t"
      "v_mfma_f32_16x16x32_bf16 v[68:71], a[136:139], a[40:43], v[68:71]\n\t"
      "v_exp_f32 v2, v2 clamp\n\t"
      "v_mfma_f32_16x16x32_bf16 v[68:71], a[140:143], a[44:47], v[68:71]\n\t"
      "v_exp_f32 v3, v3 clamp\n\t"
      "v_mfma_f32_16x16x32_bf16 v[72:75], a[144:147], a[40:43], v[72:75]\n\t"
      "global_load_dwordx4 a[36:39], v1, s[36:37] offset:1024 \n\t"
      "v_exp_f32 v4, v4 clamp\n\t"
      "v_mfma_f32_16x16x32_bf16 v[72:75], a[148:151], a[44:47], v[72:75]\n\t"
      "v_exp_f32 v5, v5 clamp\n\t"
      "v_mfma_f32_16x16x32_bf16 v[76:79], a[152:155], a[40:43], v[76:79]\n\t"
      "v_exp_f32 v6, v6 clamp\n\t"
      "v_mfma_f32_16x16x32_bf16 v[76:79], a[156:159], a[44:47], v[76:79]\n\t"
      "v_cvt_pk_bf16_f32 v32, v32, v33\n\t"
      "v_cvt_pk_bf16_f32 v33, v34, v35\n\t"
      "v_mfma_f32_16x16x32_bf16 v[80:83], a[160:163], a[40:43], v[80:83]\n\t"
      "v_exp_f32 v7, v7 clamp\n\t"
      "v_exp_f32 v8, v8 clamp\n\t"
      "v_exp_f32 v9, v9 clamp\n\t"
      "v_pk_fma_f32 v[2:3], v[2:3], s[58:59], s[58:59] op_sel:[0,0,1] op_sel_hi:[1,0,1]\n\t"
      "v_cvt_pk_bf16_f32 v34, v36, v37\n\t"
      "v_pk_fma_f32 v[4:5], v[4:5], s[58:59], s[58:59] op_sel:[0,0,1] op_sel_hi:[1,0,1]\n\t"
      "v_cvt_pk_bf16_f32 v35, v38, v39\n\t"
      "v_pk_fma_f32 v[6:7], v[6:7], s[58:59], s[58:59] op_sel:[0,0,1] op_sel_hi:[1,0,1]\n\t"
      "v_pk_fma_f32 v[8:9], v[8:9], s[58:59], s[58:59] op_sel:[0,0,1] op_sel_hi:[1,0,1]\n\t"
      "v_rcp_f32 v2, v2\n\t"
      "v_permlane16_swap_b32 v32, v34\n\t"
      "v_rcp_f32 v3, v3\n\t"
      "v_permlane16_swap_b32 v33, v35\n\t"
      "v_rcp_f32 v4, v4\n\t"
      "v_pk_mul_f32 v[40:41], v[40:41], v[2:3]\n\t"
      "v_rcp_f32 v5, v5\n\t"
      "v_pk_fma_f32 v[18:19], v[48:49], s[54:55], s[54:55] op_sel:[0,0,1] op_sel_hi:[1,0,1]\n\t"
      "v_rcp_f32 v6, v6\n\t"
      "v_pk_mul_f32 v[42:43], v[42:43], v[4:5]\n\t"
      "v_rcp_f32 v7, v7\n\t"
      "v_pk_fma_f32 v[20:21], v[50:51], s[54:55], s[54:55] op_sel:[0,0,1] op_sel_hi:[1,0,1]\n\t"
      "v_rcp_f32 v8, v8\n\t"
      "v_pk_mul_f32 v[44:45], v[44:45], v[6:7]\n\t"
      "v_pk_fma_f32 v[22:23], v[52:53], s[54:55], s[54:55] op_sel:[0,0,1] op_sel_hi:[1,0,1]\n\t"
      "v_rcp_f32 v9, v9\n\t"
      "v_pk_fma_f32 v[24:25], v[54:55], s[54:55], s[54:55] op_sel:[0,0,1] op_sel_hi:[1,0,1]\n\t"
      "v_pk_mul_f32 v[46:47], v[46:47], v[8:9]\n\t"
      "v_mfma_f32_16x16x32_bf16 v[80:83], a[164:167], a[44:47], v[80:83]\n\t"
      "v_exp_f32 v18, v18 clamp\n\t"
      "v_mfma_f32_16x16x32_bf16 v[84:87], a[168:171], a[40:43], v[84:87]\n\t"
      "v_exp_f32 v19, v19 clamp\n\t"
      "v_mfma_f32_16x16x32_bf16 v[84:87], a[172:175], a[44:47], v[84:87]\n\t"
      "v_exp_f32 v20, v20 clamp\n\t"
      "v_mfma_f32_16x16x32_bf16 v[88:91], a[176:179], a[40:43], v[88:91]\n\t"
      "v_exp_f32 v21, v21 clamp\n\t"
      "v_mfma_f32_16x16x32_bf16 v[88:91], a[180:183], a[44:47], v[88:91]\n\t"
      "v_exp_f32 v22, v22 clamp\n\t"
      "v_mfma_f32_16x16x32_bf16 v[92:95], a[184:187], a[40:43], v[92:95]\n\t"
      "v_cvt_pk_bf16_f32 v40, v40, v41\n\t"
      "v_cvt_pk_bf16_f32 v41, v42, v43\n\t"
      "v_mfma_f32_16x16x32_bf16 v[92:95], a[188:191], a[44:47], v[92:95]\n\t"
      "v_cvt_pk_bf16_f32 v42, v44, v45\n\t"
      "v_cvt_pk_bf16_f32 v43, v46, v47\n\t"
      "s_waitcnt vmcnt(13)\n\t"
      "v_mfma_f32_16x16x32_bf16 v[96:99], a[128:131], a[48:51], v[96:99]\n\t"
      "global_load_dwordx4 a[40:43], v16, s[36:37] offset:0 \n\t"
      "global_store_dwordx4 v26, v[32:35], s[16:17] offset:0 sc0 sc1\n\t"
      "v_exp_f32 v23, v23 clamp\n\t"
      "v_exp_f32 v24, v24 clamp\n\t"
      "v_exp_f32 v25, v25 clamp\n\t"
      "v_pk_fma_f32 v[18:19], v[18:19], s[58:59], s[58:59] op_sel:[0,0,1] op_sel_hi:[1,0,1]\n\t"
      "v_pk_fma_f32 v[20:21], v[20:21], s[58:59], s[58:59] op_sel:[0,0,1] op_sel_hi:[1,0,1]\n\t"
      "v_pk_fma_f32 v[22:23], v[22:23], s[58:59], s[58:59] op_sel:[0,0,1] op_sel_hi:[1,0,1]\n\t"
      "v_pk_fma_f32 v[24:25], v[24:25], s[58:59], s[58:59] op_sel:[0,0,1] op_sel_hi:[1,0,1]\n\t"
      "v_rcp_f32 v18, v18\n\t"
      "v_permlane16_swap_b32 v40, v42\n\t"
      "v_rcp_f32 v19, v19\n\t"
      "v_permlane16_swap_b32 v41, v43\n\t"
      "v_rcp_f32 v20, v20\n\t"
      "v_pk_mul_f32 v[48:49], v[48:49], v[18:19]\n\t"
      "v_rcp_f32 v21, v21\n\t"
      "v_pk_fma_f32 v[2:3], v[56:57], s[54:55], s[54:55] op_sel:[0,0,1] op_sel_hi:[1,0,1]\n\t"
      "v_rcp_f32 v22, v22\n\t"
      "v_pk_mul_f32 v[50:51], v[50:51], v[20:21]\n\t"
      "v_rcp_f32 v23, v23\n\t"
      "v_pk_fma_f32 v[4:5], v[58:59], s[54:55], s[54:55] op_sel:[0,0,1] op_sel_hi:[1,0,1]\n\t"
      "v_rcp_f32 v24, v24\n\t"
      "v_pk_mul_f32 v[52:53], v[52:53], v[22:23]\n\t"
      "v_pk_fma_f32 v[6:7], v[60:61], s[54:55], s[54:55] op_sel:[0,0,1] op_sel_hi:[1,0,1]\n\t"
      "v_rcp_f32 v25, v25\n\t"
      "v_pk_fma_f32 v[8:9], v[62:63], s[54:55], s[54:55] op_sel:[0,0,1] op_sel_hi:[1,0,1]\n\t"
      "v_pk_mul_f32 v[54:55], v[54:55], v[24:25]\n\t"
      "v_mfma_f32_16x16x32_bf16 v[96:99], a[132:135], a[52:55], v[96:99]\n\t"
      "v_exp_f32 v2, v2 clamp\n\t"
      "v_mfma_f32_16x16x32_bf16 v[100:103], a[136:139], a[48:51], v[100:103]\n\t"
      "v_exp_f32 v3, v3 clamp\n\t"
      "v_mfma_f32_16x16x32_bf16 v[100:103], a[140:143], a[52:55], v[100:103]\n\t"
      "v_exp_f32 v4, v4 clamp\n\t"
      "v_mfma_f32_16x16x32_bf16 v[104:107], a[144:147], a[48:51], v[104:107]\n\t"
      "global_load_dwordx4 a[44:47], v16, s[36:37] offset:1024 \n\t"
      "v_exp_f32 v5, v5 clamp\n\t"
      "v_mfma_f32_16x16x32_bf16 v[104:107], a[148:151], a[52:55], v[104:107]\n\t"
      "v_exp_f32 v6, v6 clamp\n\t"
      "v_mfma_f32_16x16x32_bf16 v[108:111], a[152:155], a[48:51], v[108:111]\n\t"
      "v_cvt_pk_bf16_f32 v48, v48, v49\n\t"
      "v_cvt_pk_bf16_f32 v49, v50, v51\n\t"
      "v_mfma_f32_16x16x32_bf16 v[108:111], a[156:159], a[52:55], v[108:111]\n\t"
      "global_store_dwordx4 v26, v[40:43], s[16:17] offset:64 sc0 sc1\n\t"
      "v_exp_f32 v7, v7 clamp\n\t"
      "v_exp_f32 v8, v8 clamp\n\t"
      "v_exp_f32 v9, v9 clamp\n\t"
      "v_pk_fma_f32 v[2:3], v[2:3], s[58:59], s[58:59] op_sel:[0,0,1] op_sel_hi:[1,0,1]\n\t"
      "v_cvt_pk_bf16_f32 v50, v52, v53\n\t"
      "v_pk_fma_f32 v[4:5], v[4:5], s[58:59], s[58:59] op_sel:[0,0,1] op_sel_hi:[1,0,1]\n\t"
      "v_cvt_pk_bf16_f32 v51, v54, v55\n\t"
      "v_pk_fma_f32 v[6:7], v[6:7], s[58:59], s[58:59] op_sel:[0,0,1] op_sel_hi:[1,0,1]\n\t"
      "v_pk_fma_f32 v[8:9], v[8:9], s[58:59], s[58:59] op_sel:[0,0,1] op_sel_hi:[1,0,1]\n\t"
      "v_rcp_f32 v2, v2\n\t"
      "v_permlane16_swap_b32 v48, v50\n\t"
      "v_rcp_f32 v3, v3\n\t"
      "v_permlane16_swap_b32 v49, v51\n\t"
      "v_rcp_f32 v4, v4\n\t"
      "v_pk_mul_f32 v[56:57], v[56:57], v[2:3]\n\t"
      "v_rcp_f32 v5, v5\n\t"
      "v_pk_fma_f32 v[18:19], v[64:65], s[54:55], s[54:55] op_sel:[0,0,1] op_sel_hi:[1,0,1]\n\t"
      "v_rcp_f32 v6, v6\n\t"
      "v_pk_mul_f32 v[58:59], v[58:59], v[4:5]\n\t"
      "v_rcp_f32 v7, v7\n\t"
      "v_pk_fma_f32 v[20:21], v[66:67], s[54:55], s[54:55] op_sel:[0,0,1] op_sel_hi:[1,0,1]\n\t"
      "v_rcp_f32 v8, v8\n\t"
      "v_pk_mul_f32 v[60:61], v[60:61], v[6:7]\n\t"
      "v_pk_fma_f32 v[22:23], v[68:69], s[54:55], s[54:55] op_sel:[0,0,1] op_sel_hi:[1,0,1]\n\t"
      "v_rcp_f32 v9, v9\n\t"
      "v_pk_fma_f32 v[24:25], v[70:71], s[54:55], s[54:55] op_sel:[0,0,1] op_sel_hi:[1,0,1]\n\t"
      "v_pk_mul_f32 v[62:63], v[62:63], v[8:9]\n\t"
      "v_mfma_f32_16x16x32_bf16 v[112:115], a[160:163], a[48:51], v[112:115]\n\t"
      "v_exp_f32 v18, v18 clamp\n\t"
      "v_mfma_f32_16x16x32_bf16 v[112:115], a[164:167], a[52:55], v[112:115]\n\t"
      "v_exp_f32 v19, v19 clamp\n\t"
      "v_mfma_f32_16x16x32_bf16 v[116:119], a[168:171], a[48:51], v[116:119]\n\t"
      "v_exp_f32 v20, v20 clamp\n\t"
      "v_mfma_f32_16x16x32_bf16 v[116:119], a[172:175], a[52:55], v[116:119]\n\t"
      "v_exp_f32 v21, v21 clamp\n\t"
      "v_mfma_f32_16x16x32_bf16 v[120:123], a[176:179], a[48:51], v[120:123]\n\t"
      "v_exp_f32 v22, v22 clamp\n\t"
      "v_mfma_f32_16x16x32_bf16 v[120:123], a[180:183], a[52:55], v[120:123]\n\t"
      "v_cvt_pk_bf16_f32 v56, v56, v57\n\t"
      "v_cvt_pk_bf16_f32 v57, v58, v59\n\t"
      "v_mfma_f32_16x16x32_bf16 v[124:127], a[184:187], a[48:51], v[124:127]\n\t"
      "v_cvt_pk_bf16_f32 v58, v60, v61\n\t"
      "v_cvt_pk_bf16_f32 v59, v62, v63\n\t"
      "v_mfma_f32_16x16x32_bf16 v[124:127], a[188:191], a[52:55], v[124:127]\n\t"
      "global_store_dwordx4 v26, v[48:51], s[16:17] offset:128 sc0 sc1\n\t"
      "v_exp_f32 v23, v23 clamp\n\t"
      "v_exp_f32 v24, v24 clamp\n\t"
      "v_exp_f32 v25, v25 clamp\n\t"
      "v_pk_fma_f32 v[18:19], v[18:19], s[58:59], s[58:59] op_sel:[0,0,1] op_sel_hi:[1,0,1]\n\t"
      "v_pk_fma_f32 v[20:21], v[20:21], s[58:59], s[58:59] op_sel:[0,0,1] op_sel_hi:[1,0,1]\n\t"
      "v_pk_fma_f32 v[22:23], v[22:23], s[58:59], s[58:59] op_sel:[0,0,1] op_sel_hi:[1,0,1]\n\t"
      "v_pk_fma_f32 v[24:25], v[24:25], s[58:59], s[58:59] op_sel:[0,0,1] op_sel_hi:[1,0,1]\n\t"
      "v_rcp_f32 v18, v18\n\t"
      "v_permlane16_swap_b32 v56, v58\n\t"
      "v_rcp_f32 v19, v19\n\t"
      "v_permlane16_swap_b32 v57, v59\n\t"
      "v_rcp_f32 v20, v20\n\t"
      "v_pk_mul_f32 v[64:65], v[64:65], v[18:19]\n\t"
      "v_rcp_f32 v21, v21\n\t"
      "v_pk_fma_f32 v[2:3], v[72:73], s[54:55], s[54:55] op_sel:[0,0,1] op_sel_hi:[1,0,1]\n\t"
      "v_rcp_f32 v22, v22\n\t"
      "v_pk_mul_f32 v[66:67], v[66:67], v[20:21]\n\t"
      "v_rcp_f32 v23, v23\n\t"
      "v_pk_fma_f32 v[4:5], v[74:75], s[54:55], s[54:55] op_sel:[0,0,1] op_sel_hi:[1,0,1]\n\t"
      "v_rcp_f32 v24, v24\n\t"
      "v_pk_mul_f32 v[68:69], v[68:69], v[22:23]\n\t"
      "v_pk_fma_f32 v[6:7], v[76:77], s[54:55], s[54:55] op_sel:[0,0,1] op_sel_hi:[1,0,1]\n\t"
      "v_rcp_f32 v25, v25\n\t"
      "v_pk_fma_f32 v[8:9], v[78:79], s[54:55], s[54:55] op_sel:[0,0,1] op_sel_hi:[1,0,1]\n\t"
      "v_pk_mul_f32 v[70:71], v[70:71], v[24:25]\n\t"
      "s_waitcnt vmcnt(16)\n\t"
      "v_mfma_f32_16x16x32_bf16 a[192:195], a[128:131], a[56:59], a[192:195]\n\t"
      "global_load_dwordx4 a[48:51], v14, s[36:37] offset:0 \n\t"
      "v_exp_f32 v2, v2 clamp\n\t"
      "v_mfma_f32_16x16x32_bf16 a[192:195], a[132:135], a[60:63], a[192:195]\n\t"
      "global_load_dwordx4 a[52:55], v14, s[36:37] offset:1024 \n\t"
      "v_exp_f32 v3, v3 clamp\n\t"
      "v_mfma_f32_16x16x32_bf16 a[196:199], a[136:139], a[56:59], a[196:199]\n\t"
      "v_exp_f32 v4, v4 clamp\n\t"
      "v_mfma_f32_16x16x32_bf16 a[196:199], a[140:143], a[60:63], a[196:199]\n\t"
      "v_exp_f32 v5, v5 clamp\n\t"
      "s_waitcnt lgkmcnt(0)\n\t"
      "v_mfma_f32_16x16x32_bf16 a[200:203], a[144:147], a[56:59], a[200:203]\n\t"
      "v_exp_f32 v6, v6 clamp\n\t"
      "v_mfma_f32_16x16x32_bf16 a[200:203], a[148:151], a[60:63], a[200:203]\n\t"
      "v_cvt_pk_bf16_f32 v64, v64, v65\n\t"
      "v_cvt_pk_bf16_f32 v65, v66, v67\n\t"
      "v_mfma_f32_16x16x32_bf16 a[204:207], a[152:155], a[56:59], a[204:207]\n\t"
      "v_cvt_pk_bf16_f32 v66, v68, v69\n\t"
      "v_cvt_pk_bf16_f32 v67, v70, v71\n\t"
      "v_mfma_f32_16x16x32_bf16 a[204:207], a[156:159], a[60:63], a[204:207]\n\t"
      "global_store_dwordx4 v26, v[56:59], s[16:17] offset:192 sc0 sc1\n\t"
      "v_exp_f32 v7, v7 clamp\n\t"
      "v_exp_f32 v8, v8 clamp\n\t"
      "v_exp_f32 v9, v9 clamp\n\t"
      "v_pk_fma_f32 v[2:3], v[2:3], s[58:59], s[58:59] op_sel:[0,0,1] op_sel_hi:[1,0,1]\n\t"
      "v_pk_fma_f32 v[4:5], v[4:5], s[58:59], s[58:59] op_sel:[0,0,1] op_sel_hi:[1,0,1]\n\t"
      "v_pk_fma_f32 v[6:7], v[6:7], s[58:59], s[58:59] op_sel:[0,0,1] op_sel_hi:[1,0,1]\n\t"
      "v_pk_fma_f32 v[8:9], v[8:9], s[58:59], s[58:59] op_sel:[0,0,1] op_sel_hi:[1,0,1]\n\t"
      "v_rcp_f32 v2, v2\n\t"
      "v_permlane16_swap_b32 v64, v66\n\t"
      "v_rcp_f32 v3, v3\n\t"
      "v_permlane16_swap_b32 v65, v67\n\t"
      "v_rcp_f32 v4, v4\n\t"
      "v_pk_mul_f32 v[72:73], v[72:73], v[2:3]\n\t"
      "v_rcp_f32 v5, v5\n\t"
      "v_pk_fma_f32 v[18:19], v[80:81], s[54:55], s[54:55] op_sel:[0,0,1] op_sel_hi:[1,0,1]\n\t"
      "v_rcp_f32 v6, v6\n\t"
      "v_pk_mul_f32 v[74:75], v[74:75], v[4:5]\n\t"
      "v_rcp_f32 v7, v7\n\t"
      "v_pk_fma_f32 v[20:21], v[82:83], s[54:55], s[54:55] op_sel:[0,0,1] op_sel_hi:[1,0,1]\n\t"
      "v_rcp_f32 v8, v8\n\t"
      "v_pk_mul_f32 v[76:77], v[76:77], v[6:7]\n\t"
      "v_pk_fma_f32 v[22:23], v[84:85], s[54:55], s[54:55] op_sel:[0,0,1] op_sel_hi:[1,0,1]\n\t"
      "v_rcp_f32 v9, v9\n\t"
      "v_pk_fma_f32 v[24:25], v[86:87], s[54:55], s[54:55] op_sel:[0,0,1] op_sel_hi:[1,0,1]\n\t"
      "v_pk_mul_f32 v[78:79], v[78:79], v[8:9]\n\t"
      "v_mfma_f32_16x16x32_bf16 a[208:211], a[160:163], a[56:59], a[208:211]\n\t"
      "v_exp_f32 v18, v18 clamp\n\t"
      "v_mfma_f32_16x16x32_bf16 a[208:211], a[164:167], a[60:63], a[208:211]\n\t"
      "v_exp_f32 v19, v19 clamp\n\t"
      "v_mfma_f32_16x16x32_bf16 a[212:215], a[168:171], a[56:59], a[212:215]\n\t"
      "v_exp_f32 v20, v20 clamp\n\t"
      "v_mfma_f32_16x16x32_bf16 a[212:215], a[172:175], a[60:63], a[212:215]\n\t"
      "v_exp_f32 v21, v21 clamp\n\t"
      "v_mfma_f32_16x16x32_bf16 a[216:219], a[176:179], a[56:59], a[216:219]\n\t"
      "v_exp_f32 v22, v22 clamp\n\t"
      "v_mfma_f32_16x16x32_bf16 a[216:219], a[180:183], a[60:63], a[216:219]\n\t"
      "v_cvt_pk_bf16_f32 v72, v72, v73\n\t"
      "v_cvt_pk_bf16_f32 v73, v74, v75\n\t"
      "v_mfma_f32_16x16x32_bf16 a[220:223], a[184:187], a[56:59], a[220:223]\n\t"
      "global_store_dwordx4 v26, v[64:67], s[18:19] offset:0 sc0 sc1\n\t"
      "v_exp_f32 v23, v23 clamp\n\t"
      "v_exp_f32 v24, v24 clamp\n\t"
      "v_exp_f32 v25, v25 clamp\n\t"
      "v_pk_fma_f32 v[18:19], v[18:19], s[58:59], s[58:59] op_sel:[0,0,1] op_sel_hi:[1,0,1]\n\t"
      "v_cvt_pk_bf16_f32 v74, v76, v77\n\t"
      "v_pk_fma_f32 v[20:21], v[20:21], s[58:59], s[58:59] op_sel:[0,0,1] op_sel_hi:[1,0,1]\n\t"
      "v_cvt_pk_bf16_f32 v75, v78, v79\n\t"
      "v_pk_fma_f32 v[22:23], v[22:23], s[58:59], s[58:59] op_sel:[0,0,1] op_sel_hi:[1,0,1]\n\t"
      "v_pk_fma_f32 v[24:25], v[24:25], s[58:59], s[58:59] op_sel:[0,0,1] op_sel_hi:[1,0,1]\n\t"
      "v_rcp_f32 v18, v18\n\t"
      "v_permlane16_swap_b32 v72, v74\n\t"
      "v_rcp_f32 v19, v19\n\t"
      "v_permlane16_swap_b32 v73, v75\n\t"
      "v_rcp_f32 v20, v20\n\t"
      "v_pk_mul_f32 v[80:81], v[80:81], v[18:19]\n\t"
      "v_rcp_f32 v21, v21\n\t"
      "v_pk_fma_f32 v[2:3], v[88:89], s[54:55], s[54:55] op_sel:[0,0,1] op_sel_hi:[1,0,1]\n\t"
      "v_rcp_f32 v22, v22\n\t"
      "v_pk_mul_f32 v[82:83], v[82:83], v[20:21]\n\t"
      "v_rcp_f32 v23, v23\n\t"
      "v_pk_fma_f32 v[4:5], v[90:91], s[54:55], s[54:55] op_sel:[0,0,1] op_sel_hi:[1,0,1]\n\t"
      "v_rcp_f32 v24, v24\n\t"
      "v_pk_mul_f32 v[84:85], v[84:85], v[22:23]\n\t"
      "v_pk_fma_f32 v[6:7], v[92:93], s[54:55], s[54:55] op_sel:[0,0,1] op_sel_hi:[1,0,1]\n\t"
      "v_rcp_f32 v25, v25\n\t"
      "v_pk_fma_f32 v[8:9], v[94:95], s[54:55], s[54:55] op_sel:[0,0,1] op_sel_hi:[1,0,1]\n\t"
      "v_pk_mul_f32 v[86:87], v[86:87], v[24:25]\n\t"
      "v_mfma_f32_16x16x32_bf16 a[220:223], a[188:191], a[60:63], a[220:223]\n\t"
      "v_exp_f32 v2, v2 clamp\n\t"
      "s_waitcnt vmcnt(18)\n\t"
      "v_mfma_f32_16x16x32_bf16 v[128:131], a[64:67], a[0:3], v[128:131]\n\t"
      "global_load_dwordx4 a[56:59], v31, s[36:37] offset:0 \n\t"
      "v_exp_f32 v3, v3 clamp\n\t"
      "v_mfma_f32_16x16x32_bf16 v[128:131], a[68:71], a[4:7], v[128:131]\n\t"
      "v_exp_f32 v4, v4 clamp\n\t"
      "v_mfma_f32_16x16x32_bf16 v[132:135], a[72:75], a[0:3], v[132:135]\n\t"
      "v_exp_f32 v5, v5 clamp\n\t"
      "v_mfma_f32_16x16x32_bf16 v[132:135], a[76:79], a[4:7], v[132:135]\n\t"
      "v_exp_f32 v6, v6 clamp\n\t"
      "v_mfma_f32_16x16x32_bf16 v[136:139], a[80:83], a[0:3], v[136:139]\n\t"
      "global_load_dwordx4 a[60:63], v31, s[36:37] offset:1024 \n\t"
      "v_cvt_pk_bf16_f32 v80, v80, v81\n\t"
      "v_cvt_pk_bf16_f32 v81, v82, v83\n\t"
      "v_mfma_f32_16x16x32_bf16 v[136:139], a[84:87], a[4:7], v[136:139]\n\t"
      "v_cvt_pk_bf16_f32 v82, v84, v85\n\t"
      "v_cvt_pk_bf16_f32 v83, v86, v87\n\t"
      "v_mfma_f32_16x16x32_bf16 v[140:143], a[88:91], a[0:3], v[140:143]\n\t"
      "s_add_u32 s36, s36, 16384\n\t"
      "s_addc_u32 s37, s37, 0\n\t"
      "global_store_dwordx4 v26, v[72:75], s[18:19] offset:64 sc0 sc1\n\t"
      "v_exp_f32 v7, v7 clamp\n\t"
      "v_exp_f32 v8, v8 clamp\n\t"
      "v_exp_f32 v9, v9 clamp\n\t"
      "v_pk_fma_f32 v[2:3], v[2:3], s[58:59], s[58:59] op_sel:[0,0,1] op_sel_hi:[1,0,1]\n\t"
      "v_pk_fma_f32 v[4:5], v[4:5], s[58:59], s[58:59] op_sel:[0,0,1] op_sel_hi:[1,0,1]\n\t"
      "v_pk_fma_f32 v[6:7], v[6:7], s[58:59], s[58:59] op_sel:[0,0,1] op_sel_hi:[1,0,1]\n\t"
      "v_pk_fma_f32 v[8:9], v[8:9], s[58:59], s[58:59] op_sel:[0,0,1] op_sel_hi:[1,0,1]\n\t"
      "v_rcp_f32 v2, v2\n\t"
      "v_permlane16_swap_b32 v80, v82\n\t"
      "v_rcp_f32 v3, v3\n\t"
      "v_permlane16_swap_b32 v81, v83\n\t"
      "v_rcp_f32 v4, v4\n\t"
      "v_pk_mul_f32 v[88:89], v[88:89], v[2:3]\n\t"
      "v_rcp_f32 v5, v5\n\t"
      "v_pk_fma_f32 v[18:19], v[96:97], s[54:55], s[54:55] op_sel:[0,0,1] op_sel_hi:[1,0,1]\n\t"
      "v_rcp_f32 v6, v6\n\t"
      "v_pk_mul_f32 v[90:91], v[90:91], v[4:5]\n\t"
      "v_rcp_f32 v7, v7\n\t"
      "v_pk_fma_f32 v[20:21], v[98:99], s[54:55], s[54:55] op_sel:[0,0,1] op_sel_hi:[1,0,1]\n\t"
      "v_rcp_f32 v8, v8\n\t"
      "v_pk_mul_f32 v[92:93], v[92:93], v[6:7]\n\t"
      "v_pk_fma_f32 v[22:23], v[100:101], s[54:55], s[54:55] op_sel:[0,0,1] op_sel_hi:[1,0,1]\n\t"
      "v_rcp_f32 v9, v9\n\t"
      "v_pk_fma_f32 v[24:25], v[102:103], s[54:55], s[54:55] op_sel:[0,0,1] op_sel_hi:[1,0,1]\n\t"
      "v_pk_mul_f32 v[94:95], v[94:95], v[8:9]\n\t"
      "v_mfma_f32_16x16x32_bf16 v[140:143], a[92:95], a[4:7], v[140:143]\n\t"
      "v_exp_f32 v18, v18 clamp\n\t"
      "v_mfma_f32_16x16x32_bf16 v[144:147], a[96:99], a[0:3], v[144:147]\n\t"
      "v_exp_f32 v19, v19 clamp\n\t"
      "v_mfma_f32_16x16x32_bf16 v[144:147], a[100:103], a[4:7], v[144:147]\n\t"
      "v_exp_f32 v20, v20 clamp\n\t"
      "v_mfma_f32_16x16x32_bf16 v[148:151], a[104:107], a[0:3], v[148:151]\n\t"
      "v_exp_f32 v21, v21 clamp\n\t"
      "v_mfma_f32_16x16x32_bf16 v[148:151], a[108:111], a[4:7], v[148:151]\n\t"
      "v_exp_f32 v22, v22 clamp\n\t"
      "v_mfma_f32_16x16x32_bf16 v[152:155], a[112:115], a[0:3], v[152:155]\n\t"
      "v_cvt_pk_bf16_f32 v88, v88, v89\n\t"
      "v_cvt_pk_bf16_f32 v89, v90, v91\n\t"
      "v_mfma_f32_16x16x32_bf16 v[152:155], a[116:119], a[4:7], v[152:155]\n\t"
      "global_store_dwordx4 v26, v[80:83], s[18:19] offset:128 sc0 sc1\n\t"
      "v_exp_f32 v23, v23 clamp\n\t"
      "v_exp_f32 v24, v24 clamp\n\t"
      "v_exp_f32 v25, v25 clamp\n\t"
      "v_pk_fma_f32 v[18:19], v[18:19], s[58:59], s[58:59] op_sel:[0,0,1] op_sel_hi:[1,0,1]\n\t"
      "v_cvt_pk_bf16_f32 v90, v92, v93\n\t"
      "v_pk_fma_f32 v[20:21], v[20:21], s[58:59], s[58:59] op_sel:[0,0,1] op_sel_hi:[1,0,1]\n\t"
      "v_cvt_pk_bf16_f32 v91, v94, v95\n\t"
      "v_pk_fma_f32 v[22:23], v[22:23], s[58:59], s[58:59] op_sel:[0,0,1] op_sel_hi:[1,0,1]\n\t"
      "v_pk_fma_f32 v[24:25], v[24:25], s[58:59], s[58:59] op_sel:[0,0,1] op_sel_hi:[1,0,1]\n\t"
      "v_rcp_f32 v18, v18\n\t"
      "v_permlane16_swap_b32 v88, v90\n\t"
      "v_rcp_f32 v19, v19\n\t"
      "v_permlane16_swap_b32 v89, v91\n\t"
      "v_rcp_f32 v20, v20\n\t"
      "v_pk_mul_f32 v[96:97], v[96:97], v[18:19]\n\t"
      "v_rcp_f32 v21, v21\n\t"
      "v_pk_fma_f32 v[2:3], v[104:105], s[54:55], s[54:55] op_sel:[0,0,1] op_sel_hi:[1,0,1]\n\t"
      "v_rcp_f32 v22, v22\n\t"
      "v_pk_mul_f32 v[98:99], v[98:99], v[20:21]\n\t"
      "v_rcp_f32 v23, v23\n\t"
      "v_pk_fma_f32 v[4:5], v[106:107], s[54:55], s[54:55] op_sel:[0,0,1] op_sel_hi:[1,0,1]\n\t"
      "v_rcp_f32 v24, v24\n\t"
      "v_pk_mul_f32 v[100:101], v[100:101], v[22:23]\n\t"
      "v_pk_fma_f32 v[6:7], v[108:109], s[54:55], s[54:55] op_sel:[0,0,1] op_sel_hi:[1,0,1]\n\t"
      "v_rcp_f32 v25, v25\n\t"
      "v_pk_fma_f32 v[8:9], v[110:111], s[54:55], s[54:55] op_sel:[0,0,1] op_sel_hi:[1,0,1]\n\t"
      "v_pk_mul_f32 v[102:103], v[102:103], v[24:25]\n\t"
      "v_mfma_f32_16x16x32_bf16 v[156:159], a[120:123], a[0:3], v[156:159]\n\t"
      "v_exp_f32 v2, v2 clamp\n\t"
      "v_mfma_f32_16x16x32_bf16 v[156:159], a[124:127], a[4:7], v[156:159]\n\t"
      "v_exp_f32 v3, v3 clamp\n\t"
      "s_waitcnt vmcnt(19)\n\t"
      "v_mfma_f32_16x16x32_bf16 v[160:163], a[64:67], a[8:11], v[160:163]\n\t"
      "v_exp_f32 v4, v4 clamp\n\t"
      "v_mfma_f32_16x16x32_bf16 v[160:163], a[68:71], a[12:15], v[160:163]\n\t"
      "v_exp_f32 v5, v5 clamp\n\t"
      "v_mfma_f32_16x16x32_bf16 v[164:167], a[72:75], a[8:11], v[164:167]\n\t"
      "v_exp_f32 v6, v6 clamp\n\t"
      "v_mfma_f32_16x16x32_bf16 v[164:167], a[76:79], a[12:15], v[164:167]\n\t"
      "v_cvt_pk_bf16_f32 v96, v96, v97\n\t"
      "v_cvt_pk_bf16_f32 v97, v98, v99\n\t"
      "v_mfma_f32_16x16x32_bf16 v[168:171], a[80:83], a[8:11], v[168:171]\n\t"
      "v_cvt_pk_bf16_f32 v98, v100, v101\n\t"
      "v_cvt_pk_bf16_f32 v99, v102, v103\n\t"
      "v_mfma_f32_16x16x32_bf16 v[168:171], a[84:87], a[12:15], v[168:171]\n\t"
      "global_store_dwordx4 v26, v[88:91], s[18:19] offset:192 sc0 sc1\n\t"
      "v_exp_f32 v7, v7 clamp\n\t"
      "v_exp_f32 v8, v8 clamp\n\t"
      "v_exp_f32 v9, v9 clamp\n\t"
      "v_pk_fma_f32 v[2:3], v[2:3], s[58:59], s[58:59] op_sel:[0,0,1] op_sel_hi:[1,0,1]\n\t"
      "v_pk_fma_f32 v[4:5], v[4:5], s[58:59], s[58:59] op_sel:[0,0,1] op_sel_hi:[1,0,1]\n\t"
      "v_pk_fma_f32 v[6:7], v[6:7], s[58:59], s[58:59] op_sel:[0,0,1] op_sel_hi:[1,0,1]\n\t"
      "v_pk_fma_f32 v[8:9], v[8:9], s[58:59], s[58:59] op_sel:[0,0,1] op_sel_hi:[1,0,1]\n\t"
      "v_rcp_f32 v2, v2\n\t"
      "v_permlane16_swap_b32 v96, v98\n\t"
      "v_rcp_f32 v3, v3\n\t"
      "v_permlane16_swap_b32 v97, v99\n\t"
      "v_rcp_f32 v4, v4\n\t"
      "v_pk_mul_f32 v[104:105], v[104:105], v[2:3]\n\t"
      "v_rcp_f32 v5, v5\n\t"
      "v_pk_fma_f32 v[18:19], v[112:113], s[54:55], s[54:55] op_sel:[0,0,1] op_sel_hi:[1,0,1]\n\t"
      "v_rcp_f32 v6, v6\n\t"
      "v_pk_mul_f32 v[106:107], v[106:107], v[4:5]\n\t"
      "v_rcp_f32 v7, v7\n\t"
      "v_pk_fma_f32 v[20:21], v[114:115], s[54:55], s[54:55] op_sel:[0,0,1] op_sel_hi:[1,0,1]\n\t"
      "v_rcp_f32 v8, v8\n\t"
      "v_pk_mul_f32 v[108:109], v[108:109], v[6:7]\n\t"
      "v_pk_fma_f32 v[22:23], v[116:117], s[54:55], s[54:55] op_sel:[0,0,1] op_sel_hi:[1,0,1]\n\t"
      "v_rcp_f32 v9, v9\n\t"
      "v_pk_fma_f32 v[24:25], v[118:119], s[54:55], s[54:55] op_sel:[0,0,1] op_sel_hi:[1,0,1]\n\t"
      "v_pk_mul_f32 v[110:111], v[110:111], v[8:9]\n\t"
      "v_mfma_f32_16x16x32_bf16 v[172:175], a[88:91], a[8:11], v[172:175]\n\t"
      "v_exp_f32 v18, v18 clamp\n\t"
      "v_mfma_f32_16x16x32_bf16 v[172:175], a[92:95], a[12:15], v[172:175]\n\t"
      "v_exp_f32 v19, v19 clamp\n\t"
      "v_mfma_f32_16x16x32_bf16 v[176:179], a[96:99], a[8:11], v[176:179]\n\t"
      "v_exp_f32 v20, v20 clamp\n\t"
      "v_mfma_f32_16x16x32_bf16 v[176:179], a[100:103], a[12:15], v[176:179]\n\t"
      "v_exp_f32 v21, v21 clamp\n\t"
      "v_mfma_f32_16x16x32_bf16 v[180:183], a[104:107], a[8:11], v[180:183]\n\t"
      "v_exp_f32 v22, v22 clamp\n\t"
      "v_mfma_f32_16x16x32_bf16 v[180:183], a[108:111], a[12:15], v[180:183]\n\t"
      "v_cvt_pk_bf16_f32 v104, v104, v105\n\t"
      "v_cvt_pk_bf16_f32 v105, v106, v107\n\t"
      "v_mfma_f32_16x16x32_bf16 v[184:187], a[112:115], a[8:11], v[184:187]\n\t"
      "v_cvt_pk_bf16_f32 v106, v108, v109\n\t"
      "v_cvt_pk_bf16_f32 v107, v110, v111\n\t"
      "v_mfma_f32_16x16x32_bf16 v[184:187], a[116:119], a[12:15], v[184:187]\n\t"
      "global_store_dwordx4 v26, v[96:99], s[20:21] offset:0 sc0 sc1\n\t"
      "v_exp_f32 v23, v23 clamp\n\t"
      "v_exp_f32 v24, v24 clamp\n\t"
      "v_exp_f32 v25, v25 clamp\n\t"
      "v_pk_fma_f32 v[18:19], v[18:19], s[58:59], s[58:59] op_sel:[0,0,1] op_sel_hi:[1,0,1]\n\t"
      "v_pk_fma_f32 v[20:21], v[20:21], s[58:59], s[58:59] op_sel:[0,0,1] op_sel_hi:[1,0,1]\n\t"
      "v_pk_fma_f32 v[22:23], v[22:23], s[58:59], s[58:59] op_sel:[0,0,1] op_sel_hi:[1,0,1]\n\t"
      "v_pk_fma_f32 v[24:25], v[24:25], s[58:59], s[58:59] op_sel:[0,0,1] op_sel_hi:[1,0,1]\n\t"
      "v_rcp_f32 v18, v18\n\t"
      "v_permlane16_swap_b32 v104, v106\n\t"
      "v_rcp_f32 v19, v19\n\t"
      "v_permlane16_swap_b32 v105, v107\n\t"
      "v_rcp_f32 v20, v20\n\t"
      "v_pk_mul_f32 v[112:113], v[112:113], v[18:19]\n\t"
      "v_rcp_f32 v21, v21\n\t"
      "v_pk_fma_f32 v[2:3], v[120:121], s[54:55], s[54:55] op_sel:[0,0,1] op_sel_hi:[1,0,1]\n\t"
      "v_rcp_f32 v22, v22\n\t"
      "v_pk_mul_f32 v[114:115], v[114:115], v[20:21]\n\t"
      "v_rcp_f32 v23, v23\n\t"
      "v_pk_fma_f32 v[4:5], v[122:123], s[54:55], s[54:55] op_sel:[0,0,1] op_sel_hi:[1,0,1]\n\t"
      "v_rcp_f32 v24, v24\n\t"
      "v_pk_mul_f32 v[116:117], v[116:117], v[22:23]\n\t"
      "v_pk_fma_f32 v[6:7], v[124:125], s[54:55], s[54:55] op_sel:[0,0,1] op_sel_hi:[1,0,1]\n\t"
      "v_rcp_f32 v25, v25\n\t"
      "v_pk_fma_f32 v[8:9], v[126:127], s[54:55], s[54:55] op_sel:[0,0,1] op_sel_hi:[1,0,1]\n\t"
      "v_pk_mul_f32 v[118:119], v[118:119], v[24:25]\n\t"
      "v_mfma_f32_16x16x32_bf16 v[188:191], a[120:123], a[8:11], v[188:191]\n\t"
      "v_exp_f32 v2, v2 clamp\n\t"
      "v_mfma_f32_16x16x32_bf16 v[188:191], a[124:127], a[12:15], v[188:191]\n\t"
      "v_exp_f32 v3, v3 clamp\n\t"
      "s_waitcnt vmcnt(19)\n\t"
      "v_mfma_f32_16x16x32_bf16 v[192:195], a[64:67], a[16:19], v[192:195]\n\t"
      "v_exp_f32 v4, v4 clamp\n\t"
      "v_mfma_f32_16x16x32_bf16 v[192:195], a[68:71], a[20:23], v[192:195]\n\t"
      "v_exp_f32 v5, v5 clamp\n\t"
      "v_mfma_f32_16x16x32_bf16 v[196:199], a[72:75], a[16:19], v[196:199]\n\t"
      "v_exp_f32 v6, v6 clamp\n\t"
      "v_mfma_f32_16x16x32_bf16 v[196:199], a[76:79], a[20:23], v[196:199]\n\t"
      "v_cvt_pk_bf16_f32 v112, v112, v113\n\t"
      "v_cvt_pk_bf16_f32 v113, v114, v115\n\t"
      "v_mfma_f32_16x16x32_bf16 v[200:203], a[80:83], a[16:19], v[200:203]\n\t"
      "global_store_dwordx4 v26, v[104:107], s[20:21] offset:64 sc0 sc1\n\t"
      "v_exp_f32 v7, v7 clamp\n\t"
      "v_exp_f32 v8, v8 clamp\n\t"
      "v_exp_f32 v9, v9 clamp\n\t"
      "v_pk_fma_f32 v[2:3], v[2:3], s[58:59], s[58:59] op_sel:[0,0,1] op_sel_hi:[1,0,1]\n\t"
      "v_cvt_pk_bf16_f32 v114, v116, v117\n\t"
      "v_pk_fma_f32 v[4:5], v[4:5], s[58:59], s[58:59] op_sel:[0,0,1] op_sel_hi:[1,0,1]\n\t"
      "v_cvt_pk_bf16_f32 v115, v118, v119\n\t"
      "v_pk_fma_f32 v[6:7], v[6:7], s[58:59], s[58:59] op_sel:[0,0,1] op_sel_hi:[1,0,1]\n\t"
      "v_pk_fma_f32 v[8:9], v[8:9], s[58:59], s[58:59] op_sel:[0,0,1] op_sel_hi:[1,0,1]\n\t"
      "v_rcp_f32 v2, v2\n\t"
      "v_permlane16_swap_b32 v112, v114\n\t"
      "v_rcp_f32 v3, v3\n\t"
      "v_permlane16_swap_b32 v113, v115\n\t"
      "v_rcp_f32 v4, v4\n\t"
      "v_accvgpr_read_b32 v32, a192\n\t"
      "v_accvgpr_read_b32 v33, a193\n\t"
      "v_rcp_f32 v5, v5\n\t"
      "v_pk_mul_f32 v[120:121], v[120:121], v[2:3]\n\t"
      "v_accvgpr_read_b32 v34, a194\n\t"
      "v_accvgpr_read_b32 v35, a195\n\t"
      "v_rcp_f32 v6, v6\n\t"
      "v_accvgpr_read_b32 v36, a196\n\t"
      "v_accvgpr_read_b32 v37, a197\n\t"
      "v_rcp_f32 v7, v7\n\t"
      "v_pk_mul_f32 v[122:123], v[122:123], v[4:5]\n\t"
      "v_accvgpr_read_b32 v38, a198\n\t"
      "v_accvgpr_read_b32 v39, a199\n\t"
      "v_rcp_f32 v8, v8\n\t"
      "v_pk_fma_f32 v[18:19], v[32:33], s[54:55], s[54:55] op_sel:[0,0,1] op_sel_hi:[1,0,1]\n\t"
      "v_pk_mul_f32 v[124:125], v[124:125], v[6:7]\n\t"
      "v_pk_fma_f32 v[20:21], v[34:35], s[54:55], s[54:55] op_sel:[0,0,1] op_sel_hi:[1,0,1]\n\t"
      "v_rcp_f32 v9, v9\n\t"
      "v_pk_fma_f32 v[22:23], v[36:37], s[54:55], s[54:55] op_sel:[0,0,1] op_sel_hi:[1,0,1]\n\t"
      "v_pk_fma_f32 v[24:25], v[38:39], s[54:55], s[54:55] op_sel:[0,0,1] op_sel_hi:[1,0,1]\n\t"
      "v_pk_mul_f32 v[126:127], v[126:127], v[8:9]\n\t"
      "v_mfma_f32_16x16x32_bf16 v[200:203], a[84:87], a[20:23], v[200:203]\n\t"
      "v_exp_f32 v18, v18 clamp\n\t"
      "v_mfma_f32_16x16x32_bf16 v[204:207], a[88:91], a[16:19], v[204:207]\n\t"
      "v_exp_f32 v19, v19 clamp\n\t"
      "v_mfma_f32_16x16x32_bf16 v[204:207], a[92:95], a[20:23], v[204:207]\n\t"
      "v_exp_f32 v20, v20 clamp\n\t"
      "v_mfma_f32_16x16x32_bf16 v[208:211], a[96:99], a[16:19], v[208:211]\n\t"
      "v_exp_f32 v21, v21 clamp\n\t"
      "v_mfma_f32_16x16x32_bf16 v[208:211], a[100:103], a[20:23], v[208:211]\n\t"
      "v_exp_f32 v22, v22 clamp\n\t"
      "v_mfma_f32_16x16x32_bf16 v[212:215], a[104:107], a[16:19], v[212:215]\n\t"
      "v_cvt_pk_bf16_f32 v120, v120, v121\n\t"
      "v_cvt_pk_bf16_f32 v121, v122, v123\n\t"
      "v_mfma_f32_16x16x32_bf16 v[212:215], a[108:111], a[20:23], v[212:215]\n\t"
      "v_cvt_pk_bf16_f32 v122, v124, v125\n\t"
      "v_cvt_pk_bf16_f32 v123, v126, v127\n\t"
      "v_mfma_f32_16x16x32_bf16 v[216:219], a[112:115], a[16:19], v[216:219]\n\t"
      "global_store_dwordx4 v26, v[112:115], s[20:21] offset:128 sc0 sc1\n\t"
      "v_exp_f32 v23, v23 clamp\n\t"
      "v_exp_f32 v24, v24 clamp\n\t"
      "v_exp_f32 v25, v25 clamp\n\t"
      "v_pk_fma_f32 v[18:19], v[18:19], s[58:59], s[58:59] op_sel:[0,0,1] op_sel_hi:[1,0,1]\n\t"
      "v_pk_fma_f32 v[20:21], v[20:21], s[58:59], s[58:59] op_sel:[0,0,1] op_sel_hi:[1,0,1]\n\t"
      "v_pk_fma_f32 v[22:23], v[22:23], s[58:59], s[58:59] op_sel:[0,0,1] op_sel_hi:[1,0,1]\n\t"
      "v_pk_fma_f32 v[24:25], v[24:25], s[58:59], s[58:59] op_sel:[0,0,1] op_sel_hi:[1,0,1]\n\t"
      "v_rcp_f32 v18, v18\n\t"
      "v_permlane16_swap_b32 v120, v122\n\t"
      "v_rcp_f32 v19, v19\n\t"
      "v_permlane16_swap_b32 v121, v123\n\t"
      "v_rcp_f32 v20, v20\n\t"
      "v_accvgpr_read_b32 v40, a200\n\t"
      "v_accvgpr_read_b32 v41, a201\n\t"
      "v_rcp_f32 v21, v21\n\t"
      "v_pk_mul_f32 v[32:33], v[32:33], v[18:19]\n\t"
      "v_accvgpr_read_b32 v42, a202\n\t"
      "v_accvgpr_read_b32 v43, a203\n\t"
      "v_rcp_f32 v22, v22\n\t"
      "v_accvgpr_read_b32 v44, a204\n\t"
      "v_accvgpr_read_b32 v45, a205\n\t"
      "v_rcp_f32 v23, v23\n\t"
      "v_pk_mul_f32 v[34:35], v[34:35], v[20:21]\n\t"
      "v_accvgpr_read_b32 v46, a206\n\t"
      "v_accvgpr_read_b32 v47, a207\n\t"
      "v_rcp_f32 v24, v24\n\t"
      "v_pk_fma_f32 v[2:3], v[40:41], s[54:55], s[54:55] op_sel:[0,0,1] op_sel_hi:[1,0,1]\n\t"
      "v_pk_mul_f32 v[36:37], v[36:37], v[22:23]\n\t"
      "v_pk_fma_f32 v[4:5], v[42:43], s[54:55], s[54:55] op_sel:[0,0,1] op_sel_hi:[1,0,1]\n\t"
      "v_rcp_f32 v25, v25\n\t"
      "v_pk_fma_f32 v[6:7], v[44:45], s[54:55], s[54:55] op_sel:[0,0,1] op_sel_hi:[1,0,1]\n\t"
      "v_pk_fma_f32 v[8:9], v[46:47], s[54:55], s[54:55] op_sel:[0,0,1] op_sel_hi:[1,0,1]\n\t"
      "v_pk_mul_f32 v[38:39], v[38:39], v[24:25]\n\t"
      "v_mfma_f32_16x16x32_bf16 v[216:219], a[116:119], a[20:23], v[216:219]\n\t"
      "v_exp_f32 v2, v2 clamp\n\t"
      "v_mfma_f32_16x16x32_bf16 v[220:223], a[120:123], a[16:19], v[220:223]\n\t"
      "v_exp_f32 v3, v3 clamp\n\t"
      "v_mfma_f32_16x16x32_bf16 v[220:223], a[124:127], a[20:23], v[220:223]\n\t"
      "v_exp_f32 v4, v4 clamp\n\t"
      "s_waitcnt vmcnt(19)\n\t"
      "v_mfma_f32_16x16x32_bf16 v[224:227], a[64:67], a[24:27], v[224:227]\n\t"
      "v_exp_f32 v5, v5 clamp\n\t"
      "v_mfma_f32_16x16x32_bf16 v[224:227], a[68:71], a[28:31], v[224:227]\n\t"
      "v_exp_f32 v6, v6 clamp\n\t"
      "v_mfma_f32_16x16x32_bf16 v[228:231], a[72:75], a[24:27], v[228:231]\n\t"
      "v_cvt_pk_bf16_f32 v32, v32, v33\n\t"
      "v_cvt_pk_bf16_f32 v33, v34, v35\n\t"
      "v_mfma_f32_16x16x32_bf16 v[228:231], a[76:79], a[28:31], v[228:231]\n\t"
      "global_store_dwordx4 v26, v[120:123], s[20:21] offset:192 sc0 sc1\n\t"
      "v_exp_f32 v7, v7 clamp\n\t"
      "v_exp_f32 v8, v8 clamp\n\t"
      "v_exp_f32 v9, v9 clamp\n\t"
      "v_pk_fma_f32 v[2:3], v[2:3], s[58:59], s[58:59] op_sel:[0,0,1] op_sel_hi:[1,0,1]\n\t"
      "v_cvt_pk_bf16_f32 v34, v36, v37\n\t"
      "v_pk_fma_f32 v[4:5], v[4:5], s[58:59], s[58:59] op_sel:[0,0,1] op_sel_hi:[1,0,1]\n\t"
      "v_cvt_pk_bf16_f32 v35, v38, v39\n\t"
      "v_pk_fma_f32 v[6:7], v[6:7], s[58:59], s[58:59] op_sel:[0,0,1] op_sel_hi:[1,0,1]\n\t"
      "v_pk_fma_f32 v[8:9], v[8:9], s[58:59], s[58:59] op_sel:[0,0,1] op_sel_hi:[1,0,1]\n\t"
      "v_rcp_f32 v2, v2\n\t"
      "v_permlane16_swap_b32 v32, v34\n\t"
      "v_rcp_f32 v3, v3\n\t"
      "v_permlane16_swap_b32 v33, v35\n\t"
      "v_rcp_f32 v4, v4\n\t"
      "v_accvgpr_read_b32 v48, a208\n\t"
      "v_accvgpr_read_b32 v49, a209\n\t"
      "v_rcp_f32 v5, v5\n\t"
      "v_pk_mul_f32 v[40:41], v[40:41], v[2:3]\n\t"
      "v_accvgpr_read_b32 v50, a210\n\t"
      "v_accvgpr_read_b32 v51, a211\n\t"
      "v_rcp_f32 v6, v6\n\t"
      "v_accvgpr_read_b32 v52, a212\n\t"
      "v_accvgpr_read_b32 v53, a213\n\t"
      "v_rcp_f32 v7, v7\n\t"
      "v_pk_mul_f32 v[42:43], v[42:43], v[4:5]\n\t"
      "v_accvgpr_read_b32 v54, a214\n\t"
      "v_accvgpr_read_b32 v55, a215\n\t"
      "v_rcp_f32 v8, v8\n\t"
      "v_pk_fma_f32 v[18:19], v[48:49], s[54:55], s[54:55] op_sel:[0,0,1] op_sel_hi:[1,0,1]\n\t"
      "v_pk_mul_f32 v[44:45], v[44:45], v[6:7]\n\t"
      "v_pk_fma_f32 v[20:21], v[50:51], s[54:55], s[54:55] op_sel:[0,0,1] op_sel_hi:[1,0,1]\n\t"
      "v_rcp_f32 v9, v9\n\t"
      "v_pk_fma_f32 v[22:23], v[52:53], s[54:55], s[54:55] op_sel:[0,0,1] op_sel_hi:[1,0,1]\n\t"
      "v_pk_fma_f32 v[24:25], v[54:55], s[54:55], s[54:55] op_sel:[0,0,1] op_sel_hi:[1,0,1]\n\t"
      "v_pk_mul_f32 v[46:47], v[46:47], v[8:9]\n\t"
      "v_mfma_f32_16x16x32_bf16 v[232:235], a[80:83], a[24:27], v[232:235]\n\t"
      "v_exp_f32 v18, v18 clamp\n\t"
      "v_mfma_f32_16x16x32_bf16 v[232:235], a[84:87], a[28:31], v[232:235]\n\t"
      "v_exp_f32 v19, v19 clamp\n\t"
      "v_mfma_f32_16x16x32_bf16 v[236:239], a[88:91], a[24:27], v[236:239]\n\t"
      "v_exp_f32 v20, v20 clamp\n\t"
      "v_mfma_f32_16x16x32_bf16 v[236:239], a[92:95], a[28:31], v[236:239]\n\t"
      "v_exp_f32 v21, v21 clamp\n\t"
      "v_mfma_f32_16x16x32_bf16 v[240:243], a[96:99], a[24:27], v[240:243]\n\t"
      "v_exp_f32 v22, v22 clamp\n\t"
      "v_mfma_f32_16x16x32_bf16 v[240:243], a[100:103], a[28:31], v[240:243]\n\t"
      "v_cvt_pk_bf16_f32 v40, v40, v41\n\t"
      "v_cvt_pk_bf16_f32 v41, v42, v43\n\t"
      "v_mfma_f32_16x16x32_bf16 v[244:247], a[104:107], a[24:27], v[244:247]\n\t"
      "v_cvt_pk_bf16_f32 v42, v44, v45\n\t"
      "v_cvt_pk_bf16_f32 v43, v46, v47\n\t"
      "v_mfma_f32_16x16x32_bf16 v[244:247], a[108:111], a[28:31], v[244:247]\n\t"
      "global_store_dwordx4 v26, v[32:35], s[22:23] offset:0 sc0 sc1\n\t"
      "v_exp_f32 v23, v23 clamp\n\t"
      "v_exp_f32 v24, v24 clamp\n\t"
      "v_exp_f32 v25, v25 clamp\n\t"
      "v_pk_fma_f32 v[18:19], v[18:19], s[58:59], s[58:59] op_sel:[0,0,1] op_sel_hi:[1,0,1]\n\t"
      "v_pk_fma_f32 v[20:21], v[20:21], s[58:59], s[58:59] op_sel:[0,0,1] op_sel_hi:[1,0,1]\n\t"
      "v_pk_fma_f32 v[22:23], v[22:23], s[58:59], s[58:59] op_sel:[0,0,1] op_sel_hi:[1,0,1]\n\t"
      "v_pk_fma_f32 v[24:25], v[24:25], s[58:59], s[58:59] op_sel:[0,0,1] op_sel_hi:[1,0,1]\n\t"
      "v_rcp_f32 v18, v18\n\t"
      "v_permlane16_swap_b32 v40, v42\n\t"
      "v_rcp_f32 v19, v19\n\t"
      "v_permlane16_swap_b32 v41, v43\n\t"
      "v_rcp_f32 v20, v20\n\t"
      "v_accvgpr_read_b32 v56, a216\n\t"
      "v_accvgpr_read_b32 v57, a217\n\t"
      "v_rcp_f32 v21, v21\n\t"
      "v_pk_mul_f32 v[48:49], v[48:49], v[18:19]\n\t"
      "v_accvgpr_read_b32 v58, a218\n\t"
      "v_accvgpr_read_b32 v59, a219\n\t"
      "v_rcp_f32 v22, v22\n\t"
      "v_accvgpr_read_b32 v60, a220\n\t"
      "v_accvgpr_read_b32 v61, a221\n\t"
      "v_rcp_f32 v23, v23\n\t"
      "v_pk_mul_f32 v[50:51], v[50:51], v[20:21]\n\t"
      "v_accvgpr_read_b32 v62, a222\n\t"
      "v_accvgpr_read_b32 v63, a223\n\t"
      "v_rcp_f32 v24, v24\n\t"
      "v_pk_fma_f32 v[2:3], v[56:57], s[54:55], s[54:55] op_sel:[0,0,1] op_sel_hi:[1,0,1]\n\t"
      "v_pk_mul_f32 v[52:53], v[52:53], v[22:23]\n\t"
      "v_pk_fma_f32 v[4:5], v[58:59], s[54:55], s[54:55] op_sel:[0,0,1] op_sel_hi:[1,0,1]\n\t"
      "v_rcp_f32 v25, v25\n\t"
      "v_pk_fma_f32 v[6:7], v[60:61], s[54:55], s[54:55] op_sel:[0,0,1] op_sel_hi:[1,0,1]\n\t"
      "v_pk_fma_f32 v[8:9], v[62:63], s[54:55], s[54:55] op_sel:[0,0,1] op_sel_hi:[1,0,1]\n\t"
      "v_pk_mul_f32 v[54:55], v[54:55], v[24:25]\n\t"
      "v_mfma_f32_16x16x32_bf16 v[248:251], a[112:115], a[24:27], v[248:251]\n\t"
      "v_exp_f32 v2, v2 clamp\n\t"
      "v_mfma_f32_16x16x32_bf16 v[248:251], a[116:119], a[28:31], v[248:251]\n\t"
      "v_exp_f32 v3, v3 clamp\n\t"
      "v_mfma_f32_16x16x32_bf16 v[252:255], a[120:123], a[24:27], v[252:255]\n\t"
      "v_exp_f32 v4, v4 clamp\n\t"
      "v_mfma_f32_16x16x32_bf16 v[252:255], a[124:127], a[28:31], v[252:255]\n\t"
      "v_exp_f32 v5, v5 clamp\n\t"
      "s_waitcnt vmcnt(19)\n\t"
      "v_mfma_f32_16x16x32_bf16 v[128:131], a[128:131], a[32:35], v[128:131]\n\t"
      "v_exp_f32 v6, v6 clamp\n\t"
      "v_mfma_f32_16x16x32_bf16 v[128:131], a[132:135], a[36:39], v[128:131]\n\t"
      "v_cvt_pk_bf16_f32 v48, v48, v49\n\t"
      "v_cvt_pk_bf16_f32 v49, v50, v51\n\t"
      "v_mfma_f32_16x16x32_bf16 v[132:135], a[136:139], a[32:35], v[132:135]\n\t"
      "v_cvt_pk_bf16_f32 v50, v52, v53\n\t"
      "v_cvt_pk_bf16_f32 v51, v54, v55\n\t"
      "v_mfma_f32_16x16x32_bf16 v[132:135], a[140:143], a[36:39], v[132:135]\n\t"
      "global_store_dwordx4 v26, v[40:43], s[22:23] offset:64 sc0 sc1\n\t"
      "v_exp_f32 v7, v7 clamp\n\t"
      "v_exp_f32 v8, v8 clamp\n\t"
      "v_exp_f32 v9, v9 clamp\n\t"
      "v_pk_fma_f32 v[2:3], v[2:3], s[58:59], s[58:59] op_sel:[0,0,1] op_sel_hi:[1,0,1]\n\t"
      "v_pk_fma_f32 v[4:5], v[4:5], s[58:59], s[58:59] op_sel:[0,0,1] op_sel_hi:[1,0,1]\n\t"
      "v_pk_fma_f32 v[6:7], v[6:7], s[58:59], s[58:59] op_sel:[0,0,1] op_sel_hi:[1,0,1]\n\t"
      "v_pk_fma_f32 v[8:9], v[8:9], s[58:59], s[58:59] op_sel:[0,0,1] op_sel_hi:[1,0,1]\n\t"
      "v_rcp_f32 v2, v2\n\t"
      "v_permlane16_swap_b32 v48, v50\n\t"
      "v_rcp_f32 v3, v3\n\t"
      "v_permlane16_swap_b32 v49, v51\n\t"
      "v_rcp_f32 v4, v4\n\t"
      "v_pk_mul_f32 v[56:57], v[56:57], v[2:3]\n\t"
      "v_rcp_f32 v5, v5\n\t"
      "v_pk_fma_f32 v[18:19], v[128:129], s[54:55], s[54:55] op_sel:[0,0,1] op_sel_hi:[1,0,1]\n\t"
      "v_rcp_f32 v6, v6\n\t"
      "v_pk_mul_f32 v[58:59], v[58:59], v[4:5]\n\t"
      "v_rcp_f32 v7, v7\n\t"
      "v_pk_fma_f32 v[20:21], v[130:131], s[54:55], s[54:55] op_sel:[0,0,1] op_sel_hi:[1,0,1]\n\t"
      "v_rcp_f32 v8, v8\n\t"
      "v_pk_mul_f32 v[60:61], v[60:61], v[6:7]\n\t"
      "v_pk_fma_f32 v[22:23], v[132:133], s[54:55], s[54:55] op_sel:[0,0,1] op_sel_hi:[1,0,1]\n\t"
      "v_rcp_f32 v9, v9\n\t"
      "v_pk_fma_f32 v[24:25], v[134:135], s[54:55], s[54:55] op_sel:[0,0,1] op_sel_hi:[1,0,1]\n\t"
      "v_pk_mul_f32 v[62:63], v[62:63], v[8:9]\n\t"
      "v_mfma_f32_16x16x32_bf16 v[136:139], a[144:147], a[32:35], v[136:139]\n\t"
      "v_exp_f32 v18, v18 clamp\n\t"
      "v_mfma_f32_16x16x32_bf16 v[136:139], a[148:151], a[36:39], v[136:139]\n\t"
      "v_exp_f32 v19, v19 clamp\n\t"
      "v_mfma_f32_16x16x32_bf16 v[140:143], a[152:155], a[32:35], v[140:143]\n\t"
      "v_exp_f32 v20, v20 clamp\n\t"
      "v_mfma_f32_16x16x32_bf16 v[140:143], a[156:159], a[36:39], v[140:143]\n\t"
      "global_store_dwordx4 v26, v[48:51], s[22:23] offset:128 sc0 sc1\n\t"
      "v_exp_f32 v21, v21 clamp\n\t"
      "v_exp_f32 v22, v22 clamp\n\t"
      "v_exp_f32 v23, v23 clamp\n\t"
      "v_pk_fma_f32 v[18:19], v[18:19], s[58:59], s[58:59] op_sel:[0,0,1] op_sel_hi:[1,0,1]\n\t"
      "v_exp_f32 v24, v24 clamp\n\t"
      "v_pk_fma_f32 v[20:21], v[20:21], s[58:59], s[58:59] op_sel:[0,0,1] op_sel_hi:[1,0,1]\n\t"
      "v_exp_f32 v25, v25 clamp\n\t"
      "v_pk_fma_f32 v[22:23], v[22:23], s[58:59], s[58:59] op_sel:[0,0,1] op_sel_hi:[1,0,1]\n\t"
      "v_cvt_pk_bf16_f32 v56, v56, v57\n\t"
      "v_cvt_pk_bf16_f32 v57, v58, v59\n\t"
      "v_pk_fma_f32 v[24:25], v[24:25], s[58:59], s[58:59] op_sel:[0,0,1] op_sel_hi:[1,0,1]\n\t"
      "v_cvt_pk_bf16_f32 v58, v60, v61\n\t"
      "v_cvt_pk_bf16_f32 v59, v62, v63\n\t"
      "v_rcp_f32 v18, v18\n\t"
      "v_permlane16_swap_b32 v56, v58\n\t"
      "v_rcp_f32 v19, v19\n\t"
      "v_permlane16_swap_b32 v57, v59\n\t"
      "v_rcp_f32 v20, v20\n\t"
      "v_pk_mul_f32 v[128:129], v[128:129], v[18:19]\n\t"
      "v_rcp_f32 v21, v21\n\t"
      "v_pk_fma_f32 v[2:3], v[136:137], s[54:55], s[54:55] op_sel:[0,0,1] op_sel_hi:[1,0,1]\n\t"
      "v_rcp_f32 v22, v22\n\t"
      "v_pk_mul_f32 v[130:131], v[130:131], v[20:21]\n\t"
      "v_rcp_f32 v23, v23\n\t"
      "v_pk_fma_f32 v[4:5], v[138:139], s[54:55], s[54:55] op_sel:[0,0,1] op_sel_hi:[1,0,1]\n\t"
      "v_rcp_f32 v24, v24\n\t"
      "v_pk_mul_f32 v[132:133], v[132:133], v[22:23]\n\t"
      "v_pk_fma_f32 v[6:7], v[140:141], s[54:55], s[54:55] op_sel:[0,0,1] op_sel_hi:[1,0,1]\n\t"
      "v_rcp_f32 v25, v25\n\t"
      "v_pk_fma_f32 v[8:9], v[142:143], s[54:55], s[54:55] op_sel:[0,0,1] op_sel_hi:[1,0,1]\n\t"
      "v_pk_mul_f32 v[134:135], v[134:135], v[24:25]\n\t"
      "v_mfma_f32_16x16x32_bf16 v[144:147], a[160:163], a[32:35], v[144:147]\n\t"
      "v_exp_f32 v2, v2 clamp\n\t"
      "v_mfma_f32_16x16x32_bf16 v[144:147], a[164:167], a[36:39], v[144:147]\n\t"
      "v_exp_f32 v3, v3 clamp\n\t"
      "v_mfma_f32_16x16x32_bf16 v[148:151], a[168:171], a[32:35], v[148:151]\n\t"
      "v_exp_f32 v4, v4 clamp\n\t"
      "v_mfma_f32_16x16x32_bf16 v[148:151], a[172:175], a[36:39], v[148:151]\n\t"
      "global_store_dwordx4 v26, v[56:59], s[22:23] offset:192 sc0 sc1\n\t"
      "v_exp_f32 v5, v5 clamp\n\t"
      "v_exp_f32 v6, v6 clamp\n\t"
      "v_exp_f32 v7, v7 clamp\n\t"
      "v_pk_fma_f32 v[2:3], v[2:3], s[58:59], s[58:59] op_sel:[0,0,1] op_sel_hi:[1,0,1]\n\t"
      "v_exp_f32 v8, v8 clamp\n\t"
      "v_pk_fma_f32 v[4:5], v[4:5], s[58:59], s[58:59] op_sel:[0,0,1] op_sel_hi:[1,0,1]\n\t"
      "v_exp_f32 v9, v9 clamp\n\t"
      "v_pk_fma_f32 v[6:7], v[6:7], s[58:59], s[58:59] op_sel:[0,0,1] op_sel_hi:[1,0,1]\n\t"
      "v_cvt_pk_bf16_f32 v128, v128, v129\n\t"
      "v_cvt_pk_bf16_f32 v129, v130, v131\n\t"
      "v_pk_fma_f32 v[8:9], v[8:9], s[58:59], s[58:59] op_sel:[0,0,1] op_sel_hi:[1,0,1]\n\t"
      "v_cvt_pk_bf16_f32 v130, v132, v133\n\t"
      "v_cvt_pk_bf16_f32 v131, v134, v135\n\t"
      "v_rcp_f32 v2, v2\n\t"
      "v_permlane16_swap_b32 v128, v130\n\t"
      "v_rcp_f32 v3, v3\n\t"
      "v_permlane16_swap_b32 v129, v131\n\t"
      "v_rcp_f32 v4, v4\n\t"
      "v_pk_mul_f32 v[136:137], v[136:137], v[2:3]\n\t"
      "v_rcp_f32 v5, v5\n\t"
      "v_pk_fma_f32 v[18:19], v[144:145], s[54:55], s[54:55] op_sel:[0,0,1] op_sel_hi:[1,0,1]\n\t"
      "v_rcp_f32 v6, v6\n\t"
      "v_pk_mul_f32 v[138:139], v[138:139], v[4:5]\n\t"
      "v_rcp_f32 v7, v7\n\t"
      "v_pk_fma_f32 v[20:21], v[146:147], s[54:55], s[54:55] op_sel:[0,0,1] op_sel_hi:[1,0,1]\n\t"
      "v_rcp_f32 v8, v8\n\t"
      "v_pk_mul_f32 v[140:141], v[140:141], v[6:7]\n\t"
      "v_pk_fma_f32 v[22:23], v[148:149], s[54:55], s[54:55] op_sel:[0,0,1] op_sel_hi:[1,0,1]\n\t"
      "v_rcp_f32 v9, v9\n\t"
      "v_pk_fma_f32 v[24:25], v[150:151], s[54:55], s[54:55] op_sel:[0,0,1] op_sel_hi:[1,0,1]\n\t"
      "v_pk_mul_f32 v[142:143], v[142:143], v[8:9]\n\t"
      "v_mfma_f32_16x16x32_bf16 v[152:155], a[176:179], a[32:35], v[152:155]\n\t"
      "v_exp_f32 v18, v18 clamp\n\t"
      "v_mfma_f32_16x16x32_bf16 v[152:155], a[180:183], a[36:39], v[152:155]\n\t"
      "v_exp_f32 v19, v19 clamp\n\t"
      "v_mfma_f32_16x16x32_bf16 v[156:159], a[184:187], a[32:35], v[156:159]\n\t"
      "v_exp_f32 v20, v20 clamp\n\t"
      "v_mfma_f32_16x16x32_bf16 v[156:159], a[188:191], a[36:39], v[156:159]\n\t"
      "global_store_dwordx4 v26, v[128:131], s[24:25] offset:0 sc0 sc1\n\t"
      "v_exp_f32 v21, v21 clamp\n\t"
      "v_exp_f32 v22, v22 clamp\n\t"
      "v_exp_f32 v23, v23 clamp\n\t"
      "v_pk_fma_f32 v[18:19], v[18:19], s[58:59], s[58:59] op_sel:[0,0,1] op_sel_hi:[1,0,1]\n\t"
      "v_exp_f32 v24, v24 clamp\n\t"
      "v_pk_fma_f32 v[20:21], v[20:21], s[58:59], s[58:59] op_sel:[0,0,1] op_sel_hi:[1,0,1]\n\t"
      "v_exp_f32 v25, v25 clamp\n\t"
      "v_pk_fma_f32 v[22:23], v[22:23], s[58:59], s[58:59] op_sel:[0,0,1] op_sel_hi:[1,0,1]\n\t"
      "v_cvt_pk_bf16_f32 v136, v136, v137\n\t"
      "v_cvt_pk_bf16_f32 v137, v138, v139\n\t"
      "v_pk_fma_f32 v[24:25], v[24:25], s[58:59], s[58:59] op_sel:[0,0,1] op_sel_hi:[1,0,1]\n\t"
      "v_cvt_pk_bf16_f32 v138, v140, v141\n\t"
      "v_cvt_pk_bf16_f32 v139, v142, v143\n\t"
      "v_rcp_f32 v18, v18\n\t"
      "v_permlane16_swap_b32 v136, v138\n\t"
      "v_rcp_f32 v19, v19\n\t"
      "v_permlane16_swap_b32 v137, v139\n\t"
      "v_rcp_f32 v20, v20\n\t"
      "v_pk_mul_f32 v[144:145], v[144:145], v[18:19]\n\t"
      "v_rcp_f32 v21, v21\n\t"
      "v_pk_fma_f32 v[2:3], v[152:153], s[54:55], s[54:55] op_sel:[0,0,1] op_sel_hi:[1,0,1]\n\t"
      "v_rcp_f32 v22, v22\n\t"
      "v_pk_mul_f32 v[146:147], v[146:147], v[20:21]\n\t"
      "v_rcp_f32 v23, v23\n\t"
      "v_pk_fma_f32 v[4:5], v[154:155], s[54:55], s[54:55] op_sel:[0,0,1] op_sel_hi:[1,0,1]\n\t"
      "v_rcp_f32 v24, v24\n\t"
      "v_pk_mul_f32 v[148:149], v[148:149], v[22:23]\n\t"
      "v_pk_fma_f32 v[6:7], v[156:157], s[54:55], s[54:55] op_sel:[0,0,1] op_sel_hi:[1,0,1]\n\t"
      "v_rcp_f32 v25, v25\n\t"
      "v_pk_fma_f32 v[8:9], v[158:159], s[54:55], s[54:55] op_sel:[0,0,1] op_sel_hi:[1,0,1]\n\t"
      "v_pk_mul_f32 v[150:151], v[150:151], v[24:25]\n\t"
      "s_waitcnt vmcnt(20)\n\t"
      "v_mfma_f32_16x16x32_bf16 v[160:163], a[128:131], a[40:43], v[160:163]\n\t"
      "v_exp_f32 v2, v2 clamp\n\t"
      "v_mfma_f32_16x16x32_bf16 v[160:163], a[132:135], a[44:47], v[160:163]\n\t"
      "v_exp_f32 v3, v3 clamp\n\t"
      "v_mfma_f32_16x16x32_bf16 v[164:167], a[136:139], a[40:43], v[164:167]\n\t"
      "v_exp_f32 v4, v4 clamp\n\t"
      "v_mfma_f32_16x16x32_bf16 v[164:167], a[140:143], a[44:47], v[164:167]\n\t"
      "global_store_dwordx4 v26, v[136:139], s[24:25] offset:64 sc0 sc1\n\t"
      "v_exp_f32 v5, v5 clamp\n\t"
      "v_exp_f32 v6, v6 clamp\n\t"
      "v_exp_f32 v7, v7 clamp\n\t"
      "v_pk_fma_f32 v[2:3], v[2:3], s[58:59], s[58:59] op_sel:[0,0,1] op_sel_hi:[1,0,1]\n\t"
      "v_exp_f32 v8, v8 clamp\n\t"
      "v_pk_fma_f32 v[4:5], v[4:5], s[58:59], s[58:59] op_sel:[0,0,1] op_sel_hi:[1,0,1]\n\t"
      "v_exp_f32 v9, v9 clamp\n\t"
      "v_pk_fma_f32 v[6:7], v[6:7], s[58:59], s[58:59] op_sel:[0,0,1] op_sel_hi:[1,0,1]\n\t"
      "v_cvt_pk_bf16_f32 v144, v144, v145\n\t"
      "v_cvt_pk_bf16_f32 v145, v146, v147\n\t"
      "v_pk_fma_f32 v[8:9], v[8:9], s[58:59], s[58:59] op_sel:[0,0,1] op_sel_hi:[1,0,1]\n\t"
      "v_cvt_pk_bf16_f32 v146, v148, v149\n\t"
      "v_cvt_pk_bf16_f32 v147, v150, v151\n\t"
      "v_rcp_f32 v2, v2\n\t"
      "v_permlane16_swap_b32 v144, v146\n\t"
      "v_rcp_f32 v3, v3\n\t"
      "v_permlane16_swap_b32 v145, v147\n\t"
      "v_rcp_f32 v4, v4\n\t"
      "v_pk_mul_f32 v[152:153], v[152:153], v[2:3]\n\t"
      "v_rcp_f32 v5, v5\n\t"
      "v_pk_fma_f32 v[18:19], v[160:161], s[54:55], s[54:55] op_sel:[0,0,1] op_sel_hi:[1,0,1]\n\t"
      "v_rcp_f32 v6, v6\n\t"
      "v_pk_mul_f32 v[154:155], v[154:155], v[4:5]\n\t"
      "v_rcp_f32 v7, v7\n\t"
      "v_pk_fma_f32 v[20:21], v[162:163], s[54:55], s[54:55] op_sel:[0,0,1] op_sel_hi:[1,0,1]\n\t"
      "v_rcp_f32 v8, v8\n\t"
      "v_pk_mul_f32 v[156:157], v[156:157], v[6:7]\n\t"
      "v_pk_fma_f32 v[22:23], v[164:165], s[54:55], s[54:55] op_sel:[0,0,1] op_sel_hi:[1,0,1]\n\t"
      "v_rcp_f32 v9, v9\n\t"
      "v_pk_fma_f32 v[24:25], v[166:167], s[54:55], s[54:55] op_sel:[0,0,1] op_sel_hi:[1,0,1]\n\t"
      "v_pk_mul_f32 v[158:159], v[158:159], v[8:9]\n\t"
      "v_mfma_f32_16x16x32_bf16 v[168:171], a[144:147], a[40:43], v[168:171]\n\t"
      "v_exp_f32 v18, v18 clamp\n\t"
      "v_mfma_f32_16x16x32_bf16 v[168:171], a[148:151], a[44:47], v[168:171]\n\t"
      "v_exp_f32 v19, v19 clamp\n\t"
      "v_mfma_f32_16x16x32_bf16 v[172:175], a[152:155], a[40:43], v[172:175]\n\t"
      "v_exp_f32 v20, v20 clamp\n\t"
      "v_mfma_f32_16x16x32_bf16 v[172:175], a[156:159], a[44:47], v[172:175]\n\t"
      "global_store_dwordx4 v26, v[144:147], s[24:25] offset:128 sc0 sc1\n\t"
      "v_exp_f32 v21, v21 clamp\n\t"
      "v_exp_f32 v22, v22 clamp\n\t"
      "v_exp_f32 v23, v23 clamp\n\t"
      "v_pk_fma_f32 v[18:19], v[18:19], s[58:59], s[58:59] op_sel:[0,0,1] op_sel_hi:[1,0,1]\n\t"
      "v_exp_f32 v24, v24 clamp\n\t"
      "v_pk_fma_f32 v[20:21], v[20:21], s[58:59], s[58:59] op_sel:[0,0,1] op_sel_hi:[1,0,1]\n\t"
      "v_exp_f32 v25, v25 clamp\n\t"
      "v_pk_fma_f32 v[22:23], v[22:23], s[58:59], s[58:59] op_sel:[0,0,1] op_sel_hi:[1,0,1]\n\t"
      "v_cvt_pk_bf16_f32 v152, v152, v153\n\t"
      "v_cvt_pk_bf16_f32 v153, v154, v155\n\t"
      "v_pk_fma_f32 v[24:25], v[24:25], s[58:59], s[58:59] op_sel:[0,0,1] op_sel_hi:[1,0,1]\n\t"
      "v_cvt_pk_bf16_f32 v154, v156, v157\n\t"
      "v_cvt_pk_bf16_f32 v155, v158, v159\n\t"
      "v_rcp_f32 v18, v18\n\t"
      "v_permlane16_swap_b32 v152, v154\n\t"
      "v_rcp_f32 v19, v19\n\t"
      "v_permlane16_swap_b32 v153, v155\n\t"
      "v_rcp_f32 v20, v20\n\t"
      "v_pk_mul_f32 v[160:161], v[160:161], v[18:19]\n\t"
      "v_rcp_f32 v21, v21\n\t"
      "v_pk_fma_f32 v[2:3], v[168:169], s[54:55], s[54:55] op_sel:[0,0,1] op_sel_hi:[1,0,1]\n\t"
      "v_rcp_f32 v22, v22\n\t"
      "v_pk_mul_f32 v[162:163], v[162:163], v[20:21]\n\t"
      "v_rcp_f32 v23, v23\n\t"
      "v_pk_fma_f32 v[4:5], v[170:171], s[54:55], s[54:55] op_sel:[0,0,1] op_sel_hi:[1,0,1]\n\t"
      "v_rcp_f32 v24, v24\n\t"
      "v_pk_mul_f32 v[164:165], v[164:165], v[22:23]\n\t"
      "v_pk_fma_f32 v[6:7], v[172:173], s[54:55], s[54:55] op_sel:[0,0,1] op_sel_hi:[1,0,1]\n\t"
      "v_rcp_f32 v25, v25\n\t"
      "v_pk_fma_f32 v[8:9], v[174:175], s[54:55], s[54:55] op_sel:[0,0,1] op_sel_hi:[1,0,1]\n\t"
      "v_pk_mul_f32 v[166:167], v[166:167], v[24:25]\n\t"
      "v_mfma_f32_16x16x32_bf16 v[176:179], a[160:163], a[40:43], v[176:179]\n\t"
      "v_exp_f32 v2, v2 clamp\n\t"
      "v_mfma_f32_16x16x32_bf16 v[176:179], a[164:167], a[44:47], v[176:179]\n\t"
      "v_exp_f32 v3, v3 clamp\n\t"
      "v_mfma_f32_16x16x32_bf16 v[180:183], a[168:171], a[40:43], v[180:183]\n\t"
      "v_exp_f32 v4, v4 clamp\n\t"
      "v_mfma_f32_16x16x32_bf16 v[180:183], a[172:175], a[44:47], v[180:183]\n\t"
      "global_store_dwordx4 v26, v[152:155], s[24:25] offset:192 sc0 sc1\n\t"
      "v_exp_f32 v5, v5 clamp\n\t"
      "v_exp_f32 v6, v6 clamp\n\t"
      "v_exp_f32 v7, v7 clamp\n\t"
      "v_pk_fma_f32 v[2:3], v[2:3], s[58:59], s[58:59] op_sel:[0,0,1] op_sel_hi:[1,0,1]\n\t"
      "v_exp_f32 v8, v8 clamp\n\t"
      "v_pk_fma_f32 v[4:5], v[4:5], s[58:59], s[58:59] op_sel:[0,0,1] op_sel_hi:[1,0,1]\n\t"
      "v_exp_f32 v9, v9 clamp\n\t"
      "v_pk_fma_f32 v[6:7], v[6:7], s[58:59], s[58:59] op_sel:[0,0,1] op_sel_hi:[1,0,1]\n\t"
      "v_cvt_pk_bf16_f32 v160, v160, v161\n\t"
      "v_cvt_pk_bf16_f32 v161, v162, v163\n\t"
      "v_pk_fma_f32 v[8:9], v[8:9], s[58:59], s[58:59] op_sel:[0,0,1] op_sel_hi:[1,0,1]\n\t"
      "v_cvt_pk_bf16_f32 v162, v164, v165\n\t"
      "v_cvt_pk_bf16_f32 v163, v166, v167\n\t"
      "v_rcp_f32 v2, v2\n\t"
      "v_permlane16_swap_b32 v160, v162\n\t"
      "v_rcp_f32 v3, v3\n\t"
      "v_permlane16_swap_b32 v161, v163\n\t"
      "v_rcp_f32 v4, v4\n\t"
      "v_pk_mul_f32 v[168:169], v[168:169], v[2:3]\n\t"
      "v_rcp_f32 v5, v5\n\t"
      "v_pk_fma_f32 v[18:19], v[176:177], s[54:55], s[54:55] op_sel:[0,0,1] op_sel_hi:[1,0,1]\n\t"
      "v_rcp_f32 v6, v6\n\t"
      "v_pk_mul_f32 v[170:171], v[170:171], v[4:5]\n\t"
      "v_rcp_f32 v7, v7\n\t"
      "v_pk_fma_f32 v[20:21], v[178:179], s[54:55], s[54:55] op_sel:[0,0,1] op_sel_hi:[1,0,1]\n\t"
      "v_rcp_f32 v8, v8\n\t"
      "v_pk_mul_f32 v[172:173], v[172:173], v[6:7]\n\t"
      "v_pk_fma_f32 v[22:23], v[180:181], s[54:55], s[54:55] op_sel:[0,0,1] op_sel_hi:[1,0,1]\n\t"
      "v_rcp_f32 v9, v9\n\t"
      "v_pk_fma_f32 v[24:25], v[182:183], s[54:55], s[54:55] op_sel:[0,0,1] op_sel_hi:[1,0,1]\n\t"
      "v_pk_mul_f32 v[174:175], v[174:175], v[8:9]\n\t"
      "v_mfma_f32_16x16x32_bf16 v[184:187], a[176:179], a[40:43], v[184:187]\n\t"
      "v_exp_f32 v18, v18 clamp\n\t"
      "v_mfma_f32_16x16x32_bf16 v[184:187], a[180:183], a[44:47], v[184:187]\n\t"
      "v_exp_f32 v19, v19 clamp\n\t"
      "v_mfma_f32_16x16x32_bf16 v[188:191], a[184:187], a[40:43], v[188:191]\n\t"
      "v_exp_f32 v20, v20 clamp\n\t"
      "v_mfma_f32_16x16x32_bf16 v[188:191], a[188:191], a[44:47], v[188:191]\n\t"
      "global_store_dwordx4 v26, v[160:163], s[26:27] offset:0 sc0 sc1\n\t"
      "v_exp_f32 v21, v21 clamp\n\t"
      "v_exp_f32 v22, v22 clamp\n\t"
      "v_exp_f32 v23, v23 clamp\n\t"
      "v_pk_fma_f32 v[18:19], v[18:19], s[58:59], s[58:59] op_sel:[0,0,1] op_sel_hi:[1,0,1]\n\t"
      "v_exp_f32 v24, v24 clamp\n\t"
      "v_pk_fma_f32 v[20:21], v[20:21], s[58:59], s[58:59] op_sel:[0,0,1] op_sel_hi:[1,0,1]\n\t"
      "v_exp_f32 v25, v25 clamp\n\t"
      "v_pk_fma_f32 v[22:23], v[22:23], s[58:59], s[58:59] op_sel:[0,0,1] op_sel_hi:[1,0,1]\n\t"
      "v_cvt_pk_bf16_f32 v168, v168, v169\n\t"
      "v_cvt_pk_bf16_f32 v169, v170, v171\n\t"
      "v_pk_fma_f32 v[24:25], v[24:25], s[58:59], s[58:59] op_sel:[0,0,1] op_sel_hi:[1,0,1]\n\t"
      "v_cvt_pk_bf16_f32 v170, v172, v173\n\t"
      "v_cvt_pk_bf16_f32 v171, v174, v175\n\t"
      "v_rcp_f32 v18, v18\n\t"
      "v_permlane16_swap_b32 v168, v170\n\t"
      "v_rcp_f32 v19, v19\n\t"
      "v_permlane16_swap_b32 v169, v171\n\t"
      "v_rcp_f32 v20, v20\n\t"
      "v_pk_mul_f32 v[176:177], v[176:177], v[18:19]\n\t"
      "v_rcp_f32 v21, v21\n\t"
      "v_pk_fma_f32 v[2:3], v[184:185], s[54:55], s[54:55] op_sel:[0,0,1] op_sel_hi:[1,0,1]\n\t"
      "v_rcp_f32 v22, v22\n\t"
      "v_pk_mul_f32 v[178:179], v[178:179], v[20:21]\n\t"
      "v_rcp_f32 v23, v23\n\t"
      "v_pk_fma_f32 v[4:5], v[186:187], s[54:55], s[54:55] op_sel:[0,0,1] op_sel_hi:[1,0,1]\n\t"
      "v_rcp_f32 v24, v24\n\t"
      "v_pk_mul_f32 v[180:181], v[180:181], v[22:23]\n\t"
      "v_pk_fma_f32 v[6:7], v[188:189], s[54:55], s[54:55] op_sel:[0,0,1] op_sel_hi:[1,0,1]\n\t"
      "v_rcp_f32 v25, v25\n\t"
      "v_pk_fma_f32 v[8:9], v[190:191], s[54:55], s[54:55] op_sel:[0,0,1] op_sel_hi:[1,0,1]\n\t"
      "v_pk_mul_f32 v[182:183], v[182:183], v[24:25]\n\t"
      "s_waitcnt vmcnt(20)\n\t"
      "v_mfma_f32_16x16x32_bf16 v[192:195], a[128:131], a[48:51], v[192:195]\n\t"
      "v_exp_f32 v2, v2 clamp\n\t"
      "v_mfma_f32_16x16x32_bf16 v[192:195], a[132:135], a[52:55], v[192:195]\n\t"
      "v_exp_f32 v3, v3 clamp\n\t"
      "v_mfma_f32_16x16x32_bf16 v[196:199], a[136:139], a[48:51], v[196:199]\n\t"
      "v_exp_f32 v4, v4 clamp\n\t"
      "v_mfma_f32_16x16x32_bf16 v[196:199], a[140:143], a[52:55], v[196:199]\n\t"
      "global_store_dwordx4 v26, v[168:171], s[26:27] offset:64 sc0 sc1\n\t"
      "v_exp_f32 v5, v5 clamp\n\t"
      "v_exp_f32 v6, v6 clamp\n\t"
      "v_exp_f32 v7, v7 clamp\n\t"
      "v_pk_fma_f32 v[2:3], v[2:3], s[58:59], s[58:59] op_sel:[0,0,1] op_sel_hi:[1,0,1]\n\t"
      "v_exp_f32 v8, v8 clamp\n\t"
      "v_pk_fma_f32 v[4:5], v[4:5], s[58:59], s[58:59] op_sel:[0,0,1] op_sel_hi:[1,0,1]\n\t"
      "v_exp_f32 v9, v9 clamp\n\t"
      "v_pk_fma_f32 v[6:7], v[6:7], s[58:59], s[58:59] op_sel:[0,0,1] op_sel_hi:[1,0,1]\n\t"
      "v_cvt_pk_bf16_f32 v176, v176, v177\n\t"
      "v_cvt_pk_bf16_f32 v177, v178, v179\n\t"
      "v_pk_fma_f32 v[8:9], v[8:9], s[58:59], s[58:59] op_sel:[0,0,1] op_sel_hi:[1,0,1]\n\t"
      "v_cvt_pk_bf16_f32 v178, v180, v181\n\t"
      "v_cvt_pk_bf16_f32 v179, v182, v183\n\t"
      "v_rcp_f32 v2, v2\n\t"
      "v_permlane16_swap_b32 v176, v178\n\t"
      "v_rcp_f32 v3, v3\n\t"
      "v_permlane16_swap_b32 v177, v179\n\t"
      "v_rcp_f32 v4, v4\n\t"
      "v_pk_mul_f32 v[184:185], v[184:185], v[2:3]\n\t"
      "v_rcp_f32 v5, v5\n\t"
      "v_pk_fma_f32 v[18:19], v[192:193], s[54:55], s[54:55] op_sel:[0,0,1] op_sel_hi:[1,0,1]\n\t"
      "v_rcp_f32 v6, v6\n\t"
      "v_pk_mul_f32 v[186:187], v[186:187], v[4:5]\n\t"
      "v_rcp_f32 v7, v7\n\t"
      "v_pk_fma_f32 v[20:21], v[194:195], s[54:55], s[54:55] op_sel:[0,0,1] op_sel_hi:[1,0,1]\n\t"
      "v_rcp_f32 v8, v8\n\t"
      "v_pk_mul_f32 v[188:189], v[188:189], v[6:7]\n\t"
      "v_pk_fma_f32 v[22:23], v[196:197], s[54:55], s[54:55] op_sel:[0,0,1] op_sel_hi:[1,0,1]\n\t"
      "v_rcp_f32 v9, v9\n\t"
      "v_pk_fma_f32 v[24:25], v[198:199], s[54:55], s[54:55] op_sel:[0,0,1] op_sel_hi:[1,0,1]\n\t"
      "v_pk_mul_f32 v[190:191], v[190:191], v[8:9]\n\t"
      "v_mfma_f32_16x16x32_bf16 v[200:203], a[144:147], a[48:51], v[200:203]\n\t"
      "v_exp_f32 v18, v18 clamp\n\t"
      "v_mfma_f32_16x16x32_bf16 v[200:203], a[148:151], a[52:55], v[200:203]\n\t"
      "v_exp_f32 v19, v19 clamp\n\t"
      "v_mfma_f32_16x16x32_bf16 v[204:207], a[152:155], a[48:51], v[204:207]\n\t"
      "v_exp_f32 v20, v20 clamp\n\t"
      "v_mfma_f32_16x16x32_bf16 v[204:207], a[156:159], a[52:55], v[204:207]\n\t"
      "global_store_dwordx4 v26, v[176:179], s[26:27] offset:128 sc0 sc1\n\t"
      "v_exp_f32 v21, v21 clamp\n\t"
      "v_exp_f32 v22, v22 clamp\n\t"
      "v_exp_f32 v23, v23 clamp\n\t"
      "v_pk_fma_f32 v[18:19], v[18:19], s[58:59], s[58:59] op_sel:[0,0,1] op_sel_hi:[1,0,1]\n\t"
      "v_exp_f32 v24, v24 clamp\n\t"
      "v_pk_fma_f32 v[20:21], v[20:21], s[58:59], s[58:59] op_sel:[0,0,1] op_sel_hi:[1,0,1]\n\t"
      "v_exp_f32 v25, v25 clamp\n\t"
      "v_pk_fma_f32 v[22:23], v[22:23], s[58:59], s[58:59] op_sel:[0,0,1] op_sel_hi:[1,0,1]\n\t"
      "v_cvt_pk_bf16_f32 v184, v184, v185\n\t"
      "v_cvt_pk_bf16_f32 v185, v186, v187\n\t"
      "v_pk_fma_f32 v[24:25], v[24:25], s[58:59], s[58:59] op_sel:[0,0,1] op_sel_hi:[1,0,1]\n\t"
      "v_cvt_pk_bf16_f32 v186, v188, v189\n\t"
      "v_cvt_pk_bf16_f32 v187, v190, v191\n\t"
      "v_rcp_f32 v18, v18\n\t"
      "v_permlane16_swap_b32 v184, v186\n\t"
      "v_rcp_f32 v19, v19\n\t"
      "v_permlane16_swap_b32 v185, v187\n\t"
      "v_rcp_f32 v20, v20\n\t"
      "v_pk_mul_f32 v[192:193], v[192:193], v[18:19]\n\t"
      "v_rcp_f32 v21, v21\n\t"
      "v_pk_fma_f32 v[2:3], v[200:201], s[54:55], s[54:55] op_sel:[0,0,1] op_sel_hi:[1,0,1]\n\t"
      "v_rcp_f32 v22, v22\n\t"
      "v_pk_mul_f32 v[194:195], v[194:195], v[20:21]\n\t"
      "v_rcp_f32 v23, v23\n\t"
      "v_pk_fma_f32 v[4:5], v[202:203], s[54:55], s[54:55] op_sel:[0,0,1] op_sel_hi:[1,0,1]\n\t"
      "v_rcp_f32 v24, v24\n\t"
      "v_pk_mul_f32 v[196:197], v[196:197], v[22:23]\n\t"
      "v_pk_fma_f32 v[6:7], v[204:205], s[54:55], s[54:55] op_sel:[0,0,1] op_sel_hi:[1,0,1]\n\t"
      "v_rcp_f32 v25, v25\n\t"
      "v_pk_fma_f32 v[8:9], v[206:207], s[54:55], s[54:55] op_sel:[0,0,1] op_sel_hi:[1,0,1]\n\t"
      "v_pk_mul_f32 v[198:199], v[198:199], v[24:25]\n\t"
      "v_mfma_f32_16x16x32_bf16 v[208:211], a[160:163], a[48:51], v[208:211]\n\t"
      "v_exp_f32 v2, v2 clamp\n\t"
      "v_mfma_f32_16x16x32_bf16 v[208:211], a[164:167], a[52:55], v[208:211]\n\t"
      "v_exp_f32 v3, v3 clamp\n\t"
      "v_mfma_f32_16x16x32_bf16 v[212:215], a[168:171], a[48:51], v[212:215]\n\t"
      "v_exp_f32 v4, v4 clamp\n\t"
      "v_mfma_f32_16x16x32_bf16 v[212:215], a[172:175], a[52:55], v[212:215]\n\t"
      "global_store_dwordx4 v26, v[184:187], s[26:27] offset:192 sc0 sc1\n\t"
      "v_exp_f32 v5, v5 clamp\n\t"
      "v_exp_f32 v6, v6 clamp\n\t"
      "v_exp_f32 v7, v7 clamp\n\t"
      "v_pk_fma_f32 v[2:3], v[2:3], s[58:59], s[58:59] op_sel:[0,0,1] op_sel_hi:[1,0,1]\n\t"
      "v_exp_f32 v8, v8 clamp\n\t"
      "v_pk_fma_f32 v[4:5], v[4:5], s[58:59], s[58:59] op_sel:[0,0,1] op_sel_hi:[1,0,1]\n\t"
      "v_exp_f32 v9, v9 clamp\n\t"
      "v_pk_fma_f32 v[6:7], v[6:7], s[58:59], s[58:59] op_sel:[0,0,1] op_sel_hi:[1,0,1]\n\t"
      "v_cvt_pk_bf16_f32 v192, v192, v193\n\t"
      "v_cvt_pk_bf16_f32 v193, v194, v195\n\t"
      "v_pk_fma_f32 v[8:9], v[8:9], s[58:59], s[58:59] op_sel:[0,0,1] op_sel_hi:[1,0,1]\n\t"
      "v_cvt_pk_bf16_f32 v194, v196, v197\n\t"
      "v_cvt_pk_bf16_f32 v195, v198, v199\n\t"
      "v_rcp_f32 v2, v2\n\t"
      "v_permlane16_swap_b32 v192, v194\n\t"
      "v_rcp_f32 v3, v3\n\t"
      "v_permlane16_swap_b32 v193, v195\n\t"
      "v_rcp_f32 v4, v4\n\t"
      "v_pk_mul_f32 v[200:201], v[200:201], v[2:3]\n\t"
      "v_rcp_f32 v5, v5\n\t"
      "v_pk_fma_f32 v[18:19], v[208:209], s[54:55], s[54:55] op_sel:[0,0,1] op_sel_hi:[1,0,1]\n\t"
      "v_rcp_f32 v6, v6\n\t"
      "v_pk_mul_f32 v[202:203], v[202:203], v[4:5]\n\t"
      "v_rcp_f32 v7, v7\n\t"
      "v_pk_fma_f32 v[20:21], v[210:211], s[54:55], s[54:55] op_sel:[0,0,1] op_sel_hi:[1,0,1]\n\t"
      "v_rcp_f32 v8, v8\n\t"
      "v_pk_mul_f32 v[204:205], v[204:205], v[6:7]\n\t"
      "v_pk_fma_f32 v[22:23], v[212:213], s[54:55], s[54:55] op_sel:[0,0,1] op_sel_hi:[1,0,1]\n\t"
      "v_rcp_f32 v9, v9\n\t"
      "v_pk_fma_f32 v[24:25], v[214:215], s[54:55], s[54:55] op_sel:[0,0,1] op_sel_hi:[1,0,1]\n\t"
      "v_pk_mul_f32 v[206:207], v[206:207], v[8:9]\n\t"
      "v_mfma_f32_16x16x32_bf16 v[216:219], a[176:179], a[48:51], v[216:219]\n\t"
      "v_exp_f32 v18, v18 clamp\n\t"
      "v_mfma_f32_16x16x32_bf16 v[216:219], a[180:183], a[52:55], v[216:219]\n\t"
      "v_exp_f32 v19, v19 clamp\n\t"
      "v_mfma_f32_16x16x32_bf16 v[220:223], a[184:187], a[48:51], v[220:223]\n\t"
      "v_exp_f32 v20, v20 clamp\n\t"
      "v_mfma_f32_16x16x32_bf16 v[220:223], a[188:191], a[52:55], v[220:223]\n\t"
      "global_store_dwordx4 v26, v[192:195], s[28:29] offset:0 sc0 sc1\n\t"
      "v_exp_f32 v21, v21 clamp\n\t"
      "v_exp_f32 v22, v22 clamp\n\t"
      "v_exp_f32 v23, v23 clamp\n\t"
      "v_pk_fma_f32 v[18:19], v[18:19], s[58:59], s[58:59] op_sel:[0,0,1] op_sel_hi:[1,0,1]\n\t"
      "v_exp_f32 v24, v24 clamp\n\t"
      "v_pk_fma_f32 v[20:21], v[20:21], s[58:59], s[58:59] op_sel:[0,0,1] op_sel_hi:[1,0,1]\n\t"
      "v_exp_f32 v25, v25 clamp\n\t"
      "v_pk_fma_f32 v[22:23], v[22:23], s[58:59], s[58:59] op_sel:[0,0,1] op_sel_hi:[1,0,1]\n\t"
      "v_cvt_pk_bf16_f32 v200, v200, v201\n\t"
      "v_cvt_pk_bf16_f32 v201, v202, v203\n\t"
      "v_pk_fma_f32 v[24:25], v[24:25], s[58:59], s[58:59] op_sel:[0,0,1] op_sel_hi:[1,0,1]\n\t"
      "v_cvt_pk_bf16_f32 v202, v204, v205\n\t"
      "v_cvt_pk_bf16_f32 v203, v206, v207\n\t"
      "v_rcp_f32 v18, v18\n\t"
      "v_permlane16_swap_b32 v200, v202\n\t"
      "v_rcp_f32 v19, v19\n\t"
      "v_permlane16_swap_b32 v201, v203\n\t"
      "v_rcp_f32 v20, v20\n\t"
      "v_pk_mul_f32 v[208:209], v[208:209], v[18:19]\n\t"
      "v_rcp_f32 v21, v21\n\t"
      "v_pk_fma_f32 v[2:3], v[216:217], s[54:55], s[54:55] op_sel:[0,0,1] op_sel_hi:[1,0,1]\n\t"
      "v_rcp_f32 v22, v22\n\t"
      "v_pk_mul_f32 v[210:211], v[210:211], v[20:21]\n\t"
      "v_rcp_f32 v23, v23\n\t"
      "v_pk_fma_f32 v[4:5], v[218:219], s[54:55], s[54:55] op_sel:[0,0,1] op_sel_hi:[1,0,1]\n\t"
      "v_rcp_f32 v24, v24\n\t"
      "v_pk_mul_f32 v[212:213], v[212:213], v[22:23]\n\t"
      "v_pk_fma_f32 v[6:7], v[220:221], s[54:55], s[54:55] op_sel:[0,0,1] op_sel_hi:[1,0,1]\n\t"
      "v_rcp_f32 v25, v25\n\t"
      "v_pk_fma_f32 v[8:9], v[222:223], s[54:55], s[54:55] op_sel:[0,0,1] op_sel_hi:[1,0,1]\n\t"
      "v_pk_mul_f32 v[214:215], v[214:215], v[24:25]\n\t"
      "s_waitcnt vmcnt(20)\n\t"
      "v_mfma_f32_16x16x32_bf16 v[224:227], a[128:131], a[56:59], v[224:227]\n\t"
      "v_exp_f32 v2, v2 clamp\n\t"
      "v_mfma_f32_16x16x32_bf16 v[224:227], a[132:135], a[60:63], v[224:227]\n\t"
      "v_exp_f32 v3, v3 clamp\n\t"
      "v_mfma_f32_16x16x32_bf16 v[228:231], a[136:139], a[56:59], v[228:231]\n\t"
      "v_exp_f32 v4, v4 clamp\n\t"
      "v_mfma_f32_16x16x32_bf16 v[228:231], a[140:143], a[60:63], v[228:231]\n\t"
      "global_store_dwordx4 v26, v[200:203], s[28:29] offset:64 sc0 sc1\n\t"
      "v_exp_f32 v5, v5 clamp\n\t"
      "v_exp_f32 v6, v6 clamp\n\t"
      "v_exp_f32 v7, v7 clamp\n\t"
      "v_pk_fma_f32 v[2:3], v[2:3], s[58:59], s[58:59] op_sel:[0,0,1] op_sel_hi:[1,0,1]\n\t"
      "v_exp_f32 v8, v8 clamp\n\t"
      "v_pk_fma_f32 v[4:5], v[4:5], s[58:59], s[58:59] op_sel:[0,0,1] op_sel_hi:[1,0,1]\n\t"
      "v_exp_f32 v9, v9 clamp\n\t"
      "v_pk_fma_f32 v[6:7], v[6:7], s[58:59], s[58:59] op_sel:[0,0,1] op_sel_hi:[1,0,1]\n\t"
      "v_cvt_pk_bf16_f32 v208, v208, v209\n\t"
      "v_cvt_pk_bf16_f32 v209, v210, v211\n\t"
      "v_pk_fma_f32 v[8:9], v[8:9], s[58:59], s[58:59] op_sel:[0,0,1] op_sel_hi:[1,0,1]\n\t"
      "v_cvt_pk_bf16_f32 v210, v212, v213\n\t"
      "v_cvt_pk_bf16_f32 v211, v214, v215\n\t"
      "v_rcp_f32 v2, v2\n\t"
      "v_permlane16_swap_b32 v208, v210\n\t"
      "v_rcp_f32 v3, v3\n\t"
      "v_permlane16_swap_b32 v209, v211\n\t"
      "v_rcp_f32 v4, v4\n\t"
      "v_pk_mul_f32 v[216:217], v[216:217], v[2:3]\n\t"
      "v_rcp_f32 v5, v5\n\t"
      "v_pk_fma_f32 v[18:19], v[224:225], s[54:55], s[54:55] op_sel:[0,0,1] op_sel_hi:[1,0,1]\n\t"
      "v_rcp_f32 v6, v6\n\t"
      "v_pk_mul_f32 v[218:219], v[218:219], v[4:5]\n\t"
      "v_rcp_f32 v7, v7\n\t"
      "v_pk_fma_f32 v[20:21], v[226:227], s[54:55], s[54:55] op_sel:[0,0,1] op_sel_hi:[1,0,1]\n\t"
      "v_rcp_f32 v8, v8\n\t"
      "v_pk_mul_f32 v[220:221], v[220:221], v[6:7]\n\t"
      "v_pk_fma_f32 v[22:23], v[228:229], s[54:55], s[54:55] op_sel:[0,0,1] op_sel_hi:[1,0,1]\n\t"
      "v_rcp_f32 v9, v9\n\t"
      "v_pk_fma_f32 v[24:25], v[230:231], s[54:55], s[54:55] op_sel:[0,0,1] op_sel_hi:[1,0,1]\n\t"
      "v_pk_mul_f32 v[222:223], v[222:223], v[8:9]\n\t"
      "v_mfma_f32_16x16x32_bf16 v[232:235], a[144:147], a[56:59], v[232:235]\n\t"
      "v_exp_f32 v18, v18 clamp\n\t"
      "v_mfma_f32_16x16x32_bf16 v[232:235], a[148:151], a[60:63], v[232:235]\n\t"
      "v_exp_f32 v19, v19 clamp\n\t"
      "v_mfma_f32_16x16x32_bf16 v[236:239], a[152:155], a[56:59], v[236:239]\n\t"
      "v_exp_f32 v20, v20 clamp\n\t"
      "v_mfma_f32_16x16x32_bf16 v[236:239], a[156:159], a[60:63], v[236:239]\n\t"
      "global_store_dwordx4 v26, v[208:211], s[28:29] offset:128 sc0 sc1\n\t"
      "v_exp_f32 v21, v21 clamp\n\t"
      "v_exp_f32 v22, v22 clamp\n\t"
      "v_exp_f32 v23, v23 clamp\n\t"
      "v_pk_fma_f32 v[18:19], v[18:19], s[58:59], s[58:59] op_sel:[0,0,1] op_sel_hi:[1,0,1]\n\t"
      "v_exp_f32 v24, v24 clamp\n\t"
      "v_pk_fma_f32 v[20:21], v[20:21], s[58:59], s[58:59] op_sel:[0,0,1] op_sel_hi:[1,0,1]\n\t"
      "v_exp_f32 v25, v25 clamp\n\t"
      "v_pk_fma_f32 v[22:23], v[22:23], s[58:59], s[58:59] op_sel:[0,0,1] op_sel_hi:[1,0,1]\n\t"
      "v_cvt_pk_bf16_f32 v216, v216, v217\n\t"
      "v_cvt_pk_bf16_f32 v217, v218, v219\n\t"
      "v_pk_fma_f32 v[24:25], v[24:25], s[58:59], s[58:59] op_sel:[0,0,1] op_sel_hi:[1,0,1]\n\t"
      "v_cvt_pk_bf16_f32 v218, v220, v221\n\t"
      "v_cvt_pk_bf16_f32 v219, v222, v223\n\t"
      "v_rcp_f32 v18, v18\n\t"
      "v_permlane16_swap_b32 v216, v218\n\t"
      "v_rcp_f32 v19, v19\n\t"
      "v_permlane16_swap_b32 v217, v219\n\t"
      "v_rcp_f32 v20, v20\n\t"
      "v_pk_mul_f32 v[224:225], v[224:225], v[18:19]\n\t"
      "v_rcp_f32 v21, v21\n\t"
      "v_pk_fma_f32 v[2:3], v[232:233], s[54:55], s[54:55] op_sel:[0,0,1] op_sel_hi:[1,0,1]\n\t"
      "v_rcp_f32 v22, v22\n\t"
      "v_pk_mul_f32 v[226:227], v[226:227], v[20:21]\n\t"
      "v_rcp_f32 v23, v23\n\t"
      "v_pk_fma_f32 v[4:5], v[234:235], s[54:55], s[54:55] op_sel:[0,0,1] op_sel_hi:[1,0,1]\n\t"
      "v_rcp_f32 v24, v24\n\t"
      "v_pk_mul_f32 v[228:229], v[228:229], v[22:23]\n\t"
      "v_pk_fma_f32 v[6:7], v[236:237], s[54:55], s[54:55] op_sel:[0,0,1] op_sel_hi:[1,0,1]\n\t"
      "v_rcp_f32 v25, v25\n\t"
      "v_pk_fma_f32 v[8:9], v[238:239], s[54:55], s[54:55] op_sel:[0,0,1] op_sel_hi:[1,0,1]\n\t"
      "v_pk_mul_f32 v[230:231], v[230:231], v[24:25]\n\t"
      "v_mfma_f32_16x16x32_bf16 v[240:243], a[160:163], a[56:59], v[240:243]\n\t"
      "v_exp_f32 v2, v2 clamp\n\t"
      "v_mfma_f32_16x16x32_bf16 v[240:243], a[164:167], a[60:63], v[240:243]\n\t"
      "v_exp_f32 v3, v3 clamp\n\t"
      "v_mfma_f32_16x16x32_bf16 v[244:247], a[168:171], a[56:59], v[244:247]\n\t"
      "v_exp_f32 v4, v4 clamp\n\t"
      "v_mfma_f32_16x16x32_bf16 v[244:247], a[172:175], a[60:63], v[244:247]\n\t"
      "global_store_dwordx4 v26, v[216:219], s[28:29] offset:192 sc0 sc1\n\t"
      "v_exp_f32 v5, v5 clamp\n\t"
      "v_exp_f32 v6, v6 clamp\n\t"
      "v_exp_f32 v7, v7 clamp\n\t"
      "v_pk_fma_f32 v[2:3], v[2:3], s[58:59], s[58:59] op_sel:[0,0,1] op_sel_hi:[1,0,1]\n\t"
      "v_exp_f32 v8, v8 clamp\n\t"
      "v_pk_fma_f32 v[4:5], v[4:5], s[58:59], s[58:59] op_sel:[0,0,1] op_sel_hi:[1,0,1]\n\t"
      "v_exp_f32 v9, v9 clamp\n\t"
      "v_pk_fma_f32 v[6:7], v[6:7], s[58:59], s[58:59] op_sel:[0,0,1] op_sel_hi:[1,0,1]\n\t"
      "v_cvt_pk_bf16_f32 v224, v224, v225\n\t"
      "v_cvt_pk_bf16_f32 v225, v226, v227\n\t"
      "v_pk_fma_f32 v[8:9], v[8:9], s[58:59], s[58:59] op_sel:[0,0,1] op_sel_hi:[1,0,1]\n\t"
      "v_cvt_pk_bf16_f32 v226, v228, v229\n\t"
      "v_cvt_pk_bf16_f32 v227, v230, v231\n\t"
      "v_rcp_f32 v2, v2\n\t"
      "v_permlane16_swap_b32 v224, v226\n\t"
      "v_rcp_f32 v3, v3\n\t"
      "v_permlane16_swap_b32 v225, v227\n\t"
      "v_rcp_f32 v4, v4\n\t"
      "v_pk_mul_f32 v[232:233], v[232:233], v[2:3]\n\t"
      "v_rcp_f32 v5, v5\n\t"
      "v_pk_fma_f32 v[18:19], v[240:241], s[54:55], s[54:55] op_sel:[0,0,1] op_sel_hi:[1,0,1]\n\t"
      "v_rcp_f32 v6, v6\n\t"
      "v_pk_mul_f32 v[234:235], v[234:235], v[4:5]\n\t"
      "v_rcp_f32 v7, v7\n\t"
      "v_pk_fma_f32 v[20:21], v[242:243], s[54:55], s[54:55] op_sel:[0,0,1] op_sel_hi:[1,0,1]\n\t"
      "v_rcp_f32 v8, v8\n\t"
      "v_pk_mul_f32 v[236:237], v[236:237], v[6:7]\n\t"
      "v_pk_fma_f32 v[22:23], v[244:245], s[54:55], s[54:55] op_sel:[0,0,1] op_sel_hi:[1,0,1]\n\t"
      "v_rcp_f32 v9, v9\n\t"
      "v_pk_fma_f32 v[24:25], v[246:247], s[54:55], s[54:55] op_sel:[0,0,1] op_sel_hi:[1,0,1]\n\t"
      "v_pk_mul_f32 v[238:239], v[238:239], v[8:9]\n\t"
      "v_mfma_f32_16x16x32_bf16 v[248:251], a[176:179], a[56:59], v[248:251]\n\t"
      "v_exp_f32 v18, v18 clamp\n\t"
      "v_mfma_f32_16x16x32_bf16 v[248:251], a[180:183], a[60:63], v[248:251]\n\t"
      "v_exp_f32 v19, v19 clamp\n\t"
      "v_mfma_f32_16x16x32_bf16 v[252:255], a[184:187], a[56:59], v[252:255]\n\t"
      "v_exp_f32 v20, v20 clamp\n\t"
      "v_mfma_f32_16x16x32_bf16 v[252:255], a[188:191], a[60:63], v[252:255]\n\t"
      "global_store_dwordx4 v26, v[224:227], s[30:31] offset:0 sc0 sc1\n\t"
      "v_exp_f32 v21, v21 clamp\n\t"
      "v_exp_f32 v22, v22 clamp\n\t"
      "v_exp_f32 v23, v23 clamp\n\t"
      "v_pk_fma_f32 v[18:19], v[18:19], s[58:59], s[58:59] op_sel:[0,0,1] op_sel_hi:[1,0,1]\n\t"
      "v_exp_f32 v24, v24 clamp\n\t"
      "v_pk_fma_f32 v[20:21], v[20:21], s[58:59], s[58:59] op_sel:[0,0,1] op_sel_hi:[1,0,1]\n\t"
      "v_exp_f32 v25, v25 clamp\n\t"
      "v_pk_fma_f32 v[22:23], v[22:23], s[58:59], s[58:59] op_sel:[0,0,1] op_sel_hi:[1,0,1]\n\t"
      "v_cvt_pk_bf16_f32 v232, v232, v233\n\t"
      "v_cvt_pk_bf16_f32 v233, v234, v235\n\t"
      "v_pk_fma_f32 v[24:25], v[24:25], s[58:59], s[58:59] op_sel:[0,0,1] op_sel_hi:[1,0,1]\n\t"
      "v_cvt_pk_bf16_f32 v234, v236, v237\n\t"
      "v_cvt_pk_bf16_f32 v235, v238, v239\n\t"
      "v_rcp_f32 v18, v18\n\t"
      "v_permlane16_swap_b32 v232, v234\n\t"
      "v_rcp_f32 v19, v19\n\t"
      "v_permlane16_swap_b32 v233, v235\n\t"
      "v_rcp_f32 v20, v20\n\t"
      "v_pk_mul_f32 v[240:241], v[240:241], v[18:19]\n\t"
      "v_rcp_f32 v21, v21\n\t"
      "v_pk_fma_f32 v[2:3], v[248:249], s[54:55], s[54:55] op_sel:[0,0,1] op_sel_hi:[1,0,1]\n\t"
      "v_rcp_f32 v22, v22\n\t"
      "v_pk_mul_f32 v[242:243], v[242:243], v[20:21]\n\t"
      "v_rcp_f32 v23, v23\n\t"
      "v_pk_fma_f32 v[4:5], v[250:251], s[54:55], s[54:55] op_sel:[0,0,1] op_sel_hi:[1,0,1]\n\t"
      "v_rcp_f32 v24, v24\n\t"
      "v_pk_mul_f32 v[244:245], v[244:245], v[22:23]\n\t"
      "v_pk_fma_f32 v[6:7], v[252:253], s[54:55], s[54:55] op_sel:[0,0,1] op_sel_hi:[1,0,1]\n\t"
      "v_rcp_f32 v25, v25\n\t"
      "v_pk_fma_f32 v[8:9], v[254:255], s[54:55], s[54:55] op_sel:[0,0,1] op_sel_hi:[1,0,1]\n\t"
      "v_pk_mul_f32 v[246:247], v[246:247], v[24:25]\n\t"
      "v_exp_f32 v2, v2 clamp\n\t"
      "global_store_dwordx4 v26, v[232:235], s[30:31] offset:64 sc0 sc1\n\t"
      "v_cvt_pk_bf16_f32 v240, v240, v241\n\t"
      "v_exp_f32 v3, v3 clamp\n\t"
      "v_cvt_pk_bf16_f32 v241, v242, v243\n\t"
      "v_exp_f32 v4, v4 clamp\n\t"
      "v_pk_fma_f32 v[2:3], v[2:3], s[58:59], s[58:59] op_sel:[0,0,1] op_sel_hi:[1,0,1]\n\t"
      "v_exp_f32 v5, v5 clamp\n\t"
      "v_cvt_pk_bf16_f32 v242, v244, v245\n\t"
      "v_exp_f32 v6, v6 clamp\n\t"
      "v_pk_fma_f32 v[4:5], v[4:5], s[58:59], s[58:59] op_sel:[0,0,1] op_sel_hi:[1,0,1]\n\t"
      "v_exp_f32 v7, v7 clamp\n\t"
      "v_cvt_pk_bf16_f32 v243, v246, v247\n\t"
      "v_exp_f32 v8, v8 clamp\n\t"
      "v_exp_f32 v9, v9 clamp\n\t"
      "v_pk_fma_f32 v[6:7], v[6:7], s[58:59], s[58:59] op_sel:[0,0,1] op_sel_hi:[1,0,1]\n\t"
      "v_pk_fma_f32 v[8:9], v[8:9], s[58:59], s[58:59] op_sel:[0,0,1] op_sel_hi:[1,0,1]\n\t"
      "v_rcp_f32 v2, v2\n\t"
      "v_permlane16_swap_b32 v240, v242\n\t"
      "v_rcp_f32 v3, v3\n\t"
      "v_permlane16_swap_b32 v241, v243\n\t"
      "v_rcp_f32 v4, v4\n\t"
      "v_pk_mul_f32 v[248:249], v[248:249], v[2:3]\n\t"
      "v_rcp_f32 v5, v5\n\t"
      "v_rcp_f32 v6, v6\n\t"
      "v_pk_mul_f32 v[250:251], v[250:251], v[4:5]\n\t"
      "global_store_dwordx4 v26, v[240:243], s[30:31] offset:128 sc0 sc1\n\t"
      "v_rcp_f32 v7, v7\n\t"
      "v_rcp_f32 v8, v8\n\t"
      "v_pk_mul_f32 v[252:253], v[252:253], v[6:7]\n\t"
      "v_rcp_f32 v9, v9\n\t"
      "s_nop 0\n\t"
      "v_pk_mul_f32 v[254:255], v[254:255], v[8:9]\n\t"
      "v_cvt_pk_bf16_f32 v248, v248, v249\n\t"
      "v_cvt_pk_bf16_f32 v249, v250, v251\n\t"
      "v_cvt_pk_bf16_f32 v250, v252, v253\n\t"
      "v_cvt_pk_bf16_f32 v251, v254, v255\n\t"
      "s_nop 1\n\t"
      "v_permlane16_swap_b32 v248, v250\n\t"
      "v_permlane16_swap_b32 v249, v251\n\t"
      "global_store_dwordx4 v26, v[248:251], s[30:31] offset:192 sc0 sc1\n\t"
      "END_%=:\n\t"
      "s_waitcnt vmcnt(0)\n\t"
      "s_branch DONE_%=\n\t"
      "SLOW2_%=:\n\t"
      "s_memrealtime s[34:35]\n\t"
      "s_waitcnt lgkmcnt(0)\n\t"
      "POLL2_%=:\n\t"
      "global_load_dword v2, v13, s[90:91] sc0 sc1\n\t"
      "global_load_dword v29, v27, s[40:41] sc1\n\t"
      "s_waitcnt vmcnt(1)\n\t"
      "v_cmp_gt_u32 vcc, s52, v2\n\t"
      "s_nop 4\n\t"
      "s_cbranch_vccz POLLOK2_%=\n\t"
      "s_memrealtime s[54:55]\n\t"
      "s_waitcnt vmcnt(0) lgkmcnt(0)\n\t"
      "v_readfirstlane_b32 s53, v29\n\t"
      "s_nop 3\n\t"
      "s_cmp_eq_u32 s53, s52\n\t"
      "s_cbranch_scc1 POLLTMO2_%=\n\t"
      "s_sub_u32 s54, s54, s34\n\t"
      "s_subb_u32 s55, s55, s35\n\t"
      "s_cmp_lg_u32 s55, 0\n\t"
      "s_cbranch_scc1 POLLTMO2_%=\n\t"
      "s_cmp_gt_u32 s54, 50000000\n\t"
      "s_cbranch_scc1 POLLTMO2_%=\n\t"
      "POLLMORE2_%=:\n\t"
      "s_sleep 1\n\t"
      "s_branch POLL2_%=\n\t"
      "POLLTMO2_%=:\n\t"
      "global_store_dword v27, v28, s[40:41] sc1\n\t"
      "v_mov_b32 v2, v28\n\t"
      "v_mov_b32 v3, v28\n\t"
      "POLLOK2_%=:\n\t"
      "s_branch BACK2_%=\n\t"
      "SLOW3_%=:\n\t"
      "s_memrealtime s[34:35]\n\t"
      "s_waitcnt lgkmcnt(0)\n\t"
      "POLL3_%=:\n\t"
      "global_load_dwordx2 v[2:3], v11, s[88:89] sc0 sc1\n\t"
      "global_load_dword v29, v27, s[40:41] sc1\n\t"
      "s_waitcnt vmcnt(1)\n\t"
      "v_min_u32 v2, v2, v3\n\t"
      "v_cmp_gt_u32 vcc, s52, v2\n\t"
      "s_nop 4\n\t"
      "s_cbranch_vccz POLLOK3_%=\n\t"
      "s_memrealtime s[54:55]\n\t"
      "s_waitcnt vmcnt(0) lgkmcnt(0)\n\t"
      "v_readfirstlane_b32 s53, v29\n\t"
      "s_nop 3\n\t"
      "s_cmp_eq_u32 s53, s52\n\t"
      "s_cbranch_scc1 POLLTMO3_%=\n\t"
      "s_sub_u32 s54, s54, s34\n\t"
      "s_subb_u32 s55, s55, s35\n\t"
      "s_cmp_lg_u32 s55, 0\n\t"
      "s_cbranch_scc1 POLLTMO3_%=\n\t"
      "s_cmp_gt_u32 s54, 50000000\n\t"
      "s_cbranch_scc1 POLLTMO3_%=\n\t"
      "POLLMORE3_%=:\n\t"
      "s_sleep 1\n\t"
      "s_branch POLL3_%=\n\t"
      "POLLTMO3_%=:\n\t"
      "global_store_dword v27, v28, s[40:41] sc1\n\t"
      "v_mov_b32 v2, v28\n\t"
      "v_mov_b32 v3, v28\n\t"
      "POLLOK3_%=:\n\t"
      "s_branch BACK3_%=\n\t"
      "DONE_%=:\n\t"
      :
      : "s"(__builtin_amdgcn_kernarg_segment_ptr()), "s"(blockIdx.x), "v"(threadIdx.x), "s"(ldsb)
      : "v1","v2","v3","v4","v5","v6","v7","v8","v9","v10","v11","v12","v13","v14","v15","v16","v17","v18","v19","v20","v21","v22","v23","v24",
        "v25","v26","v27","v28","v29","v30","v31","v32","v33","v34","v35","v36","v37","v38","v39","v40","v41","v42","v43","v44","v45","v46","v47","v48",
        "v49","v50","v51","v52","v53","v54","v55","v56","v57","v58","v59","v60","v61","v62","v63","v64","v65","v66","v67","v68","v69","v70","v71","v72",
        "v73","v74","v75","v76","v77","v78","v79","v80","v81","v82","v83","v84","v85","v86","v87","v88","v89","v90","v91","v92","v93","v94","v95","v96",
        "v97","v98","v99","v100","v101","v102","v103","v104","v105","v106","v107","v108","v109","v110","v111","v112","v113","v114","v115","v116","v117","v118","v119","v120",
        "v121","v122","v123","v124","v125","v126","v127","v128","v129","v130","v131","v132","v133","v134","v135","v136","v137","v138","v139","v140","v141","v142","v143","v144",
        "v145","v146","v147","v148","v149","v150","v151","v152","v153","v154","v155","v156","v157","v158","v159","v160","v161","v162","v163","v164","v165","v166","v167","v168",
        "v169","v170","v171","v172","v173","v174","v175","v176","v177","v178","v179","v180","v181","v182","v183","v184","v185","v186","v187","v188","v189","v190","v191","v192",
        "v193","v194","v195","v196","v197","v198","v199","v200","v201","v202","v203","v204","v205","v206","v207","v208","v209","v210","v211","v212","v213","v214","v215","v216",
        "v217","v218","v219","v220","v221","v222","v223","v224","v225","v226","v227","v228","v229","v230","v231","v232","v233","v234","v235","v236","v237","v238","v239","v240",
        "v241","v242","v243","v244","v245","v246","v247","v248","v249","v250","v251","v252","v253","v254","v255","a0","a1","a2","a3","a4","a5","a6","a7","a8",
        "a9","a10","a11","a12","a13","a14","a15","a16","a17","a18","a19","a20","a21","a22","a23","a24","a25","a26","a27","a28","a29","a30","a31","a32",
        "a33","a34","a35","a36","a37","a38","a39","a40","a41","a42","a43","a44","a45","a46","a47","a48","a49","a50","a51","a52","a53","a54","a55","a56",
        "a57","a58","a59","a60","a61","a62","a63","a64","a65","a66","a67","a68","a69","a70","a71","a72","a73","a74","a75","a76","a77","a78","a79","a80",
        "a81","a82","a83","a84","a85","a86","a87","a88","a89","a90","a91","a92","a93","a94","a95","a96","a97","a98","a99","a100","a101","a102","a103","a104",
        "a105","a106","a107","a108","a109","a110","a111","a112","a113","a114","a115","a116","a117","a118","a119","a120","a121","a122","a123","a124","a125","a126","a127","a128",
        "a129","a130","a131","a132","a133","a134","a135","a136","a137","a138","a139","a140","a141","a142","a143","a144","a145","a146","a147","a148","a149","a150","a151","a152",
        "a153","a154","a155","a156","a157","a158","a159","a160","a161","a162","a163","a164","a165","a166","a167","a168","a169","a170","a171","a172","a173","a174","a175","a176",
        "a177","a178","a179","a180","a181","a182","a183","a184","a185","a186","a187","a188","a189","a190","a191","a192","a193","a194","a195","a196","a197","a198","a199","a200",
        "a201","a202","a203","a204","a205","a206","a207","a208","a209","a210","a211","a212","a213","a214","a215","a216","a217","a218","a219","a220","a221","a222","a223","a224",
        "a225","a226","a227","a228","a229","a230","a231","a232","a233","a234","a235","a236","a237","a238","a239","a240","a241","a242","a243","a244","a245","a246","a247","a248",
        "a249","a250","a251","a252","a253","a254","a255","s4","s5","s6","s7","s8","s9","s10","s11","s12","s13","s14","s15","s16","s17","s18","s19","s20",
        "s21","s22","s23","s24","s25","s26","s27","s28","s29","s30","s31","s32","s33","s34","s35","s36","s37","s38","s39","s40","s41","s42","s43","s44",
        "s45","s46","s47","s48","s49","s50","s51","s52","s53","s54","s55","s56","s57","s58","s59","s60","s61","s62","s63","s64","s65","s66","s67","s68",
        "s69","s70","s71","s72","s73","s74","s75","s76","s77","s78","s79","s80","s81","s82","s83","s84","s85","s86","s87","s88","s89","s90","s91","s92",
        "s93","s94","s95","s96","s97","s98","s99","scc","vcc","memory");
}

extern "C" void kernel_launch(int rank, int nranks, ncclComm_t comm, hipStream_t stream,
                              void* const* d_in, const int* in_sizes, int n_in,
                              void* d_out, int out_size, const int* in_dtypes, int out_dtype,
                              void* const* d_out_all, void* const* d_ws_all, size_t ws_bytes) {
  if (nranks != NR || (unsigned)rank >= (unsigned)NR || n_in != 2 || ws_bytes < WS_END ||
      in_sizes[0] != 512 * 4096 || in_sizes[1] != 4096 * 32768 || out_size != 4096 * 4096) {
    fprintf(stderr, "unexpected problem shape\n");
    return;
  }
  static bool attr_set = false;
  if (!attr_set) {
    HIP_CHECK(hipFuncSetAttribute((const void*)fused_ag_gemm_silu, hipFuncAttributeMaxDynamicSharedMemorySize, LDSB));
    attr_set = true;
  }
  Params p;
  p.x = (const char*)d_in[0]; p.w = (const char*)d_in[1] + (size_t)rank * 8192; p.out = (char*)d_out;
  p.myws = (char*)d_ws_all[rank];
  p.rank = rank; p.tab = 0;
  int grp[2][4] = {{0, 2, 3, 6}, {1, 4, 5, 7}};
  {
    int dev = -1, ndev = 0, n[2] = {0, 0}, t[2][NR];
    if (hipGetDevice(&dev) == hipSuccess && dev == rank && hipGetDeviceCount(&ndev) == hipSuccess && ndev == NR)
      for (int r = 0; r < NR; ++r) {
        char id[32] = {0}, path[96];
        int phys = -1;
        if (hipDeviceGetPCIBusId(id, sizeof id, r) != hipSuccess) break;
        for (char* c = id; *c; ++c) if (*c >= 'A' && *c <= 'F') *c += 'a' - 'A';
        snprintf(path, sizeof path, "/sys/bus/pci/devices/%s/xgmi_physical_id", id);
        if (FILE* f = fopen(path, "r")) { if (fscanf(f, "%d", &phys) != 1) phys = -1; fclose(f); }
        if ((unsigned)phys >= (unsigned)NR) break;
        const int s = ((phys + 1) >> 1) & 1;
        t[s][n[s]++] = r;
      }
    if (n[0] == 4 && n[1] == 4) for (int a = 0; a < 2; ++a) for (int b = 0; b < 4; ++b) grp[a][b] = t[a][b];
  }
  int g = 0, at = 0;
  for (int a = 0; a < 2; ++a) for (int b = 0; b < 4; ++b) if (grp[a][b] == rank) { g = a; at = b; }
  for (int r = 0; r < NR; ++r) {
    p.ws[r] = (char*)d_ws_all[r];
    int q = r % 4;
    if (r < 4 && OWNX >= 0) q = q == OWNX ? at : q == at ? OWNX : q;
    p.tab |= (unsigned)grp[r < 4 ? g : 1 - g][q] << (4 * r);
  }
  if (ROT) p.tab = ROT;
  p.rtab = RTAB;
  p.pad = 0; p.rot = (unsigned)((p.rtab >> (8 * rank)) & 63) * 128;
  fused_ag_gemm_silu<<<256, 256, LDSB, stream>>>(p);
  HIP_CHECK(hipGetLastError());
}
```
